```python
import numpy as np

kernel_name = "gpu_ag_gemm_m4096_k4096_n16384_bf16_qe4m3"


ABSMAX_RTOL = 0.09

COMPUTE_DTYPE = "bfloat16"
EPILOGUE = "quant_e4m3"
IN_QUANT = None
OUT_QUANT = "e4m3"
IN_DTYPES = {"x": "bfloat16", "w_mat": "bfloat16"}
WORLD_SIZE = 8

def _to_bf16(x):
    u = np.ascontiguousarray(x, dtype=np.float32).view(np.uint32)
    return ((u + 0x7FFF + ((u >> 16) & 1)) & 0xFFFF0000).view(np.float32)

def setup_inputs():

    x = _to_bf16(np.random.randn(4096, 4096).astype(np.float32))
    w_mat = _to_bf16((np.random.randn(4096, 16384) / np.sqrt(4096)).astype(np.float32))
    return {"x": x, "w_mat": w_mat}

def _snap_fp8(x, mant_bits, max_abs, min_normal):
    a = np.abs(x.astype(np.float64))
    _m, _e = np.frexp(a)
    q = np.where(a >= min_normal, np.ldexp(1.0, _e - 1 - mant_bits),
                 min_normal * 2.0 ** -mant_bits)
    snapped = np.minimum(np.round(a / q) * q, max_abs)
    return (np.sign(x) * snapped).astype(np.float32)
def _quant_dequant(y, amax):
    scale = amax / 448.0
    q = _snap_fp8(y / scale, 3, 448.0, 2.0 ** -6)
    return (q * scale).astype(np.float32)
def _epilogue(y):
    return _quant_dequant(y, np.abs(y).max())

def reference(x, w_mat):
    y = (x @ w_mat).astype(np.float32)
    return _epilogue(y)
```

```cpp
#include <hip/hip_runtime.h>
#include <rccl/rccl.h>
#include <cstdio>
#include <cstdint>
#include <cstddef>

typedef unsigned u32;
typedef unsigned long long u64;
typedef u32 u32x2 __attribute__((ext_vector_type(2)));
typedef u32 u32x3 __attribute__((ext_vector_type(3)));
typedef u32 u32x4 __attribute__((ext_vector_type(4)));
typedef float f32x32 __attribute__((ext_vector_type(32)));
typedef unsigned short u16x2 __attribute__((ext_vector_type(2)));

#define NPRE 12
#define SW0 262144
#define NPF 2u
#define XRED 1
#define PSW 2
#define GEMM_PRE \
  "s_nop 4\n" \
  "s_add_u32 m0, %[ma], 0\n" \
  "s_nop 0\n" \
  "buffer_load_dwordx4 %[vd0], %[wr], %[sw] offen offset:0 lds\n" \
  "buffer_load_dwordx4 %[vd1], %[wr], %[sw] offen offset:1024 lds\n" \
  "buffer_load_dwordx4 %[vd2], %[wr], %[sw] offen offset:2048 lds\n" \
  "buffer_load_dwordx4 %[vd3], %[wr], %[sw] offen offset:3072 lds\n" \
  "s_add_u32 m0, %[mb], 0\n" \
  "s_nop 0\n" \
  "buffer_load_dwordx4 %[vd4], %[wr], %[sw] offen lds\n" \
  "s_add_u32 %[sw], %[sw], 262144\n" \
  "s_add_u32 m0, %[ma], 16384\n" \
  "s_nop 0\n" \
  "buffer_load_dwordx4 %[vd0], %[wr], %[sw] offen offset:0 lds\n" \
  "buffer_load_dwordx4 %[vd1], %[wr], %[sw] offen offset:1024 lds\n" \
  "buffer_load_dwordx4 %[vd2], %[wr], %[sw] offen offset:2048 lds\n" \
  "buffer_load_dwordx4 %[vd3], %[wr], %[sw] offen offset:3072 lds\n" \
  "s_add_u32 m0, %[mb], 4096\n" \
  "s_nop 0\n" \
  "buffer_load_dwordx4 %[vd4], %[wr], %[sw] offen lds\n" \
  "buffer_load_dword %[dm], %[vpf], %[wr], %[po0] offen\n" \
  "buffer_load_dword %[dm], %[vpf], %[wr], %[po1] offen\n" \
  ""
#define GEMM_ASM \
  "s_mov_b64 s[64:65], 0\n" \
  "v_mov_b32 v96, 0\n" \
  "v_mov_b32 v97, 0\n" \
  "v_mov_b32 v98, 0\n" \
  "v_mov_b32 v99, 0\n" \
  "v_mov_b32 v100, 0\n" \
  "v_mov_b32 v101, 0\n" \
  "v_mov_b32 v102, 0\n" \
  "v_mov_b32 v103, 0\n" \
  "v_mov_b32 v104, 0\n" \
  "v_mov_b32 v105, 0\n" \
  "v_mov_b32 v106, 0\n" \
  "v_mov_b32 v107, 0\n" \
  "v_mov_b32 v108, 0\n" \
  "v_mov_b32 v109, 0\n" \
  "v_mov_b32 v110, 0\n" \
  "v_mov_b32 v111, 0\n" \
  "v_mov_b32 v112, 0\n" \
  "v_mov_b32 v113, 0\n" \
  "v_mov_b32 v114, 0\n" \
  "v_mov_b32 v115, 0\n" \
  "v_mov_b32 v116, 0\n" \
  "v_mov_b32 v117, 0\n" \
  "v_mov_b32 v118, 0\n" \
  "v_mov_b32 v119, 0\n" \
  "v_mov_b32 v120, 0\n" \
  "v_mov_b32 v121, 0\n" \
  "v_mov_b32 v122, 0\n" \
  "v_mov_b32 v123, 0\n" \
  "v_mov_b32 v124, 0\n" \
  "v_mov_b32 v125, 0\n" \
  "v_mov_b32 v126, 0\n" \
  "v_mov_b32 v127, 0\n" \
  "v_mov_b32 v128, 0\n" \
  "v_mov_b32 v129, 0\n" \
  "v_mov_b32 v130, 0\n" \
  "v_mov_b32 v131, 0\n" \
  "v_mov_b32 v132, 0\n" \
  "v_mov_b32 v133, 0\n" \
  "v_mov_b32 v134, 0\n" \
  "v_mov_b32 v135, 0\n" \
  "v_mov_b32 v136, 0\n" \
  "v_mov_b32 v137, 0\n" \
  "v_mov_b32 v138, 0\n" \
  "v_mov_b32 v139, 0\n" \
  "v_mov_b32 v140, 0\n" \
  "v_mov_b32 v141, 0\n" \
  "v_mov_b32 v142, 0\n" \
  "v_mov_b32 v143, 0\n" \
  "v_mov_b32 v144, 0\n" \
  "v_mov_b32 v145, 0\n" \
  "v_mov_b32 v146, 0\n" \
  "v_mov_b32 v147, 0\n" \
  "v_mov_b32 v148, 0\n" \
  "v_mov_b32 v149, 0\n" \
  "v_mov_b32 v150, 0\n" \
  "v_mov_b32 v151, 0\n" \
  "v_mov_b32 v152, 0\n" \
  "v_mov_b32 v153, 0\n" \
  "v_mov_b32 v154, 0\n" \
  "v_mov_b32 v155, 0\n" \
  "v_mov_b32 v156, 0\n" \
  "v_mov_b32 v157, 0\n" \
  "v_mov_b32 v158, 0\n" \
  "v_mov_b32 v159, 0\n" \
  "v_mov_b32 v160, 0\n" \
  "v_mov_b32 v161, 0\n" \
  "v_mov_b32 v162, 0\n" \
  "v_mov_b32 v163, 0\n" \
  "v_mov_b32 v164, 0\n" \
  "v_mov_b32 v165, 0\n" \
  "v_mov_b32 v166, 0\n" \
  "v_mov_b32 v167, 0\n" \
  "v_mov_b32 v168, 0\n" \
  "v_mov_b32 v169, 0\n" \
  "v_mov_b32 v170, 0\n" \
  "v_mov_b32 v171, 0\n" \
  "v_mov_b32 v172, 0\n" \
  "v_mov_b32 v173, 0\n" \
  "v_mov_b32 v174, 0\n" \
  "v_mov_b32 v175, 0\n" \
  "v_mov_b32 v176, 0\n" \
  "v_mov_b32 v177, 0\n" \
  "v_mov_b32 v178, 0\n" \
  "v_mov_b32 v179, 0\n" \
  "v_mov_b32 v180, 0\n" \
  "v_mov_b32 v181, 0\n" \
  "v_mov_b32 v182, 0\n" \
  "v_mov_b32 v183, 0\n" \
  "v_mov_b32 v184, 0\n" \
  "v_mov_b32 v185, 0\n" \
  "v_mov_b32 v186, 0\n" \
  "v_mov_b32 v187, 0\n" \
  "v_mov_b32 v188, 0\n" \
  "v_mov_b32 v189, 0\n" \
  "v_mov_b32 v190, 0\n" \
  "v_mov_b32 v191, 0\n" \
  "v_mov_b32 v192, 0\n" \
  "v_mov_b32 v193, 0\n" \
  "v_mov_b32 v194, 0\n" \
  "v_mov_b32 v195, 0\n" \
  "v_mov_b32 v196, 0\n" \
  "v_mov_b32 v197, 0\n" \
  "v_mov_b32 v198, 0\n" \
  "v_mov_b32 v199, 0\n" \
  "v_mov_b32 v200, 0\n" \
  "v_mov_b32 v201, 0\n" \
  "v_mov_b32 v202, 0\n" \
  "v_mov_b32 v203, 0\n" \
  "v_mov_b32 v204, 0\n" \
  "v_mov_b32 v205, 0\n" \
  "v_mov_b32 v206, 0\n" \
  "v_mov_b32 v207, 0\n" \
  "v_mov_b32 v208, 0\n" \
  "v_mov_b32 v209, 0\n" \
  "v_mov_b32 v210, 0\n" \
  "v_mov_b32 v211, 0\n" \
  "v_mov_b32 v212, 0\n" \
  "v_mov_b32 v213, 0\n" \
  "v_mov_b32 v214, 0\n" \
  "v_mov_b32 v215, 0\n" \
  "v_mov_b32 v216, 0\n" \
  "v_mov_b32 v217, 0\n" \
  "v_mov_b32 v218, 0\n" \
  "v_mov_b32 v219, 0\n" \
  "v_mov_b32 v220, 0\n" \
  "v_mov_b32 v221, 0\n" \
  "v_mov_b32 v222, 0\n" \
  "v_mov_b32 v223, 0\n" \
  "v_mov_b32 v224, 0\n" \
  "v_mov_b32 v225, 0\n" \
  "v_mov_b32 v226, 0\n" \
  "v_mov_b32 v227, 0\n" \
  "v_mov_b32 v228, 0\n" \
  "v_mov_b32 v229, 0\n" \
  "v_mov_b32 v230, 0\n" \
  "v_mov_b32 v231, 0\n" \
  "v_mov_b32 v232, 0\n" \
  "v_mov_b32 v233, 0\n" \
  "v_mov_b32 v234, 0\n" \
  "v_mov_b32 v235, 0\n" \
  "v_mov_b32 v236, 0\n" \
  "v_mov_b32 v237, 0\n" \
  "v_mov_b32 v238, 0\n" \
  "v_mov_b32 v239, 0\n" \
  "v_mov_b32 v240, 0\n" \
  "v_mov_b32 v241, 0\n" \
  "v_mov_b32 v242, 0\n" \
  "v_mov_b32 v243, 0\n" \
  "s_cmp_ge_u32 s64, %[tag]\n" \
  "s_cbranch_scc0 .Lslow0_%=\n" \
  ".Lback0_%=:\n" \
  "s_add_u32 %[tag], %[tag], 1\n" \
  "s_add_u32 %[sw], %[sw], 262144\n" \
  "buffer_load_dwordx4 a[0:3], %[vx0], %[xr], %[sx] offen sc1\n" \
  "buffer_load_dwordx4 a[4:7], %[vx1], %[xr], %[sx] offen sc1\n" \
  "buffer_load_dwordx4 a[8:11], %[vx2], %[xr], %[sx] offen sc1\n" \
  "buffer_load_dwordx4 a[12:15], %[vx3], %[xr], %[sx] offen sc1\n" \
  "s_load_dword s64, %[fp], %[sl] offset:0x4 glc\n" \
  "s_add_u32 %[sx], %[sx], 4096\n" \
  "s_cmp_ge_u32 s65, %[tag]\n" \
  "s_cbranch_scc0 .Lslow1_%=\n" \
  ".Lback1_%=:\n" \
  "s_add_u32 %[tag], %[tag], 1\n" \
  "buffer_load_dwordx4 a[16:19], %[vx0], %[xr], %[sx] offen sc1\n" \
  "buffer_load_dwordx4 a[20:23], %[vx1], %[xr], %[sx] offen sc1\n" \
  "buffer_load_dwordx4 a[24:27], %[vx2], %[xr], %[sx] offen sc1\n" \
  "s_waitcnt vmcnt(12)\n" \
  "s_barrier\n" \
  "ds_read_b64_tr_b16 v[64:65], %[va0] offset:0\n" \
  "s_load_dword s65, %[fp], %[sl] offset:0x804 glc\n" \
  "s_add_u32 m0, %[ma], 32768\n" \
  "s_nop 0\n" \
  "ds_read_b64_tr_b16 v[66:67], %[va0] offset:512\n" \
  "buffer_load_dwordx4 a[28:31], %[vx3], %[xr], %[sx] offen sc1\n" \
  "ds_read_b64_tr_b16 v[68:69], %[va1] offset:0\n" \
  "s_add_u32 %[sx], %[sx], 4096\n" \
  "ds_read_b64_tr_b16 v[70:71], %[va1] offset:512\n" \
  "ds_read_b64_tr_b16 v[72:73], %[va2] offset:0\n" \
  "buffer_load_dwordx4 %[vd0], %[wr], %[sw] offen offset:0 lds\n" \
  "ds_read_b64_tr_b16 v[74:75], %[va2] offset:512\n" \
  "ds_read_b64_tr_b16 v[76:77], %[va3] offset:0\n" \
  "ds_read_b64_tr_b16 v[78:79], %[va3] offset:512\n" \
  "ds_read_b64_tr_b16 v[80:81], %[va0] offset:8192\n" \
  "buffer_load_dwordx4 %[vd1], %[wr], %[sw] offen offset:1024 lds\n" \
  "ds_read_b64_tr_b16 v[82:83], %[va0] offset:8704\n" \
  "s_cmp_lt_u32 %[ct], 8\n" \
  "s_cbranch_scc0 .LloopB_%=\n" \
  ".Lloop_%=:\n" \
  "s_waitcnt vmcnt(6) lgkmcnt(8)\n" \
  "v_mfma_f32_16x16x32_bf16 v[96:99], v[64:67], a[0:3], v[96:99]\n" \
  "ds_read_b64_tr_b16 v[84:85], %[va1] offset:8192\n" \
  "v_mfma_f32_16x16x32_bf16 v[100:103], v[64:67], a[4:7], v[100:103]\n" \
  "buffer_load_dwordx4 %[vd2], %[wr], %[sw] offen offset:2048 lds\n" \
  "v_mfma_f32_16x16x32_bf16 v[104:107], v[64:67], a[8:11], v[104:107]\n" \
  "ds_read_b64_tr_b16 v[86:87], %[va1] offset:8704\n" \
  "v_mfma_f32_16x16x32_bf16 v[108:111], v[64:67], a[12:15], v[108:111]\n" \
  "s_waitcnt lgkmcnt(8)\n" \
  "v_mfma_f32_16x16x32_bf16 v[124:127], v[68:71], a[12:15], v[124:127]\n" \
  "ds_read_b64_tr_b16 v[88:89], %[va2] offset:8192\n" \
  "v_mfma_f32_16x16x32_bf16 v[120:123], v[68:71], a[8:11], v[120:123]\n" \
  "buffer_load_dwordx4 %[vd3], %[wr], %[sw] offen offset:3072 lds\n" \
  "v_mfma_f32_16x16x32_bf16 v[116:119], v[68:71], a[4:7], v[116:119]\n" \
  "ds_read_b64_tr_b16 v[90:91], %[va2] offset:8704\n" \
  "s_add_u32 m0, %[mb], 8192\n" \
  "v_mfma_f32_16x16x32_bf16 v[112:115], v[68:71], a[0:3], v[112:115]\n" \
  "s_waitcnt lgkmcnt(8)\n" \
  "v_mfma_f32_16x16x32_bf16 v[128:131], v[72:75], a[0:3], v[128:131]\n" \
  "ds_read_b64_tr_b16 v[92:93], %[va3] offset:8192\n" \
  "v_mfma_f32_16x16x32_bf16 v[132:135], v[72:75], a[4:7], v[132:135]\n" \
  "buffer_load_dwordx4 %[vd4], %[wr], %[sw] offen lds\n" \
  "v_mfma_f32_16x16x32_bf16 v[136:139], v[72:75], a[8:11], v[136:139]\n" \
  "ds_read_b64_tr_b16 v[94:95], %[va3] offset:8704\n" \
  "s_cmp_ge_u32 s64, %[tag]\n" \
  "v_mfma_f32_16x16x32_bf16 v[140:143], v[72:75], a[12:15], v[140:143]\n" \
  "s_waitcnt lgkmcnt(8)\n" \
  "ds_read_b64_tr_b16 v[64:65], %[vb0] offset:0\n" \
  "v_mfma_f32_16x16x32_bf16 v[156:159], v[76:79], a[12:15], v[156:159]\n" \
  "ds_read_b64_tr_b16 v[66:67], %[vb0] offset:256\n" \
  "s_cbranch_scc0 .Lslow2_%=\n" \
  ".Lback2_%=:\n" \
  "v_mfma_f32_16x16x32_bf16 v[152:155], v[76:79], a[8:11], v[152:155]\n" \
  "s_add_u32 %[tag], %[tag], 1\n" \
  "s_add_u32 %[sw], %[sw], 262144\n" \
  "v_mfma_f32_16x16x32_bf16 v[148:151], v[76:79], a[4:7], v[148:151]\n" \
  "ds_read_b64_tr_b16 v[68:69], %[vb1] offset:0\n" \
  "v_mfma_f32_16x16x32_bf16 v[144:147], v[76:79], a[0:3], v[144:147]\n" \
  "s_waitcnt lgkmcnt(9)\n" \
  "v_mfma_f32_16x16x32_bf16 v[160:163], v[80:83], a[0:3], v[160:163]\n" \
  "ds_read_b64_tr_b16 v[70:71], %[vb1] offset:256\n" \
  "v_mfma_f32_16x16x32_bf16 v[164:167], v[80:83], a[4:7], v[164:167]\n" \
  "buffer_load_dwordx4 a[32:35], %[vx0], %[xr], %[sx] offen sc1\n" \
  "v_mfma_f32_16x16x32_bf16 v[168:171], v[80:83], a[8:11], v[168:171]\n" \
  "ds_read_b64_tr_b16 v[72:73], %[va0] offset:4096\n" \
  "v_mfma_f32_16x16x32_bf16 v[172:175], v[80:83], a[12:15], v[172:175]\n" \
  "s_waitcnt lgkmcnt(9)\n" \
  "v_mfma_f32_16x16x32_bf16 v[188:191], v[84:87], a[12:15], v[188:191]\n" \
  "ds_read_b64_tr_b16 v[74:75], %[va0] offset:4608\n" \
  "v_mfma_f32_16x16x32_bf16 v[184:187], v[84:87], a[8:11], v[184:187]\n" \
  "buffer_load_dwordx4 a[36:39], %[vx1], %[xr], %[sx] offen sc1\n" \
  "v_mfma_f32_16x16x32_bf16 v[180:183], v[84:87], a[4:7], v[180:183]\n" \
  "ds_read_b64_tr_b16 v[76:77], %[va1] offset:4096\n" \
  "v_mfma_f32_16x16x32_bf16 v[176:179], v[84:87], a[0:3], v[176:179]\n" \
  "s_waitcnt lgkmcnt(9)\n" \
  "ds_read_b64_tr_b16 v[78:79], %[va1] offset:4608\n" \
  "v_mfma_f32_16x16x32_bf16 v[192:195], v[88:91], a[0:3], v[192:195]\n" \
  "ds_read_b64_tr_b16 v[80:81], %[va2] offset:4096\n" \
  "v_mfma_f32_16x16x32_bf16 v[196:199], v[88:91], a[4:7], v[196:199]\n" \
  "buffer_load_dwordx4 a[40:43], %[vx2], %[xr], %[sx] offen sc1\n" \
  "v_mfma_f32_16x16x32_bf16 v[200:203], v[88:91], a[8:11], v[200:203]\n" \
  "ds_read_b64_tr_b16 v[82:83], %[va2] offset:4608\n" \
  "v_mfma_f32_16x16x32_bf16 v[204:207], v[88:91], a[12:15], v[204:207]\n" \
  "s_waitcnt lgkmcnt(10)\n" \
  "v_mfma_f32_16x16x32_bf16 v[220:223], v[92:95], a[12:15], v[220:223]\n" \
  "ds_read_b64_tr_b16 v[84:85], %[va3] offset:4096\n" \
  "v_mfma_f32_16x16x32_bf16 v[216:219], v[92:95], a[8:11], v[216:219]\n" \
  "buffer_load_dwordx4 a[44:47], %[vx3], %[xr], %[sx] offen sc1\n" \
  "v_mfma_f32_16x16x32_bf16 v[212:215], v[92:95], a[4:7], v[212:215]\n" \
  "ds_read_b64_tr_b16 v[86:87], %[va3] offset:4608\n" \
  "s_load_dword s64, %[fp], %[sl] offset:0x8 glc\n" \
  "v_mfma_f32_16x16x32_bf16 v[208:211], v[92:95], a[0:3], v[208:211]\n" \
  "s_waitcnt lgkmcnt(8)\n" \
  "s_add_u32 %[sx], %[sx], 4096\n" \
  "v_mfma_f32_16x16x32_bf16 v[224:227], v[64:67], a[0:3], v[224:227]\n" \
  "ds_read_b64_tr_b16 v[88:89], %[va0] offset:12288\n" \
  "v_mfma_f32_16x16x32_bf16 v[228:231], v[64:67], a[4:7], v[228:231]\n" \
  "v_mfma_f32_16x16x32_bf16 v[232:235], v[64:67], a[8:11], v[232:235]\n" \
  "ds_read_b64_tr_b16 v[90:91], %[va0] offset:12800\n" \
  "v_mfma_f32_16x16x32_bf16 v[236:239], v[64:67], a[12:15], v[236:239]\n" \
  "v_mfma_f32_16x16x32_bf16 v[240:243], v[68:71], a[12:15], v[240:243]\n" \
  "s_waitcnt vmcnt(9) lgkmcnt(8)\n" \
  "s_cmp_ge_u32 s65, %[tag]\n" \
  "v_mfma_f32_16x16x32_bf16 v[96:99], v[72:75], a[16:19], v[96:99]\n" \
  "ds_read_b64_tr_b16 v[92:93], %[va1] offset:12288\n" \
  "s_cbranch_scc0 .Lslow3_%=\n" \
  ".Lback3_%=:\n" \
  "v_mfma_f32_16x16x32_bf16 v[100:103], v[72:75], a[20:23], v[100:103]\n" \
  "s_add_u32 %[tag], %[tag], 1\n" \
  "v_mfma_f32_16x16x32_bf16 v[104:107], v[72:75], a[24:27], v[104:107]\n" \
  "ds_read_b64_tr_b16 v[94:95], %[va1] offset:12800\n" \
  "v_mfma_f32_16x16x32_bf16 v[108:111], v[72:75], a[28:31], v[108:111]\n" \
  "s_waitcnt lgkmcnt(8)\n" \
  "v_mfma_f32_16x16x32_bf16 v[124:127], v[76:79], a[28:31], v[124:127]\n" \
  "ds_read_b64_tr_b16 v[64:65], %[va2] offset:12288\n" \
  "v_mfma_f32_16x16x32_bf16 v[120:123], v[76:79], a[24:27], v[120:123]\n" \
  "buffer_load_dwordx4 a[48:51], %[vx0], %[xr], %[sx] offen sc1\n" \
  "v_mfma_f32_16x16x32_bf16 v[116:119], v[76:79], a[20:23], v[116:119]\n" \
  "ds_read_b64_tr_b16 v[66:67], %[va2] offset:12800\n" \
  "v_mfma_f32_16x16x32_bf16 v[112:115], v[76:79], a[16:19], v[112:115]\n" \
  "s_waitcnt lgkmcnt(8)\n" \
  "v_mfma_f32_16x16x32_bf16 v[128:131], v[80:83], a[16:19], v[128:131]\n" \
  "ds_read_b64_tr_b16 v[68:69], %[va3] offset:12288\n" \
  "v_mfma_f32_16x16x32_bf16 v[132:135], v[80:83], a[20:23], v[132:135]\n" \
  "buffer_load_dwordx4 a[52:55], %[vx1], %[xr], %[sx] offen sc1\n" \
  "v_mfma_f32_16x16x32_bf16 v[136:139], v[80:83], a[24:27], v[136:139]\n" \
  "ds_read_b64_tr_b16 v[70:71], %[va3] offset:12800\n" \
  "v_mfma_f32_16x16x32_bf16 v[140:143], v[80:83], a[28:31], v[140:143]\n" \
  "s_waitcnt lgkmcnt(8)\n" \
  "ds_read_b64_tr_b16 v[72:73], %[vb0] offset:2048\n" \
  "v_mfma_f32_16x16x32_bf16 v[156:159], v[84:87], a[28:31], v[156:159]\n" \
  "ds_read_b64_tr_b16 v[74:75], %[vb0] offset:2304\n" \
  "v_mfma_f32_16x16x32_bf16 v[152:155], v[84:87], a[24:27], v[152:155]\n" \
  "buffer_load_dwordx4 a[56:59], %[vx2], %[xr], %[sx] offen sc1\n" \
  "v_mfma_f32_16x16x32_bf16 v[148:151], v[84:87], a[20:23], v[148:151]\n" \
  "ds_read_b64_tr_b16 v[76:77], %[vb1] offset:2048\n" \
  "v_mfma_f32_16x16x32_bf16 v[144:147], v[84:87], a[16:19], v[144:147]\n" \
  "s_waitcnt lgkmcnt(9)\n" \
  "v_mfma_f32_16x16x32_bf16 v[160:163], v[88:91], a[16:19], v[160:163]\n" \
  "ds_read_b64_tr_b16 v[78:79], %[vb1] offset:2304\n" \
  "v_mfma_f32_16x16x32_bf16 v[164:167], v[88:91], a[20:23], v[164:167]\n" \
  "s_barrier\n" \
  "v_mfma_f32_16x16x32_bf16 v[168:171], v[88:91], a[24:27], v[168:171]\n" \
  "ds_read_b64_tr_b16 v[80:81], %[va0] offset:16384\n" \
  "s_load_dword s65, %[fp], %[sl] offset:0x808 glc\n" \
  "v_mfma_f32_16x16x32_bf16 v[172:175], v[88:91], a[28:31], v[172:175]\n" \
  "s_waitcnt lgkmcnt(9)\n" \
  "s_add_u32 m0, %[ma], 49152\n" \
  "v_mfma_f32_16x16x32_bf16 v[188:191], v[92:95], a[28:31], v[188:191]\n" \
  "ds_read_b64_tr_b16 v[82:83], %[va0] offset:16896\n" \
  "v_mfma_f32_16x16x32_bf16 v[184:187], v[92:95], a[24:27], v[184:187]\n" \
  "buffer_load_dwordx4 a[60:63], %[vx3], %[xr], %[sx] offen sc1\n" \
  "v_mfma_f32_16x16x32_bf16 v[180:183], v[92:95], a[20:23], v[180:183]\n" \
  "ds_read_b64_tr_b16 v[84:85], %[va1] offset:16384\n" \
  "s_add_u32 %[sx], %[sx], 4096\n" \
  "v_mfma_f32_16x16x32_bf16 v[176:179], v[92:95], a[16:19], v[176:179]\n" \
  "s_waitcnt lgkmcnt(9)\n" \
  "ds_read_b64_tr_b16 v[86:87], %[va1] offset:16896\n" \
  "v_mfma_f32_16x16x32_bf16 v[192:195], v[64:67], a[16:19], v[192:195]\n" \
  "ds_read_b64_tr_b16 v[88:89], %[va2] offset:16384\n" \
  "v_mfma_f32_16x16x32_bf16 v[196:199], v[64:67], a[20:23], v[196:199]\n" \
  "buffer_load_dwordx4 %[vd0], %[wr], %[sw] offen offset:0 lds\n" \
  "v_mfma_f32_16x16x32_bf16 v[200:203], v[64:67], a[24:27], v[200:203]\n" \
  "ds_read_b64_tr_b16 v[90:91], %[va2] offset:16896\n" \
  "v_mfma_f32_16x16x32_bf16 v[204:207], v[64:67], a[28:31], v[204:207]\n" \
  "s_waitcnt lgkmcnt(10)\n" \
  "v_mfma_f32_16x16x32_bf16 v[220:223], v[68:71], a[28:31], v[220:223]\n" \
  "ds_read_b64_tr_b16 v[92:93], %[va3] offset:16384\n" \
  "v_mfma_f32_16x16x32_bf16 v[216:219], v[68:71], a[24:27], v[216:219]\n" \
  "v_mfma_f32_16x16x32_bf16 v[212:215], v[68:71], a[20:23], v[212:215]\n" \
  "ds_read_b64_tr_b16 v[94:95], %[va3] offset:16896\n" \
  "v_mfma_f32_16x16x32_bf16 v[208:211], v[68:71], a[16:19], v[208:211]\n" \
  "s_waitcnt lgkmcnt(8)\n" \
  "v_mfma_f32_16x16x32_bf16 v[224:227], v[72:75], a[16:19], v[224:227]\n" \
  "ds_read_b64_tr_b16 v[64:65], %[va0] offset:24576\n" \
  "v_mfma_f32_16x16x32_bf16 v[228:231], v[72:75], a[20:23], v[228:231]\n" \
  "buffer_load_dwordx4 %[vd1], %[wr], %[sw] offen offset:1024 lds\n" \
  "v_mfma_f32_16x16x32_bf16 v[232:235], v[72:75], a[24:27], v[232:235]\n" \
  "ds_read_b64_tr_b16 v[66:67], %[va0] offset:25088\n" \
  "v_mfma_f32_16x16x32_bf16 v[236:239], v[72:75], a[28:31], v[236:239]\n" \
  "v_mfma_f32_16x16x32_bf16 v[240:243], v[76:79], a[28:31], v[240:243]\n" \
  "s_waitcnt vmcnt(6) lgkmcnt(8)\n" \
  "v_mfma_f32_16x16x32_bf16 v[96:99], v[80:83], a[32:35], v[96:99]\n" \
  "ds_read_b64_tr_b16 v[68:69], %[va1] offset:24576\n" \
  "v_mfma_f32_16x16x32_bf16 v[100:103], v[80:83], a[36:39], v[100:103]\n" \
  "buffer_load_dwordx4 %[vd2], %[wr], %[sw] offen offset:2048 lds\n" \
  "v_mfma_f32_16x16x32_bf16 v[104:107], v[80:83], a[40:43], v[104:107]\n" \
  "ds_read_b64_tr_b16 v[70:71], %[va1] offset:25088\n" \
  "v_mfma_f32_16x16x32_bf16 v[108:111], v[80:83], a[44:47], v[108:111]\n" \
  "s_waitcnt lgkmcnt(8)\n" \
  "v_mfma_f32_16x16x32_bf16 v[124:127], v[84:87], a[44:47], v[124:127]\n" \
  "ds_read_b64_tr_b16 v[72:73], %[va2] offset:24576\n" \
  "v_mfma_f32_16x16x32_bf16 v[120:123], v[84:87], a[40:43], v[120:123]\n" \
  "buffer_load_dwordx4 %[vd3], %[wr], %[sw] offen offset:3072 lds\n" \
  "v_mfma_f32_16x16x32_bf16 v[116:119], v[84:87], a[36:39], v[116:119]\n" \
  "ds_read_b64_tr_b16 v[74:75], %[va2] offset:25088\n" \
  "s_add_u32 m0, %[mb], 12288\n" \
  "v_mfma_f32_16x16x32_bf16 v[112:115], v[84:87], a[32:35], v[112:115]\n" \
  "s_waitcnt lgkmcnt(8)\n" \
  "v_mfma_f32_16x16x32_bf16 v[128:131], v[88:91], a[32:35], v[128:131]\n" \
  "ds_read_b64_tr_b16 v[76:77], %[va3] offset:24576\n" \
  "v_mfma_f32_16x16x32_bf16 v[132:135], v[88:91], a[36:39], v[132:135]\n" \
  "buffer_load_dwordx4 %[vd4], %[wr], %[sw] offen lds\n" \
  "v_mfma_f32_16x16x32_bf16 v[136:139], v[88:91], a[40:43], v[136:139]\n" \
  "ds_read_b64_tr_b16 v[78:79], %[va3] offset:25088\n" \
  "s_cmp_ge_u32 s64, %[tag]\n" \
  "v_mfma_f32_16x16x32_bf16 v[140:143], v[88:91], a[44:47], v[140:143]\n" \
  "s_waitcnt lgkmcnt(8)\n" \
  "ds_read_b64_tr_b16 v[80:81], %[vb0] offset:4096\n" \
  "v_mfma_f32_16x16x32_bf16 v[156:159], v[92:95], a[44:47], v[156:159]\n" \
  "ds_read_b64_tr_b16 v[82:83], %[vb0] offset:4352\n" \
  "s_cbranch_scc0 .Lslow4_%=\n" \
  ".Lback4_%=:\n" \
  "v_mfma_f32_16x16x32_bf16 v[152:155], v[92:95], a[40:43], v[152:155]\n" \
  "s_add_u32 %[tag], %[tag], 1\n" \
  "s_add_u32 %[sw], %[sw], 262144\n" \
  "v_mfma_f32_16x16x32_bf16 v[148:151], v[92:95], a[36:39], v[148:151]\n" \
  "ds_read_b64_tr_b16 v[84:85], %[vb1] offset:4096\n" \
  "v_mfma_f32_16x16x32_bf16 v[144:147], v[92:95], a[32:35], v[144:147]\n" \
  "s_waitcnt lgkmcnt(9)\n" \
  "v_mfma_f32_16x16x32_bf16 v[160:163], v[64:67], a[32:35], v[160:163]\n" \
  "ds_read_b64_tr_b16 v[86:87], %[vb1] offset:4352\n" \
  "v_mfma_f32_16x16x32_bf16 v[164:167], v[64:67], a[36:39], v[164:167]\n" \
  "buffer_load_dwordx4 a[64:67], %[vx0], %[xr], %[sx] offen sc1\n" \
  "v_mfma_f32_16x16x32_bf16 v[168:171], v[64:67], a[40:43], v[168:171]\n" \
  "ds_read_b64_tr_b16 v[88:89], %[va0] offset:20480\n" \
  "v_mfma_f32_16x16x32_bf16 v[172:175], v[64:67], a[44:47], v[172:175]\n" \
  "s_waitcnt lgkmcnt(9)\n" \
  "v_mfma_f32_16x16x32_bf16 v[188:191], v[68:71], a[44:47], v[188:191]\n" \
  "ds_read_b64_tr_b16 v[90:91], %[va0] offset:20992\n" \
  "v_mfma_f32_16x16x32_bf16 v[184:187], v[68:71], a[40:43], v[184:187]\n" \
  "buffer_load_dwordx4 a[68:71], %[vx1], %[xr], %[sx] offen sc1\n" \
  "v_mfma_f32_16x16x32_bf16 v[180:183], v[68:71], a[36:39], v[180:183]\n" \
  "ds_read_b64_tr_b16 v[92:93], %[va1] offset:20480\n" \
  "v_mfma_f32_16x16x32_bf16 v[176:179], v[68:71], a[32:35], v[176:179]\n" \
  "s_waitcnt lgkmcnt(9)\n" \
  "ds_read_b64_tr_b16 v[94:95], %[va1] offset:20992\n" \
  "v_mfma_f32_16x16x32_bf16 v[192:195], v[72:75], a[32:35], v[192:195]\n" \
  "ds_read_b64_tr_b16 v[64:65], %[va2] offset:20480\n" \
  "v_mfma_f32_16x16x32_bf16 v[196:199], v[72:75], a[36:39], v[196:199]\n" \
  "buffer_load_dwordx4 a[72:75], %[vx2], %[xr], %[sx] offen sc1\n" \
  "v_mfma_f32_16x16x32_bf16 v[200:203], v[72:75], a[40:43], v[200:203]\n" \
  "ds_read_b64_tr_b16 v[66:67], %[va2] offset:20992\n" \
  "v_mfma_f32_16x16x32_bf16 v[204:207], v[72:75], a[44:47], v[204:207]\n" \
  "s_waitcnt lgkmcnt(10)\n" \
  "v_mfma_f32_16x16x32_bf16 v[220:223], v[76:79], a[44:47], v[220:223]\n" \
  "ds_read_b64_tr_b16 v[68:69], %[va3] offset:20480\n" \
  "v_mfma_f32_16x16x32_bf16 v[216:219], v[76:79], a[40:43], v[216:219]\n" \
  "buffer_load_dwordx4 a[76:79], %[vx3], %[xr], %[sx] offen sc1\n" \
  "v_mfma_f32_16x16x32_bf16 v[212:215], v[76:79], a[36:39], v[212:215]\n" \
  "ds_read_b64_tr_b16 v[70:71], %[va3] offset:20992\n" \
  "s_load_dword s64, %[fp], %[sl] offset:0xc glc\n" \
  "v_mfma_f32_16x16x32_bf16 v[208:211], v[76:79], a[32:35], v[208:211]\n" \
  "s_waitcnt lgkmcnt(8)\n" \
  "s_add_u32 %[sx], %[sx], 4096\n" \
  "v_mfma_f32_16x16x32_bf16 v[224:227], v[80:83], a[32:35], v[224:227]\n" \
  "ds_read_b64_tr_b16 v[72:73], %[va0] offset:28672\n" \
  "v_mfma_f32_16x16x32_bf16 v[228:231], v[80:83], a[36:39], v[228:231]\n" \
  "v_mfma_f32_16x16x32_bf16 v[232:235], v[80:83], a[40:43], v[232:235]\n" \
  "ds_read_b64_tr_b16 v[74:75], %[va0] offset:29184\n" \
  "v_mfma_f32_16x16x32_bf16 v[236:239], v[80:83], a[44:47], v[236:239]\n" \
  "v_mfma_f32_16x16x32_bf16 v[240:243], v[84:87], a[44:47], v[240:243]\n" \
  "s_waitcnt vmcnt(9) lgkmcnt(8)\n" \
  "s_cmp_ge_u32 s65, %[tag]\n" \
  "v_mfma_f32_16x16x32_bf16 v[96:99], v[88:91], a[48:51], v[96:99]\n" \
  "ds_read_b64_tr_b16 v[76:77], %[va1] offset:28672\n" \
  "s_cbranch_scc0 .Lslow5_%=\n" \
  ".Lback5_%=:\n" \
  "v_mfma_f32_16x16x32_bf16 v[100:103], v[88:91], a[52:55], v[100:103]\n" \
  "s_add_u32 %[tag], %[tag], 1\n" \
  "v_mfma_f32_16x16x32_bf16 v[104:107], v[88:91], a[56:59], v[104:107]\n" \
  "ds_read_b64_tr_b16 v[78:79], %[va1] offset:29184\n" \
  "v_mfma_f32_16x16x32_bf16 v[108:111], v[88:91], a[60:63], v[108:111]\n" \
  "s_waitcnt lgkmcnt(8)\n" \
  "v_mfma_f32_16x16x32_bf16 v[124:127], v[92:95], a[60:63], v[124:127]\n" \
  "ds_read_b64_tr_b16 v[80:81], %[va2] offset:28672\n" \
  "v_mfma_f32_16x16x32_bf16 v[120:123], v[92:95], a[56:59], v[120:123]\n" \
  "buffer_load_dwordx4 a[80:83], %[vx0], %[xr], %[sx] offen sc1\n" \
  "v_mfma_f32_16x16x32_bf16 v[116:119], v[92:95], a[52:55], v[116:119]\n" \
  "ds_read_b64_tr_b16 v[82:83], %[va2] offset:29184\n" \
  "v_mfma_f32_16x16x32_bf16 v[112:115], v[92:95], a[48:51], v[112:115]\n" \
  "s_waitcnt lgkmcnt(8)\n" \
  "v_mfma_f32_16x16x32_bf16 v[128:131], v[64:67], a[48:51], v[128:131]\n" \
  "ds_read_b64_tr_b16 v[84:85], %[va3] offset:28672\n" \
  "v_mfma_f32_16x16x32_bf16 v[132:135], v[64:67], a[52:55], v[132:135]\n" \
  "buffer_load_dwordx4 a[84:87], %[vx1], %[xr], %[sx] offen sc1\n" \
  "v_mfma_f32_16x16x32_bf16 v[136:139], v[64:67], a[56:59], v[136:139]\n" \
  "ds_read_b64_tr_b16 v[86:87], %[va3] offset:29184\n" \
  "v_mfma_f32_16x16x32_bf16 v[140:143], v[64:67], a[60:63], v[140:143]\n" \
  "s_waitcnt lgkmcnt(8)\n" \
  "ds_read_b64_tr_b16 v[88:89], %[vb0] offset:6144\n" \
  "v_mfma_f32_16x16x32_bf16 v[156:159], v[68:71], a[60:63], v[156:159]\n" \
  "ds_read_b64_tr_b16 v[90:91], %[vb0] offset:6400\n" \
  "v_mfma_f32_16x16x32_bf16 v[152:155], v[68:71], a[56:59], v[152:155]\n" \
  "buffer_load_dwordx4 a[88:91], %[vx2], %[xr], %[sx] offen sc1\n" \
  "v_mfma_f32_16x16x32_bf16 v[148:151], v[68:71], a[52:55], v[148:151]\n" \
  "ds_read_b64_tr_b16 v[92:93], %[vb1] offset:6144\n" \
  "v_mfma_f32_16x16x32_bf16 v[144:147], v[68:71], a[48:51], v[144:147]\n" \
  "s_waitcnt lgkmcnt(9)\n" \
  "v_mfma_f32_16x16x32_bf16 v[160:163], v[72:75], a[48:51], v[160:163]\n" \
  "ds_read_b64_tr_b16 v[94:95], %[vb1] offset:6400\n" \
  "v_mfma_f32_16x16x32_bf16 v[164:167], v[72:75], a[52:55], v[164:167]\n" \
  "s_barrier\n" \
  "v_mfma_f32_16x16x32_bf16 v[168:171], v[72:75], a[56:59], v[168:171]\n" \
  "ds_read_b64_tr_b16 v[64:65], %[va0] offset:32768\n" \
  "s_load_dword s65, %[fp], %[sl] offset:0x80c glc\n" \
  "v_mfma_f32_16x16x32_bf16 v[172:175], v[72:75], a[60:63], v[172:175]\n" \
  "s_waitcnt lgkmcnt(9)\n" \
  "s_add_u32 m0, %[ma], 0\n" \
  "v_mfma_f32_16x16x32_bf16 v[188:191], v[76:79], a[60:63], v[188:191]\n" \
  "ds_read_b64_tr_b16 v[66:67], %[va0] offset:33280\n" \
  "v_mfma_f32_16x16x32_bf16 v[184:187], v[76:79], a[56:59], v[184:187]\n" \
  "buffer_load_dwordx4 a[92:95], %[vx3], %[xr], %[sx] offen sc1\n" \
  "v_mfma_f32_16x16x32_bf16 v[180:183], v[76:79], a[52:55], v[180:183]\n" \
  "ds_read_b64_tr_b16 v[68:69], %[va1] offset:32768\n" \
  "s_add_u32 %[sx], %[sx], 4096\n" \
  "v_mfma_f32_16x16x32_bf16 v[176:179], v[76:79], a[48:51], v[176:179]\n" \
  "s_waitcnt lgkmcnt(9)\n" \
  "ds_read_b64_tr_b16 v[70:71], %[va1] offset:33280\n" \
  "v_mfma_f32_16x16x32_bf16 v[192:195], v[80:83], a[48:51], v[192:195]\n" \
  "ds_read_b64_tr_b16 v[72:73], %[va2] offset:32768\n" \
  "v_mfma_f32_16x16x32_bf16 v[196:199], v[80:83], a[52:55], v[196:199]\n" \
  "buffer_load_dwordx4 %[vd0], %[wr], %[sw] offen offset:0 lds\n" \
  "v_mfma_f32_16x16x32_bf16 v[200:203], v[80:83], a[56:59], v[200:203]\n" \
  "ds_read_b64_tr_b16 v[74:75], %[va2] offset:33280\n" \
  "v_mfma_f32_16x16x32_bf16 v[204:207], v[80:83], a[60:63], v[204:207]\n" \
  "s_waitcnt lgkmcnt(10)\n" \
  "v_mfma_f32_16x16x32_bf16 v[220:223], v[84:87], a[60:63], v[220:223]\n" \
  "ds_read_b64_tr_b16 v[76:77], %[va3] offset:32768\n" \
  "v_mfma_f32_16x16x32_bf16 v[216:219], v[84:87], a[56:59], v[216:219]\n" \
  "v_mfma_f32_16x16x32_bf16 v[212:215], v[84:87], a[52:55], v[212:215]\n" \
  "ds_read_b64_tr_b16 v[78:79], %[va3] offset:33280\n" \
  "v_mfma_f32_16x16x32_bf16 v[208:211], v[84:87], a[48:51], v[208:211]\n" \
  "s_waitcnt lgkmcnt(8)\n" \
  "v_mfma_f32_16x16x32_bf16 v[224:227], v[88:91], a[48:51], v[224:227]\n" \
  "ds_read_b64_tr_b16 v[80:81], %[va0] offset:40960\n" \
  "v_mfma_f32_16x16x32_bf16 v[228:231], v[88:91], a[52:55], v[228:231]\n" \
  "buffer_load_dwordx4 %[vd1], %[wr], %[sw] offen offset:1024 lds\n" \
  "v_mfma_f32_16x16x32_bf16 v[232:235], v[88:91], a[56:59], v[232:235]\n" \
  "ds_read_b64_tr_b16 v[82:83], %[va0] offset:41472\n" \
  "v_mfma_f32_16x16x32_bf16 v[236:239], v[88:91], a[60:63], v[236:239]\n" \
  "v_mfma_f32_16x16x32_bf16 v[240:243], v[92:95], a[60:63], v[240:243]\n" \
  "s_waitcnt vmcnt(6) lgkmcnt(8)\n" \
  "v_mfma_f32_16x16x32_bf16 v[96:99], v[64:67], a[64:67], v[96:99]\n" \
  "ds_read_b64_tr_b16 v[84:85], %[va1] offset:40960\n" \
  "v_mfma_f32_16x16x32_bf16 v[100:103], v[64:67], a[68:71], v[100:103]\n" \
  "buffer_load_dwordx4 %[vd2], %[wr], %[sw] offen offset:2048 lds\n" \
  "v_mfma_f32_16x16x32_bf16 v[104:107], v[64:67], a[72:75], v[104:107]\n" \
  "ds_read_b64_tr_b16 v[86:87], %[va1] offset:41472\n" \
  "v_mfma_f32_16x16x32_bf16 v[108:111], v[64:67], a[76:79], v[108:111]\n" \
  "s_waitcnt lgkmcnt(8)\n" \
  "v_mfma_f32_16x16x32_bf16 v[124:127], v[68:71], a[76:79], v[124:127]\n" \
  "ds_read_b64_tr_b16 v[88:89], %[va2] offset:40960\n" \
  "v_mfma_f32_16x16x32_bf16 v[120:123], v[68:71], a[72:75], v[120:123]\n" \
  "buffer_load_dwordx4 %[vd3], %[wr], %[sw] offen offset:3072 lds\n" \
  "v_mfma_f32_16x16x32_bf16 v[116:119], v[68:71], a[68:71], v[116:119]\n" \
  "ds_read_b64_tr_b16 v[90:91], %[va2] offset:41472\n" \
  "s_add_u32 m0, %[mb], 0\n" \
  "v_mfma_f32_16x16x32_bf16 v[112:115], v[68:71], a[64:67], v[112:115]\n" \
  "s_waitcnt lgkmcnt(8)\n" \
  "v_mfma_f32_16x16x32_bf16 v[128:131], v[72:75], a[64:67], v[128:131]\n" \
  "ds_read_b64_tr_b16 v[92:93], %[va3] offset:40960\n" \
  "v_mfma_f32_16x16x32_bf16 v[132:135], v[72:75], a[68:71], v[132:135]\n" \
  "buffer_load_dwordx4 %[vd4], %[wr], %[sw] offen lds\n" \
  "v_mfma_f32_16x16x32_bf16 v[136:139], v[72:75], a[72:75], v[136:139]\n" \
  "ds_read_b64_tr_b16 v[94:95], %[va3] offset:41472\n" \
  "s_cmp_ge_u32 s64, %[tag]\n" \
  "v_mfma_f32_16x16x32_bf16 v[140:143], v[72:75], a[76:79], v[140:143]\n" \
  "s_waitcnt lgkmcnt(8)\n" \
  "ds_read_b64_tr_b16 v[64:65], %[vb0] offset:8192\n" \
  "v_mfma_f32_16x16x32_bf16 v[156:159], v[76:79], a[76:79], v[156:159]\n" \
  "ds_read_b64_tr_b16 v[66:67], %[vb0] offset:8448\n" \
  "s_cbranch_scc0 .Lslow6_%=\n" \
  ".Lback6_%=:\n" \
  "v_mfma_f32_16x16x32_bf16 v[152:155], v[76:79], a[72:75], v[152:155]\n" \
  "s_add_u32 %[tag], %[tag], 1\n" \
  "s_add_u32 %[sw], %[sw], 262144\n" \
  "v_mfma_f32_16x16x32_bf16 v[148:151], v[76:79], a[68:71], v[148:151]\n" \
  "ds_read_b64_tr_b16 v[68:69], %[vb1] offset:8192\n" \
  "v_mfma_f32_16x16x32_bf16 v[144:147], v[76:79], a[64:67], v[144:147]\n" \
  "s_waitcnt lgkmcnt(9)\n" \
  "v_mfma_f32_16x16x32_bf16 v[160:163], v[80:83], a[64:67], v[160:163]\n" \
  "ds_read_b64_tr_b16 v[70:71], %[vb1] offset:8448\n" \
  "v_mfma_f32_16x16x32_bf16 v[164:167], v[80:83], a[68:71], v[164:167]\n" \
  "buffer_load_dwordx4 a[96:99], %[vx0], %[xr], %[sx] offen sc1\n" \
  "v_mfma_f32_16x16x32_bf16 v[168:171], v[80:83], a[72:75], v[168:171]\n" \
  "ds_read_b64_tr_b16 v[72:73], %[va0] offset:36864\n" \
  "v_mfma_f32_16x16x32_bf16 v[172:175], v[80:83], a[76:79], v[172:175]\n" \
  "s_waitcnt lgkmcnt(9)\n" \
  "v_mfma_f32_16x16x32_bf16 v[188:191], v[84:87], a[76:79], v[188:191]\n" \
  "ds_read_b64_tr_b16 v[74:75], %[va0] offset:37376\n" \
  "v_mfma_f32_16x16x32_bf16 v[184:187], v[84:87], a[72:75], v[184:187]\n" \
  "buffer_load_dwordx4 a[100:103], %[vx1], %[xr], %[sx] offen sc1\n" \
  "v_mfma_f32_16x16x32_bf16 v[180:183], v[84:87], a[68:71], v[180:183]\n" \
  "ds_read_b64_tr_b16 v[76:77], %[va1] offset:36864\n" \
  "v_mfma_f32_16x16x32_bf16 v[176:179], v[84:87], a[64:67], v[176:179]\n" \
  "s_waitcnt lgkmcnt(9)\n" \
  "ds_read_b64_tr_b16 v[78:79], %[va1] offset:37376\n" \
  "v_mfma_f32_16x16x32_bf16 v[192:195], v[88:91], a[64:67], v[192:195]\n" \
  "ds_read_b64_tr_b16 v[80:81], %[va2] offset:36864\n" \
  "v_mfma_f32_16x16x32_bf16 v[196:199], v[88:91], a[68:71], v[196:199]\n" \
  "buffer_load_dwordx4 a[104:107], %[vx2], %[xr], %[sx] offen sc1\n" \
  "v_mfma_f32_16x16x32_bf16 v[200:203], v[88:91], a[72:75], v[200:203]\n" \
  "ds_read_b64_tr_b16 v[82:83], %[va2] offset:37376\n" \
  "v_mfma_f32_16x16x32_bf16 v[204:207], v[88:91], a[76:79], v[204:207]\n" \
  "s_waitcnt lgkmcnt(10)\n" \
  "v_mfma_f32_16x16x32_bf16 v[220:223], v[92:95], a[76:79], v[220:223]\n" \
  "ds_read_b64_tr_b16 v[84:85], %[va3] offset:36864\n" \
  "v_mfma_f32_16x16x32_bf16 v[216:219], v[92:95], a[72:75], v[216:219]\n" \
  "buffer_load_dwordx4 a[108:111], %[vx3], %[xr], %[sx] offen sc1\n" \
  "v_mfma_f32_16x16x32_bf16 v[212:215], v[92:95], a[68:71], v[212:215]\n" \
  "ds_read_b64_tr_b16 v[86:87], %[va3] offset:37376\n" \
  "s_load_dword s64, %[fp], %[sl] offset:0x10 glc\n" \
  "v_mfma_f32_16x16x32_bf16 v[208:211], v[92:95], a[64:67], v[208:211]\n" \
  "s_waitcnt lgkmcnt(8)\n" \
  "s_add_u32 %[sx], %[sx], 4096\n" \
  "v_mfma_f32_16x16x32_bf16 v[224:227], v[64:67], a[64:67], v[224:227]\n" \
  "ds_read_b64_tr_b16 v[88:89], %[va0] offset:45056\n" \
  "v_mfma_f32_16x16x32_bf16 v[228:231], v[64:67], a[68:71], v[228:231]\n" \
  "v_mfma_f32_16x16x32_bf16 v[232:235], v[64:67], a[72:75], v[232:235]\n" \
  "ds_read_b64_tr_b16 v[90:91], %[va0] offset:45568\n" \
  "v_mfma_f32_16x16x32_bf16 v[236:239], v[64:67], a[76:79], v[236:239]\n" \
  "v_mfma_f32_16x16x32_bf16 v[240:243], v[68:71], a[76:79], v[240:243]\n" \
  "s_waitcnt vmcnt(9) lgkmcnt(8)\n" \
  "s_cmp_ge_u32 s65, %[tag]\n" \
  "v_mfma_f32_16x16x32_bf16 v[96:99], v[72:75], a[80:83], v[96:99]\n" \
  "ds_read_b64_tr_b16 v[92:93], %[va1] offset:45056\n" \
  "s_cbranch_scc0 .Lslow7_%=\n" \
  ".Lback7_%=:\n" \
  "v_mfma_f32_16x16x32_bf16 v[100:103], v[72:75], a[84:87], v[100:103]\n" \
  "s_add_u32 %[tag], %[tag], 1\n" \
  "v_mfma_f32_16x16x32_bf16 v[104:107], v[72:75], a[88:91], v[104:107]\n" \
  "ds_read_b64_tr_b16 v[94:95], %[va1] offset:45568\n" \
  "v_mfma_f32_16x16x32_bf16 v[108:111], v[72:75], a[92:95], v[108:111]\n" \
  "s_waitcnt lgkmcnt(8)\n" \
  "v_mfma_f32_16x16x32_bf16 v[124:127], v[76:79], a[92:95], v[124:127]\n" \
  "ds_read_b64_tr_b16 v[64:65], %[va2] offset:45056\n" \
  "v_mfma_f32_16x16x32_bf16 v[120:123], v[76:79], a[88:91], v[120:123]\n" \
  "buffer_load_dwordx4 a[112:115], %[vx0], %[xr], %[sx] offen sc1\n" \
  "v_mfma_f32_16x16x32_bf16 v[116:119], v[76:79], a[84:87], v[116:119]\n" \
  "ds_read_b64_tr_b16 v[66:67], %[va2] offset:45568\n" \
  "v_mfma_f32_16x16x32_bf16 v[112:115], v[76:79], a[80:83], v[112:115]\n" \
  "s_waitcnt lgkmcnt(8)\n" \
  "v_mfma_f32_16x16x32_bf16 v[128:131], v[80:83], a[80:83], v[128:131]\n" \
  "ds_read_b64_tr_b16 v[68:69], %[va3] offset:45056\n" \
  "v_mfma_f32_16x16x32_bf16 v[132:135], v[80:83], a[84:87], v[132:135]\n" \
  "buffer_load_dwordx4 a[116:119], %[vx1], %[xr], %[sx] offen sc1\n" \
  "v_mfma_f32_16x16x32_bf16 v[136:139], v[80:83], a[88:91], v[136:139]\n" \
  "ds_read_b64_tr_b16 v[70:71], %[va3] offset:45568\n" \
  "v_mfma_f32_16x16x32_bf16 v[140:143], v[80:83], a[92:95], v[140:143]\n" \
  "s_waitcnt lgkmcnt(8)\n" \
  "ds_read_b64_tr_b16 v[72:73], %[vb0] offset:10240\n" \
  "v_mfma_f32_16x16x32_bf16 v[156:159], v[84:87], a[92:95], v[156:159]\n" \
  "ds_read_b64_tr_b16 v[74:75], %[vb0] offset:10496\n" \
  "v_mfma_f32_16x16x32_bf16 v[152:155], v[84:87], a[88:91], v[152:155]\n" \
  "buffer_load_dwordx4 a[120:123], %[vx2], %[xr], %[sx] offen sc1\n" \
  "v_mfma_f32_16x16x32_bf16 v[148:151], v[84:87], a[84:87], v[148:151]\n" \
  "ds_read_b64_tr_b16 v[76:77], %[vb1] offset:10240\n" \
  "v_mfma_f32_16x16x32_bf16 v[144:147], v[84:87], a[80:83], v[144:147]\n" \
  "s_waitcnt lgkmcnt(9)\n" \
  "v_mfma_f32_16x16x32_bf16 v[160:163], v[88:91], a[80:83], v[160:163]\n" \
  "ds_read_b64_tr_b16 v[78:79], %[vb1] offset:10496\n" \
  "v_mfma_f32_16x16x32_bf16 v[164:167], v[88:91], a[84:87], v[164:167]\n" \
  "s_barrier\n" \
  "v_mfma_f32_16x16x32_bf16 v[168:171], v[88:91], a[88:91], v[168:171]\n" \
  "ds_read_b64_tr_b16 v[80:81], %[va0] offset:49152\n" \
  "s_load_dword s65, %[fp], %[sl] offset:0x810 glc\n" \
  "v_mfma_f32_16x16x32_bf16 v[172:175], v[88:91], a[92:95], v[172:175]\n" \
  "s_waitcnt lgkmcnt(9)\n" \
  "s_add_u32 m0, %[ma], 16384\n" \
  "v_mfma_f32_16x16x32_bf16 v[188:191], v[92:95], a[92:95], v[188:191]\n" \
  "ds_read_b64_tr_b16 v[82:83], %[va0] offset:49664\n" \
  "v_mfma_f32_16x16x32_bf16 v[184:187], v[92:95], a[88:91], v[184:187]\n" \
  "buffer_load_dwordx4 a[124:127], %[vx3], %[xr], %[sx] offen sc1\n" \
  "v_mfma_f32_16x16x32_bf16 v[180:183], v[92:95], a[84:87], v[180:183]\n" \
  "ds_read_b64_tr_b16 v[84:85], %[va1] offset:49152\n" \
  "s_add_u32 %[sx], %[sx], 4096\n" \
  "v_mfma_f32_16x16x32_bf16 v[176:179], v[92:95], a[80:83], v[176:179]\n" \
  "s_waitcnt lgkmcnt(9)\n" \
  "ds_read_b64_tr_b16 v[86:87], %[va1] offset:49664\n" \
  "v_mfma_f32_16x16x32_bf16 v[192:195], v[64:67], a[80:83], v[192:195]\n" \
  "ds_read_b64_tr_b16 v[88:89], %[va2] offset:49152\n" \
  "v_mfma_f32_16x16x32_bf16 v[196:199], v[64:67], a[84:87], v[196:199]\n" \
  "buffer_load_dwordx4 %[vd0], %[wr], %[sw] offen offset:0 lds\n" \
  "v_mfma_f32_16x16x32_bf16 v[200:203], v[64:67], a[88:91], v[200:203]\n" \
  "ds_read_b64_tr_b16 v[90:91], %[va2] offset:49664\n" \
  "v_mfma_f32_16x16x32_bf16 v[204:207], v[64:67], a[92:95], v[204:207]\n" \
  "s_waitcnt lgkmcnt(10)\n" \
  "v_mfma_f32_16x16x32_bf16 v[220:223], v[68:71], a[92:95], v[220:223]\n" \
  "ds_read_b64_tr_b16 v[92:93], %[va3] offset:49152\n" \
  "v_mfma_f32_16x16x32_bf16 v[216:219], v[68:71], a[88:91], v[216:219]\n" \
  "v_mfma_f32_16x16x32_bf16 v[212:215], v[68:71], a[84:87], v[212:215]\n" \
  "ds_read_b64_tr_b16 v[94:95], %[va3] offset:49664\n" \
  "v_mfma_f32_16x16x32_bf16 v[208:211], v[68:71], a[80:83], v[208:211]\n" \
  "s_waitcnt lgkmcnt(8)\n" \
  "v_mfma_f32_16x16x32_bf16 v[224:227], v[72:75], a[80:83], v[224:227]\n" \
  "ds_read_b64_tr_b16 v[64:65], %[va0] offset:57344\n" \
  "v_mfma_f32_16x16x32_bf16 v[228:231], v[72:75], a[84:87], v[228:231]\n" \
  "buffer_load_dwordx4 %[vd1], %[wr], %[sw] offen offset:1024 lds\n" \
  "v_mfma_f32_16x16x32_bf16 v[232:235], v[72:75], a[88:91], v[232:235]\n" \
  "ds_read_b64_tr_b16 v[66:67], %[va0] offset:57856\n" \
  "v_mfma_f32_16x16x32_bf16 v[236:239], v[72:75], a[92:95], v[236:239]\n" \
  "v_mfma_f32_16x16x32_bf16 v[240:243], v[76:79], a[92:95], v[240:243]\n" \
  "s_waitcnt vmcnt(6) lgkmcnt(8)\n" \
  "v_mfma_f32_16x16x32_bf16 v[96:99], v[80:83], a[96:99], v[96:99]\n" \
  "ds_read_b64_tr_b16 v[68:69], %[va1] offset:57344\n" \
  "v_mfma_f32_16x16x32_bf16 v[100:103], v[80:83], a[100:103], v[100:103]\n" \
  "buffer_load_dwordx4 %[vd2], %[wr], %[sw] offen offset:2048 lds\n" \
  "v_mfma_f32_16x16x32_bf16 v[104:107], v[80:83], a[104:107], v[104:107]\n" \
  "ds_read_b64_tr_b16 v[70:71], %[va1] offset:57856\n" \
  "v_mfma_f32_16x16x32_bf16 v[108:111], v[80:83], a[108:111], v[108:111]\n" \
  "s_waitcnt lgkmcnt(8)\n" \
  "v_mfma_f32_16x16x32_bf16 v[124:127], v[84:87], a[108:111], v[124:127]\n" \
  "ds_read_b64_tr_b16 v[72:73], %[va2] offset:57344\n" \
  "v_mfma_f32_16x16x32_bf16 v[120:123], v[84:87], a[104:107], v[120:123]\n" \
  "buffer_load_dwordx4 %[vd3], %[wr], %[sw] offen offset:3072 lds\n" \
  "v_mfma_f32_16x16x32_bf16 v[116:119], v[84:87], a[100:103], v[116:119]\n" \
  "ds_read_b64_tr_b16 v[74:75], %[va2] offset:57856\n" \
  "s_add_u32 m0, %[mb], 4096\n" \
  "v_mfma_f32_16x16x32_bf16 v[112:115], v[84:87], a[96:99], v[112:115]\n" \
  "s_waitcnt lgkmcnt(8)\n" \
  "v_mfma_f32_16x16x32_bf16 v[128:131], v[88:91], a[96:99], v[128:131]\n" \
  "ds_read_b64_tr_b16 v[76:77], %[va3] offset:57344\n" \
  "v_mfma_f32_16x16x32_bf16 v[132:135], v[88:91], a[100:103], v[132:135]\n" \
  "buffer_load_dwordx4 %[vd4], %[wr], %[sw] offen lds\n" \
  "v_mfma_f32_16x16x32_bf16 v[136:139], v[88:91], a[104:107], v[136:139]\n" \
  "ds_read_b64_tr_b16 v[78:79], %[va3] offset:57856\n" \
  "s_cmp_ge_u32 s64, %[tag]\n" \
  "v_mfma_f32_16x16x32_bf16 v[140:143], v[88:91], a[108:111], v[140:143]\n" \
  "s_waitcnt lgkmcnt(8)\n" \
  "ds_read_b64_tr_b16 v[80:81], %[vb0] offset:12288\n" \
  "v_mfma_f32_16x16x32_bf16 v[156:159], v[92:95], a[108:111], v[156:159]\n" \
  "ds_read_b64_tr_b16 v[82:83], %[vb0] offset:12544\n" \
  "s_cbranch_scc0 .Lslow8_%=\n" \
  ".Lback8_%=:\n" \
  "v_mfma_f32_16x16x32_bf16 v[152:155], v[92:95], a[104:107], v[152:155]\n" \
  "s_add_u32 %[tag], %[tag], 1\n" \
  "s_add_u32 %[sw], %[sw], 262144\n" \
  "v_mfma_f32_16x16x32_bf16 v[148:151], v[92:95], a[100:103], v[148:151]\n" \
  "ds_read_b64_tr_b16 v[84:85], %[vb1] offset:12288\n" \
  "v_mfma_f32_16x16x32_bf16 v[144:147], v[92:95], a[96:99], v[144:147]\n" \
  "s_waitcnt lgkmcnt(9)\n" \
  "v_mfma_f32_16x16x32_bf16 v[160:163], v[64:67], a[96:99], v[160:163]\n" \
  "ds_read_b64_tr_b16 v[86:87], %[vb1] offset:12544\n" \
  "v_mfma_f32_16x16x32_bf16 v[164:167], v[64:67], a[100:103], v[164:167]\n" \
  "buffer_load_dwordx4 a[0:3], %[vx0], %[xr], %[sx] offen sc1\n" \
  "v_mfma_f32_16x16x32_bf16 v[168:171], v[64:67], a[104:107], v[168:171]\n" \
  "ds_read_b64_tr_b16 v[88:89], %[va0] offset:53248\n" \
  "v_mfma_f32_16x16x32_bf16 v[172:175], v[64:67], a[108:111], v[172:175]\n" \
  "s_waitcnt lgkmcnt(9)\n" \
  "v_mfma_f32_16x16x32_bf16 v[188:191], v[68:71], a[108:111], v[188:191]\n" \
  "ds_read_b64_tr_b16 v[90:91], %[va0] offset:53760\n" \
  "v_mfma_f32_16x16x32_bf16 v[184:187], v[68:71], a[104:107], v[184:187]\n" \
  "buffer_load_dwordx4 a[4:7], %[vx1], %[xr], %[sx] offen sc1\n" \
  "v_mfma_f32_16x16x32_bf16 v[180:183], v[68:71], a[100:103], v[180:183]\n" \
  "ds_read_b64_tr_b16 v[92:93], %[va1] offset:53248\n" \
  "v_mfma_f32_16x16x32_bf16 v[176:179], v[68:71], a[96:99], v[176:179]\n" \
  "s_waitcnt lgkmcnt(9)\n" \
  "ds_read_b64_tr_b16 v[94:95], %[va1] offset:53760\n" \
  "v_mfma_f32_16x16x32_bf16 v[192:195], v[72:75], a[96:99], v[192:195]\n" \
  "ds_read_b64_tr_b16 v[64:65], %[va2] offset:53248\n" \
  "v_mfma_f32_16x16x32_bf16 v[196:199], v[72:75], a[100:103], v[196:199]\n" \
  "buffer_load_dwordx4 a[8:11], %[vx2], %[xr], %[sx] offen sc1\n" \
  "v_mfma_f32_16x16x32_bf16 v[200:203], v[72:75], a[104:107], v[200:203]\n" \
  "ds_read_b64_tr_b16 v[66:67], %[va2] offset:53760\n" \
  "v_mfma_f32_16x16x32_bf16 v[204:207], v[72:75], a[108:111], v[204:207]\n" \
  "s_waitcnt lgkmcnt(10)\n" \
  "v_mfma_f32_16x16x32_bf16 v[220:223], v[76:79], a[108:111], v[220:223]\n" \
  "ds_read_b64_tr_b16 v[68:69], %[va3] offset:53248\n" \
  "v_mfma_f32_16x16x32_bf16 v[216:219], v[76:79], a[104:107], v[216:219]\n" \
  "buffer_load_dwordx4 a[12:15], %[vx3], %[xr], %[sx] offen sc1\n" \
  "v_mfma_f32_16x16x32_bf16 v[212:215], v[76:79], a[100:103], v[212:215]\n" \
  "ds_read_b64_tr_b16 v[70:71], %[va3] offset:53760\n" \
  "s_load_dword s64, %[fp], %[sl] offset:0x14 glc\n" \
  "v_mfma_f32_16x16x32_bf16 v[208:211], v[76:79], a[96:99], v[208:211]\n" \
  "s_waitcnt lgkmcnt(8)\n" \
  "s_add_u32 %[sx], %[sx], 4096\n" \
  "v_mfma_f32_16x16x32_bf16 v[224:227], v[80:83], a[96:99], v[224:227]\n" \
  "ds_read_b64_tr_b16 v[72:73], %[va0] offset:61440\n" \
  "v_mfma_f32_16x16x32_bf16 v[228:231], v[80:83], a[100:103], v[228:231]\n" \
  "v_mfma_f32_16x16x32_bf16 v[232:235], v[80:83], a[104:107], v[232:235]\n" \
  "ds_read_b64_tr_b16 v[74:75], %[va0] offset:61952\n" \
  "v_mfma_f32_16x16x32_bf16 v[236:239], v[80:83], a[108:111], v[236:239]\n" \
  "v_mfma_f32_16x16x32_bf16 v[240:243], v[84:87], a[108:111], v[240:243]\n" \
  "s_waitcnt vmcnt(9) lgkmcnt(8)\n" \
  "s_cmp_ge_u32 s65, %[tag]\n" \
  "v_mfma_f32_16x16x32_bf16 v[96:99], v[88:91], a[112:115], v[96:99]\n" \
  "ds_read_b64_tr_b16 v[76:77], %[va1] offset:61440\n" \
  "s_cbranch_scc0 .Lslow9_%=\n" \
  ".Lback9_%=:\n" \
  "v_mfma_f32_16x16x32_bf16 v[100:103], v[88:91], a[116:119], v[100:103]\n" \
  "s_add_u32 %[tag], %[tag], 1\n" \
  "v_mfma_f32_16x16x32_bf16 v[104:107], v[88:91], a[120:123], v[104:107]\n" \
  "ds_read_b64_tr_b16 v[78:79], %[va1] offset:61952\n" \
  "v_mfma_f32_16x16x32_bf16 v[108:111], v[88:91], a[124:127], v[108:111]\n" \
  "s_waitcnt lgkmcnt(8)\n" \
  "v_mfma_f32_16x16x32_bf16 v[124:127], v[92:95], a[124:127], v[124:127]\n" \
  "ds_read_b64_tr_b16 v[80:81], %[va2] offset:61440\n" \
  "v_mfma_f32_16x16x32_bf16 v[120:123], v[92:95], a[120:123], v[120:123]\n" \
  "buffer_load_dwordx4 a[16:19], %[vx0], %[xr], %[sx] offen sc1\n" \
  "v_mfma_f32_16x16x32_bf16 v[116:119], v[92:95], a[116:119], v[116:119]\n" \
  "ds_read_b64_tr_b16 v[82:83], %[va2] offset:61952\n" \
  "v_mfma_f32_16x16x32_bf16 v[112:115], v[92:95], a[112:115], v[112:115]\n" \
  "s_waitcnt lgkmcnt(8)\n" \
  "v_mfma_f32_16x16x32_bf16 v[128:131], v[64:67], a[112:115], v[128:131]\n" \
  "ds_read_b64_tr_b16 v[84:85], %[va3] offset:61440\n" \
  "v_mfma_f32_16x16x32_bf16 v[132:135], v[64:67], a[116:119], v[132:135]\n" \
  "buffer_load_dwordx4 a[20:23], %[vx1], %[xr], %[sx] offen sc1\n" \
  "v_mfma_f32_16x16x32_bf16 v[136:139], v[64:67], a[120:123], v[136:139]\n" \
  "ds_read_b64_tr_b16 v[86:87], %[va3] offset:61952\n" \
  "v_mfma_f32_16x16x32_bf16 v[140:143], v[64:67], a[124:127], v[140:143]\n" \
  "s_waitcnt lgkmcnt(8)\n" \
  "ds_read_b64_tr_b16 v[88:89], %[vb0] offset:14336\n" \
  "v_mfma_f32_16x16x32_bf16 v[156:159], v[68:71], a[124:127], v[156:159]\n" \
  "ds_read_b64_tr_b16 v[90:91], %[vb0] offset:14592\n" \
  "v_mfma_f32_16x16x32_bf16 v[152:155], v[68:71], a[120:123], v[152:155]\n" \
  "buffer_load_dwordx4 a[24:27], %[vx2], %[xr], %[sx] offen sc1\n" \
  "v_mfma_f32_16x16x32_bf16 v[148:151], v[68:71], a[116:119], v[148:151]\n" \
  "ds_read_b64_tr_b16 v[92:93], %[vb1] offset:14336\n" \
  "v_mfma_f32_16x16x32_bf16 v[144:147], v[68:71], a[112:115], v[144:147]\n" \
  "s_waitcnt lgkmcnt(9)\n" \
  "v_mfma_f32_16x16x32_bf16 v[160:163], v[72:75], a[112:115], v[160:163]\n" \
  "ds_read_b64_tr_b16 v[94:95], %[vb1] offset:14592\n" \
  "v_mfma_f32_16x16x32_bf16 v[164:167], v[72:75], a[116:119], v[164:167]\n" \
  "s_barrier\n" \
  "v_mfma_f32_16x16x32_bf16 v[168:171], v[72:75], a[120:123], v[168:171]\n" \
  "ds_read_b64_tr_b16 v[64:65], %[va0] offset:0\n" \
  "s_load_dword s65, %[fp], %[sl] offset:0x814 glc\n" \
  "v_mfma_f32_16x16x32_bf16 v[172:175], v[72:75], a[124:127], v[172:175]\n" \
  "s_waitcnt lgkmcnt(9)\n" \
  "s_add_u32 m0, %[ma], 32768\n" \
  "v_mfma_f32_16x16x32_bf16 v[188:191], v[76:79], a[124:127], v[188:191]\n" \
  "ds_read_b64_tr_b16 v[66:67], %[va0] offset:512\n" \
  "v_mfma_f32_16x16x32_bf16 v[184:187], v[76:79], a[120:123], v[184:187]\n" \
  "buffer_load_dwordx4 a[28:31], %[vx3], %[xr], %[sx] offen sc1\n" \
  "v_mfma_f32_16x16x32_bf16 v[180:183], v[76:79], a[116:119], v[180:183]\n" \
  "ds_read_b64_tr_b16 v[68:69], %[va1] offset:0\n" \
  "s_add_u32 %[sx], %[sx], 4096\n" \
  "v_mfma_f32_16x16x32_bf16 v[176:179], v[76:79], a[112:115], v[176:179]\n" \
  "s_waitcnt lgkmcnt(9)\n" \
  "ds_read_b64_tr_b16 v[70:71], %[va1] offset:512\n" \
  "v_mfma_f32_16x16x32_bf16 v[192:195], v[80:83], a[112:115], v[192:195]\n" \
  "ds_read_b64_tr_b16 v[72:73], %[va2] offset:0\n" \
  "v_mfma_f32_16x16x32_bf16 v[196:199], v[80:83], a[116:119], v[196:199]\n" \
  "buffer_load_dwordx4 %[vd0], %[wr], %[sw] offen offset:0 lds\n" \
  "v_mfma_f32_16x16x32_bf16 v[200:203], v[80:83], a[120:123], v[200:203]\n" \
  "ds_read_b64_tr_b16 v[74:75], %[va2] offset:512\n" \
  "v_mfma_f32_16x16x32_bf16 v[204:207], v[80:83], a[124:127], v[204:207]\n" \
  "s_waitcnt lgkmcnt(10)\n" \
  "v_mfma_f32_16x16x32_bf16 v[220:223], v[84:87], a[124:127], v[220:223]\n" \
  "ds_read_b64_tr_b16 v[76:77], %[va3] offset:0\n" \
  "v_mfma_f32_16x16x32_bf16 v[216:219], v[84:87], a[120:123], v[216:219]\n" \
  "v_mfma_f32_16x16x32_bf16 v[212:215], v[84:87], a[116:119], v[212:215]\n" \
  "ds_read_b64_tr_b16 v[78:79], %[va3] offset:512\n" \
  "v_mfma_f32_16x16x32_bf16 v[208:211], v[84:87], a[112:115], v[208:211]\n" \
  "s_waitcnt lgkmcnt(8)\n" \
  "v_mfma_f32_16x16x32_bf16 v[224:227], v[88:91], a[112:115], v[224:227]\n" \
  "ds_read_b64_tr_b16 v[80:81], %[va0] offset:8192\n" \
  "v_mfma_f32_16x16x32_bf16 v[228:231], v[88:91], a[116:119], v[228:231]\n" \
  "buffer_load_dwordx4 %[vd1], %[wr], %[sw] offen offset:1024 lds\n" \
  "v_mfma_f32_16x16x32_bf16 v[232:235], v[88:91], a[120:123], v[232:235]\n" \
  "ds_read_b64_tr_b16 v[82:83], %[va0] offset:8704\n" \
  "s_add_u32 %[sl], %[sl], 16\n" \
  "v_mfma_f32_16x16x32_bf16 v[236:239], v[88:91], a[124:127], v[236:239]\n" \
  "s_cmp_lg_u32 %[sl], 240\n" \
  "v_mfma_f32_16x16x32_bf16 v[240:243], v[92:95], a[124:127], v[240:243]\n" \
  "s_cbranch_scc1 .Lloop_%=\n" \
  ".Lfin_%=:\n" \
  "s_waitcnt vmcnt(6) lgkmcnt(8)\n" \
  "v_mfma_f32_16x16x32_bf16 v[96:99], v[64:67], a[0:3], v[96:99]\n" \
  "ds_read_b64_tr_b16 v[84:85], %[va1] offset:8192\n" \
  "v_mfma_f32_16x16x32_bf16 v[100:103], v[64:67], a[4:7], v[100:103]\n" \
  "buffer_load_dwordx4 %[vd2], %[wr], %[sw] offen offset:2048 lds\n" \
  "v_mfma_f32_16x16x32_bf16 v[104:107], v[64:67], a[8:11], v[104:107]\n" \
  "ds_read_b64_tr_b16 v[86:87], %[va1] offset:8704\n" \
  "v_mfma_f32_16x16x32_bf16 v[108:111], v[64:67], a[12:15], v[108:111]\n" \
  "s_waitcnt lgkmcnt(8)\n" \
  "v_mfma_f32_16x16x32_bf16 v[124:127], v[68:71], a[12:15], v[124:127]\n" \
  "ds_read_b64_tr_b16 v[88:89], %[va2] offset:8192\n" \
  "v_mfma_f32_16x16x32_bf16 v[120:123], v[68:71], a[8:11], v[120:123]\n" \
  "buffer_load_dwordx4 %[vd3], %[wr], %[sw] offen offset:3072 lds\n" \
  "v_mfma_f32_16x16x32_bf16 v[116:119], v[68:71], a[4:7], v[116:119]\n" \
  "ds_read_b64_tr_b16 v[90:91], %[va2] offset:8704\n" \
  "s_add_u32 m0, %[mb], 8192\n" \
  "v_mfma_f32_16x16x32_bf16 v[112:115], v[68:71], a[0:3], v[112:115]\n" \
  "s_waitcnt lgkmcnt(8)\n" \
  "v_mfma_f32_16x16x32_bf16 v[128:131], v[72:75], a[0:3], v[128:131]\n" \
  "ds_read_b64_tr_b16 v[92:93], %[va3] offset:8192\n" \
  "v_mfma_f32_16x16x32_bf16 v[132:135], v[72:75], a[4:7], v[132:135]\n" \
  "buffer_load_dwordx4 %[vd4], %[wr], %[sw] offen lds\n" \
  "v_mfma_f32_16x16x32_bf16 v[136:139], v[72:75], a[8:11], v[136:139]\n" \
  "ds_read_b64_tr_b16 v[94:95], %[va3] offset:8704\n" \
  "s_cmp_ge_u32 s64, %[tag]\n" \
  "v_mfma_f32_16x16x32_bf16 v[140:143], v[72:75], a[12:15], v[140:143]\n" \
  "s_waitcnt lgkmcnt(8)\n" \
  "ds_read_b64_tr_b16 v[64:65], %[vb0] offset:0\n" \
  "v_mfma_f32_16x16x32_bf16 v[156:159], v[76:79], a[12:15], v[156:159]\n" \
  "ds_read_b64_tr_b16 v[66:67], %[vb0] offset:256\n" \
  "s_cbranch_scc0 .Lslow10_%=\n" \
  ".Lback10_%=:\n" \
  "v_mfma_f32_16x16x32_bf16 v[152:155], v[76:79], a[8:11], v[152:155]\n" \
  "s_add_u32 %[tag], %[tag], 1\n" \
  "s_add_u32 %[sw], %[sw], 262144\n" \
  "v_mfma_f32_16x16x32_bf16 v[148:151], v[76:79], a[4:7], v[148:151]\n" \
  "ds_read_b64_tr_b16 v[68:69], %[vb1] offset:0\n" \
  "v_mfma_f32_16x16x32_bf16 v[144:147], v[76:79], a[0:3], v[144:147]\n" \
  "s_waitcnt lgkmcnt(9)\n" \
  "v_mfma_f32_16x16x32_bf16 v[160:163], v[80:83], a[0:3], v[160:163]\n" \
  "ds_read_b64_tr_b16 v[70:71], %[vb1] offset:256\n" \
  "v_mfma_f32_16x16x32_bf16 v[164:167], v[80:83], a[4:7], v[164:167]\n" \
  "buffer_load_dwordx4 a[32:35], %[vx0], %[xr], %[sx] offen sc1\n" \
  "v_mfma_f32_16x16x32_bf16 v[168:171], v[80:83], a[8:11], v[168:171]\n" \
  "ds_read_b64_tr_b16 v[72:73], %[va0] offset:4096\n" \
  "v_mfma_f32_16x16x32_bf16 v[172:175], v[80:83], a[12:15], v[172:175]\n" \
  "s_waitcnt lgkmcnt(9)\n" \
  "v_mfma_f32_16x16x32_bf16 v[188:191], v[84:87], a[12:15], v[188:191]\n" \
  "ds_read_b64_tr_b16 v[74:75], %[va0] offset:4608\n" \
  "v_mfma_f32_16x16x32_bf16 v[184:187], v[84:87], a[8:11], v[184:187]\n" \
  "buffer_load_dwordx4 a[36:39], %[vx1], %[xr], %[sx] offen sc1\n" \
  "v_mfma_f32_16x16x32_bf16 v[180:183], v[84:87], a[4:7], v[180:183]\n" \
  "ds_read_b64_tr_b16 v[76:77], %[va1] offset:4096\n" \
  "v_mfma_f32_16x16x32_bf16 v[176:179], v[84:87], a[0:3], v[176:179]\n" \
  "s_waitcnt lgkmcnt(9)\n" \
  "ds_read_b64_tr_b16 v[78:79], %[va1] offset:4608\n" \
  "v_mfma_f32_16x16x32_bf16 v[192:195], v[88:91], a[0:3], v[192:195]\n" \
  "ds_read_b64_tr_b16 v[80:81], %[va2] offset:4096\n" \
  "v_mfma_f32_16x16x32_bf16 v[196:199], v[88:91], a[4:7], v[196:199]\n" \
  "buffer_load_dwordx4 a[40:43], %[vx2], %[xr], %[sx] offen sc1\n" \
  "v_mfma_f32_16x16x32_bf16 v[200:203], v[88:91], a[8:11], v[200:203]\n" \
  "ds_read_b64_tr_b16 v[82:83], %[va2] offset:4608\n" \
  "v_mfma_f32_16x16x32_bf16 v[204:207], v[88:91], a[12:15], v[204:207]\n" \
  "s_waitcnt lgkmcnt(10)\n" \
  "v_mfma_f32_16x16x32_bf16 v[220:223], v[92:95], a[12:15], v[220:223]\n" \
  "ds_read_b64_tr_b16 v[84:85], %[va3] offset:4096\n" \
  "v_mfma_f32_16x16x32_bf16 v[216:219], v[92:95], a[8:11], v[216:219]\n" \
  "buffer_load_dwordx4 a[44:47], %[vx3], %[xr], %[sx] offen sc1\n" \
  "v_mfma_f32_16x16x32_bf16 v[212:215], v[92:95], a[4:7], v[212:215]\n" \
  "ds_read_b64_tr_b16 v[86:87], %[va3] offset:4608\n" \
  "s_load_dword s64, %[fp], %[sl] offset:0x8 glc\n" \
  "v_mfma_f32_16x16x32_bf16 v[208:211], v[92:95], a[0:3], v[208:211]\n" \
  "s_waitcnt lgkmcnt(8)\n" \
  "s_add_u32 %[sx], %[sx], 4096\n" \
  "v_mfma_f32_16x16x32_bf16 v[224:227], v[64:67], a[0:3], v[224:227]\n" \
  "ds_read_b64_tr_b16 v[88:89], %[va0] offset:12288\n" \
  "v_mfma_f32_16x16x32_bf16 v[228:231], v[64:67], a[4:7], v[228:231]\n" \
  "v_mfma_f32_16x16x32_bf16 v[232:235], v[64:67], a[8:11], v[232:235]\n" \
  "ds_read_b64_tr_b16 v[90:91], %[va0] offset:12800\n" \
  "v_mfma_f32_16x16x32_bf16 v[236:239], v[64:67], a[12:15], v[236:239]\n" \
  "v_mfma_f32_16x16x32_bf16 v[240:243], v[68:71], a[12:15], v[240:243]\n" \
  "s_waitcnt vmcnt(9) lgkmcnt(8)\n" \
  "s_cmp_ge_u32 s65, %[tag]\n" \
  "v_mfma_f32_16x16x32_bf16 v[96:99], v[72:75], a[16:19], v[96:99]\n" \
  "ds_read_b64_tr_b16 v[92:93], %[va1] offset:12288\n" \
  "s_cbranch_scc0 .Lslow11_%=\n" \
  ".Lback11_%=:\n" \
  "v_mfma_f32_16x16x32_bf16 v[100:103], v[72:75], a[20:23], v[100:103]\n" \
  "s_add_u32 %[tag], %[tag], 1\n" \
  "v_mfma_f32_16x16x32_bf16 v[104:107], v[72:75], a[24:27], v[104:107]\n" \
  "ds_read_b64_tr_b16 v[94:95], %[va1] offset:12800\n" \
  "v_mfma_f32_16x16x32_bf16 v[108:111], v[72:75], a[28:31], v[108:111]\n" \
  "s_waitcnt lgkmcnt(8)\n" \
  "v_mfma_f32_16x16x32_bf16 v[124:127], v[76:79], a[28:31], v[124:127]\n" \
  "ds_read_b64_tr_b16 v[64:65], %[va2] offset:12288\n" \
  "v_mfma_f32_16x16x32_bf16 v[120:123], v[76:79], a[24:27], v[120:123]\n" \
  "buffer_load_dwordx4 a[48:51], %[vx0], %[xr], %[sx] offen sc1\n" \
  "v_mfma_f32_16x16x32_bf16 v[116:119], v[76:79], a[20:23], v[116:119]\n" \
  "ds_read_b64_tr_b16 v[66:67], %[va2] offset:12800\n" \
  "v_mfma_f32_16x16x32_bf16 v[112:115], v[76:79], a[16:19], v[112:115]\n" \
  "s_waitcnt lgkmcnt(8)\n" \
  "v_mfma_f32_16x16x32_bf16 v[128:131], v[80:83], a[16:19], v[128:131]\n" \
  "ds_read_b64_tr_b16 v[68:69], %[va3] offset:12288\n" \
  "v_mfma_f32_16x16x32_bf16 v[132:135], v[80:83], a[20:23], v[132:135]\n" \
  "buffer_load_dwordx4 a[52:55], %[vx1], %[xr], %[sx] offen sc1\n" \
  "v_mfma_f32_16x16x32_bf16 v[136:139], v[80:83], a[24:27], v[136:139]\n" \
  "ds_read_b64_tr_b16 v[70:71], %[va3] offset:12800\n" \
  "v_mfma_f32_16x16x32_bf16 v[140:143], v[80:83], a[28:31], v[140:143]\n" \
  "s_waitcnt lgkmcnt(8)\n" \
  "ds_read_b64_tr_b16 v[72:73], %[vb0] offset:2048\n" \
  "v_mfma_f32_16x16x32_bf16 v[156:159], v[84:87], a[28:31], v[156:159]\n" \
  "ds_read_b64_tr_b16 v[74:75], %[vb0] offset:2304\n" \
  "v_mfma_f32_16x16x32_bf16 v[152:155], v[84:87], a[24:27], v[152:155]\n" \
  "buffer_load_dwordx4 a[56:59], %[vx2], %[xr], %[sx] offen sc1\n" \
  "v_mfma_f32_16x16x32_bf16 v[148:151], v[84:87], a[20:23], v[148:151]\n" \
  "ds_read_b64_tr_b16 v[76:77], %[vb1] offset:2048\n" \
  "v_mfma_f32_16x16x32_bf16 v[144:147], v[84:87], a[16:19], v[144:147]\n" \
  "s_waitcnt lgkmcnt(9)\n" \
  "v_mfma_f32_16x16x32_bf16 v[160:163], v[88:91], a[16:19], v[160:163]\n" \
  "ds_read_b64_tr_b16 v[78:79], %[vb1] offset:2304\n" \
  "v_mfma_f32_16x16x32_bf16 v[164:167], v[88:91], a[20:23], v[164:167]\n" \
  "s_barrier\n" \
  "v_mfma_f32_16x16x32_bf16 v[168:171], v[88:91], a[24:27], v[168:171]\n" \
  "ds_read_b64_tr_b16 v[80:81], %[va0] offset:16384\n" \
  "s_load_dword s65, %[fp], %[sl] offset:0x808 glc\n" \
  "v_mfma_f32_16x16x32_bf16 v[172:175], v[88:91], a[28:31], v[172:175]\n" \
  "s_waitcnt lgkmcnt(9)\n" \
  "s_add_u32 m0, %[ma], 49152\n" \
  "v_mfma_f32_16x16x32_bf16 v[188:191], v[92:95], a[28:31], v[188:191]\n" \
  "ds_read_b64_tr_b16 v[82:83], %[va0] offset:16896\n" \
  "v_mfma_f32_16x16x32_bf16 v[184:187], v[92:95], a[24:27], v[184:187]\n" \
  "buffer_load_dwordx4 a[60:63], %[vx3], %[xr], %[sx] offen sc1\n" \
  "v_mfma_f32_16x16x32_bf16 v[180:183], v[92:95], a[20:23], v[180:183]\n" \
  "ds_read_b64_tr_b16 v[84:85], %[va1] offset:16384\n" \
  "s_add_u32 %[sx], %[sx], 4096\n" \
  "v_mfma_f32_16x16x32_bf16 v[176:179], v[92:95], a[16:19], v[176:179]\n" \
  "s_waitcnt lgkmcnt(9)\n" \
  "ds_read_b64_tr_b16 v[86:87], %[va1] offset:16896\n" \
  "v_mfma_f32_16x16x32_bf16 v[192:195], v[64:67], a[16:19], v[192:195]\n" \
  "ds_read_b64_tr_b16 v[88:89], %[va2] offset:16384\n" \
  "v_mfma_f32_16x16x32_bf16 v[196:199], v[64:67], a[20:23], v[196:199]\n" \
  "buffer_load_dwordx4 %[vd0], %[wr], %[sw] offen offset:0 lds\n" \
  "v_mfma_f32_16x16x32_bf16 v[200:203], v[64:67], a[24:27], v[200:203]\n" \
  "ds_read_b64_tr_b16 v[90:91], %[va2] offset:16896\n" \
  "v_mfma_f32_16x16x32_bf16 v[204:207], v[64:67], a[28:31], v[204:207]\n" \
  "s_waitcnt lgkmcnt(10)\n" \
  "v_mfma_f32_16x16x32_bf16 v[220:223], v[68:71], a[28:31], v[220:223]\n" \
  "ds_read_b64_tr_b16 v[92:93], %[va3] offset:16384\n" \
  "v_mfma_f32_16x16x32_bf16 v[216:219], v[68:71], a[24:27], v[216:219]\n" \
  "v_mfma_f32_16x16x32_bf16 v[212:215], v[68:71], a[20:23], v[212:215]\n" \
  "ds_read_b64_tr_b16 v[94:95], %[va3] offset:16896\n" \
  "v_mfma_f32_16x16x32_bf16 v[208:211], v[68:71], a[16:19], v[208:211]\n" \
  "s_waitcnt lgkmcnt(8)\n" \
  "v_mfma_f32_16x16x32_bf16 v[224:227], v[72:75], a[16:19], v[224:227]\n" \
  "ds_read_b64_tr_b16 v[64:65], %[va0] offset:24576\n" \
  "v_mfma_f32_16x16x32_bf16 v[228:231], v[72:75], a[20:23], v[228:231]\n" \
  "buffer_load_dwordx4 %[vd1], %[wr], %[sw] offen offset:1024 lds\n" \
  "v_mfma_f32_16x16x32_bf16 v[232:235], v[72:75], a[24:27], v[232:235]\n" \
  "ds_read_b64_tr_b16 v[66:67], %[va0] offset:25088\n" \
  "v_mfma_f32_16x16x32_bf16 v[236:239], v[72:75], a[28:31], v[236:239]\n" \
  "v_mfma_f32_16x16x32_bf16 v[240:243], v[76:79], a[28:31], v[240:243]\n" \
  "s_waitcnt vmcnt(6) lgkmcnt(8)\n" \
  "v_mfma_f32_16x16x32_bf16 v[96:99], v[80:83], a[32:35], v[96:99]\n" \
  "ds_read_b64_tr_b16 v[68:69], %[va1] offset:24576\n" \
  "v_mfma_f32_16x16x32_bf16 v[100:103], v[80:83], a[36:39], v[100:103]\n" \
  "buffer_load_dwordx4 %[vd2], %[wr], %[sw] offen offset:2048 lds\n" \
  "v_mfma_f32_16x16x32_bf16 v[104:107], v[80:83], a[40:43], v[104:107]\n" \
  "ds_read_b64_tr_b16 v[70:71], %[va1] offset:25088\n" \
  "v_mfma_f32_16x16x32_bf16 v[108:111], v[80:83], a[44:47], v[108:111]\n" \
  "s_waitcnt lgkmcnt(8)\n" \
  "v_mfma_f32_16x16x32_bf16 v[124:127], v[84:87], a[44:47], v[124:127]\n" \
  "ds_read_b64_tr_b16 v[72:73], %[va2] offset:24576\n" \
  "v_mfma_f32_16x16x32_bf16 v[120:123], v[84:87], a[40:43], v[120:123]\n" \
  "buffer_load_dwordx4 %[vd3], %[wr], %[sw] offen offset:3072 lds\n" \
  "v_mfma_f32_16x16x32_bf16 v[116:119], v[84:87], a[36:39], v[116:119]\n" \
  "ds_read_b64_tr_b16 v[74:75], %[va2] offset:25088\n" \
  "s_add_u32 m0, %[mb], 12288\n" \
  "v_mfma_f32_16x16x32_bf16 v[112:115], v[84:87], a[32:35], v[112:115]\n" \
  "s_waitcnt lgkmcnt(8)\n" \
  "v_mfma_f32_16x16x32_bf16 v[128:131], v[88:91], a[32:35], v[128:131]\n" \
  "ds_read_b64_tr_b16 v[76:77], %[va3] offset:24576\n" \
  "v_mfma_f32_16x16x32_bf16 v[132:135], v[88:91], a[36:39], v[132:135]\n" \
  "buffer_load_dwordx4 %[vd4], %[wr], %[sw] offen lds\n" \
  "v_mfma_f32_16x16x32_bf16 v[136:139], v[88:91], a[40:43], v[136:139]\n" \
  "ds_read_b64_tr_b16 v[78:79], %[va3] offset:25088\n" \
  "s_cmp_ge_u32 s64, %[tag]\n" \
  "v_mfma_f32_16x16x32_bf16 v[140:143], v[88:91], a[44:47], v[140:143]\n" \
  "s_waitcnt lgkmcnt(8)\n" \
  "ds_read_b64_tr_b16 v[80:81], %[vb0] offset:4096\n" \
  "v_mfma_f32_16x16x32_bf16 v[156:159], v[92:95], a[44:47], v[156:159]\n" \
  "ds_read_b64_tr_b16 v[82:83], %[vb0] offset:4352\n" \
  "s_cbranch_scc0 .Lslow12_%=\n" \
  ".Lback12_%=:\n" \
  "v_mfma_f32_16x16x32_bf16 v[152:155], v[92:95], a[40:43], v[152:155]\n" \
  "s_add_u32 %[tag], %[tag], 1\n" \
  "s_add_u32 %[sw], %[sw], 262144\n" \
  "v_mfma_f32_16x16x32_bf16 v[148:151], v[92:95], a[36:39], v[148:151]\n" \
  "ds_read_b64_tr_b16 v[84:85], %[vb1] offset:4096\n" \
  "v_mfma_f32_16x16x32_bf16 v[144:147], v[92:95], a[32:35], v[144:147]\n" \
  "s_waitcnt lgkmcnt(9)\n" \
  "v_mfma_f32_16x16x32_bf16 v[160:163], v[64:67], a[32:35], v[160:163]\n" \
  "ds_read_b64_tr_b16 v[86:87], %[vb1] offset:4352\n" \
  "v_mfma_f32_16x16x32_bf16 v[164:167], v[64:67], a[36:39], v[164:167]\n" \
  "buffer_load_dwordx4 a[64:67], %[vx0], %[xr], %[sx] offen sc1\n" \
  "v_mfma_f32_16x16x32_bf16 v[168:171], v[64:67], a[40:43], v[168:171]\n" \
  "ds_read_b64_tr_b16 v[88:89], %[va0] offset:20480\n" \
  "v_mfma_f32_16x16x32_bf16 v[172:175], v[64:67], a[44:47], v[172:175]\n" \
  "s_waitcnt lgkmcnt(9)\n" \
  "v_mfma_f32_16x16x32_bf16 v[188:191], v[68:71], a[44:47], v[188:191]\n" \
  "ds_read_b64_tr_b16 v[90:91], %[va0] offset:20992\n" \
  "v_mfma_f32_16x16x32_bf16 v[184:187], v[68:71], a[40:43], v[184:187]\n" \
  "buffer_load_dwordx4 a[68:71], %[vx1], %[xr], %[sx] offen sc1\n" \
  "v_mfma_f32_16x16x32_bf16 v[180:183], v[68:71], a[36:39], v[180:183]\n" \
  "ds_read_b64_tr_b16 v[92:93], %[va1] offset:20480\n" \
  "v_mfma_f32_16x16x32_bf16 v[176:179], v[68:71], a[32:35], v[176:179]\n" \
  "s_waitcnt lgkmcnt(9)\n" \
  "ds_read_b64_tr_b16 v[94:95], %[va1] offset:20992\n" \
  "v_mfma_f32_16x16x32_bf16 v[192:195], v[72:75], a[32:35], v[192:195]\n" \
  "ds_read_b64_tr_b16 v[64:65], %[va2] offset:20480\n" \
  "v_mfma_f32_16x16x32_bf16 v[196:199], v[72:75], a[36:39], v[196:199]\n" \
  "buffer_load_dwordx4 a[72:75], %[vx2], %[xr], %[sx] offen sc1\n" \
  "v_mfma_f32_16x16x32_bf16 v[200:203], v[72:75], a[40:43], v[200:203]\n" \
  "ds_read_b64_tr_b16 v[66:67], %[va2] offset:20992\n" \
  "v_mfma_f32_16x16x32_bf16 v[204:207], v[72:75], a[44:47], v[204:207]\n" \
  "s_waitcnt lgkmcnt(10)\n" \
  "v_mfma_f32_16x16x32_bf16 v[220:223], v[76:79], a[44:47], v[220:223]\n" \
  "ds_read_b64_tr_b16 v[68:69], %[va3] offset:20480\n" \
  "v_mfma_f32_16x16x32_bf16 v[216:219], v[76:79], a[40:43], v[216:219]\n" \
  "buffer_load_dwordx4 a[76:79], %[vx3], %[xr], %[sx] offen sc1\n" \
  "v_mfma_f32_16x16x32_bf16 v[212:215], v[76:79], a[36:39], v[212:215]\n" \
  "ds_read_b64_tr_b16 v[70:71], %[va3] offset:20992\n" \
  "s_load_dword s64, %[fp], %[sl] offset:0xc glc\n" \
  "v_mfma_f32_16x16x32_bf16 v[208:211], v[76:79], a[32:35], v[208:211]\n" \
  "s_waitcnt lgkmcnt(8)\n" \
  "s_add_u32 %[sx], %[sx], 4096\n" \
  "v_mfma_f32_16x16x32_bf16 v[224:227], v[80:83], a[32:35], v[224:227]\n" \
  "ds_read_b64_tr_b16 v[72:73], %[va0] offset:28672\n" \
  "v_mfma_f32_16x16x32_bf16 v[228:231], v[80:83], a[36:39], v[228:231]\n" \
  "v_mfma_f32_16x16x32_bf16 v[232:235], v[80:83], a[40:43], v[232:235]\n" \
  "ds_read_b64_tr_b16 v[74:75], %[va0] offset:29184\n" \
  "v_mfma_f32_16x16x32_bf16 v[236:239], v[80:83], a[44:47], v[236:239]\n" \
  "v_mfma_f32_16x16x32_bf16 v[240:243], v[84:87], a[44:47], v[240:243]\n" \
  "s_waitcnt vmcnt(9) lgkmcnt(8)\n" \
  "s_cmp_ge_u32 s65, %[tag]\n" \
  "v_mfma_f32_16x16x32_bf16 v[96:99], v[88:91], a[48:51], v[96:99]\n" \
  "ds_read_b64_tr_b16 v[76:77], %[va1] offset:28672\n" \
  "s_cbranch_scc0 .Lslow13_%=\n" \
  ".Lback13_%=:\n" \
  "v_mfma_f32_16x16x32_bf16 v[100:103], v[88:91], a[52:55], v[100:103]\n" \
  "s_add_u32 %[tag], %[tag], 1\n" \
  "v_mfma_f32_16x16x32_bf16 v[104:107], v[88:91], a[56:59], v[104:107]\n" \
  "ds_read_b64_tr_b16 v[78:79], %[va1] offset:29184\n" \
  "v_mfma_f32_16x16x32_bf16 v[108:111], v[88:91], a[60:63], v[108:111]\n" \
  "s_waitcnt lgkmcnt(8)\n" \
  "v_mfma_f32_16x16x32_bf16 v[124:127], v[92:95], a[60:63], v[124:127]\n" \
  "ds_read_b64_tr_b16 v[80:81], %[va2] offset:28672\n" \
  "v_mfma_f32_16x16x32_bf16 v[120:123], v[92:95], a[56:59], v[120:123]\n" \
  "buffer_load_dwordx4 a[80:83], %[vx0], %[xr], %[sx] offen sc1\n" \
  "v_mfma_f32_16x16x32_bf16 v[116:119], v[92:95], a[52:55], v[116:119]\n" \
  "ds_read_b64_tr_b16 v[82:83], %[va2] offset:29184\n" \
  "v_mfma_f32_16x16x32_bf16 v[112:115], v[92:95], a[48:51], v[112:115]\n" \
  "s_waitcnt lgkmcnt(8)\n" \
  "v_mfma_f32_16x16x32_bf16 v[128:131], v[64:67], a[48:51], v[128:131]\n" \
  "ds_read_b64_tr_b16 v[84:85], %[va3] offset:28672\n" \
  "v_mfma_f32_16x16x32_bf16 v[132:135], v[64:67], a[52:55], v[132:135]\n" \
  "buffer_load_dwordx4 a[84:87], %[vx1], %[xr], %[sx] offen sc1\n" \
  "v_mfma_f32_16x16x32_bf16 v[136:139], v[64:67], a[56:59], v[136:139]\n" \
  "ds_read_b64_tr_b16 v[86:87], %[va3] offset:29184\n" \
  "v_mfma_f32_16x16x32_bf16 v[140:143], v[64:67], a[60:63], v[140:143]\n" \
  "s_waitcnt lgkmcnt(8)\n" \
  "ds_read_b64_tr_b16 v[88:89], %[vb0] offset:6144\n" \
  "v_mfma_f32_16x16x32_bf16 v[156:159], v[68:71], a[60:63], v[156:159]\n" \
  "ds_read_b64_tr_b16 v[90:91], %[vb0] offset:6400\n" \
  "v_mfma_f32_16x16x32_bf16 v[152:155], v[68:71], a[56:59], v[152:155]\n" \
  "buffer_load_dwordx4 a[88:91], %[vx2], %[xr], %[sx] offen sc1\n" \
  "v_mfma_f32_16x16x32_bf16 v[148:151], v[68:71], a[52:55], v[148:151]\n" \
  "ds_read_b64_tr_b16 v[92:93], %[vb1] offset:6144\n" \
  "v_mfma_f32_16x16x32_bf16 v[144:147], v[68:71], a[48:51], v[144:147]\n" \
  "s_waitcnt lgkmcnt(9)\n" \
  "v_mfma_f32_16x16x32_bf16 v[160:163], v[72:75], a[48:51], v[160:163]\n" \
  "ds_read_b64_tr_b16 v[94:95], %[vb1] offset:6400\n" \
  "v_mfma_f32_16x16x32_bf16 v[164:167], v[72:75], a[52:55], v[164:167]\n" \
  "s_barrier\n" \
  "v_mfma_f32_16x16x32_bf16 v[168:171], v[72:75], a[56:59], v[168:171]\n" \
  "ds_read_b64_tr_b16 v[64:65], %[va0] offset:32768\n" \
  "s_load_dword s65, %[fp], %[sl] offset:0x80c glc\n" \
  "v_mfma_f32_16x16x32_bf16 v[172:175], v[72:75], a[60:63], v[172:175]\n" \
  "s_waitcnt lgkmcnt(9)\n" \
  "v_mfma_f32_16x16x32_bf16 v[188:191], v[76:79], a[60:63], v[188:191]\n" \
  "ds_read_b64_tr_b16 v[66:67], %[va0] offset:33280\n" \
  "v_mfma_f32_16x16x32_bf16 v[184:187], v[76:79], a[56:59], v[184:187]\n" \
  "buffer_load_dwordx4 a[92:95], %[vx3], %[xr], %[sx] offen sc1\n" \
  "v_mfma_f32_16x16x32_bf16 v[180:183], v[76:79], a[52:55], v[180:183]\n" \
  "ds_read_b64_tr_b16 v[68:69], %[va1] offset:32768\n" \
  "s_add_u32 %[sx], %[sx], 4096\n" \
  "v_mfma_f32_16x16x32_bf16 v[176:179], v[76:79], a[48:51], v[176:179]\n" \
  "s_waitcnt lgkmcnt(9)\n" \
  "ds_read_b64_tr_b16 v[70:71], %[va1] offset:33280\n" \
  "v_mfma_f32_16x16x32_bf16 v[192:195], v[80:83], a[48:51], v[192:195]\n" \
  "ds_read_b64_tr_b16 v[72:73], %[va2] offset:32768\n" \
  "v_mfma_f32_16x16x32_bf16 v[196:199], v[80:83], a[52:55], v[196:199]\n" \
  "v_mfma_f32_16x16x32_bf16 v[200:203], v[80:83], a[56:59], v[200:203]\n" \
  "ds_read_b64_tr_b16 v[74:75], %[va2] offset:33280\n" \
  "v_mfma_f32_16x16x32_bf16 v[204:207], v[80:83], a[60:63], v[204:207]\n" \
  "s_waitcnt lgkmcnt(10)\n" \
  "v_mfma_f32_16x16x32_bf16 v[220:223], v[84:87], a[60:63], v[220:223]\n" \
  "ds_read_b64_tr_b16 v[76:77], %[va3] offset:32768\n" \
  "v_mfma_f32_16x16x32_bf16 v[216:219], v[84:87], a[56:59], v[216:219]\n" \
  "v_mfma_f32_16x16x32_bf16 v[212:215], v[84:87], a[52:55], v[212:215]\n" \
  "ds_read_b64_tr_b16 v[78:79], %[va3] offset:33280\n" \
  "v_mfma_f32_16x16x32_bf16 v[208:211], v[84:87], a[48:51], v[208:211]\n" \
  "s_waitcnt lgkmcnt(8)\n" \
  "v_mfma_f32_16x16x32_bf16 v[224:227], v[88:91], a[48:51], v[224:227]\n" \
  "ds_read_b64_tr_b16 v[80:81], %[va0] offset:40960\n" \
  "v_mfma_f32_16x16x32_bf16 v[228:231], v[88:91], a[52:55], v[228:231]\n" \
  "v_mfma_f32_16x16x32_bf16 v[232:235], v[88:91], a[56:59], v[232:235]\n" \
  "ds_read_b64_tr_b16 v[82:83], %[va0] offset:41472\n" \
  "v_mfma_f32_16x16x32_bf16 v[236:239], v[88:91], a[60:63], v[236:239]\n" \
  "v_mfma_f32_16x16x32_bf16 v[240:243], v[92:95], a[60:63], v[240:243]\n" \
  "s_waitcnt vmcnt(4) lgkmcnt(8)\n" \
  "v_mfma_f32_16x16x32_bf16 v[96:99], v[64:67], a[64:67], v[96:99]\n" \
  "ds_read_b64_tr_b16 v[84:85], %[va1] offset:40960\n" \
  "v_mfma_f32_16x16x32_bf16 v[100:103], v[64:67], a[68:71], v[100:103]\n" \
  "v_mfma_f32_16x16x32_bf16 v[104:107], v[64:67], a[72:75], v[104:107]\n" \
  "ds_read_b64_tr_b16 v[86:87], %[va1] offset:41472\n" \
  "v_mfma_f32_16x16x32_bf16 v[108:111], v[64:67], a[76:79], v[108:111]\n" \
  "s_waitcnt lgkmcnt(8)\n" \
  "v_mfma_f32_16x16x32_bf16 v[124:127], v[68:71], a[76:79], v[124:127]\n" \
  "ds_read_b64_tr_b16 v[88:89], %[va2] offset:40960\n" \
  "v_mfma_f32_16x16x32_bf16 v[120:123], v[68:71], a[72:75], v[120:123]\n" \
  "v_mfma_f32_16x16x32_bf16 v[116:119], v[68:71], a[68:71], v[116:119]\n" \
  "ds_read_b64_tr_b16 v[90:91], %[va2] offset:41472\n" \
  "v_mfma_f32_16x16x32_bf16 v[112:115], v[68:71], a[64:67], v[112:115]\n" \
  "s_waitcnt lgkmcnt(8)\n" \
  "v_mfma_f32_16x16x32_bf16 v[128:131], v[72:75], a[64:67], v[128:131]\n" \
  "ds_read_b64_tr_b16 v[92:93], %[va3] offset:40960\n" \
  "v_mfma_f32_16x16x32_bf16 v[132:135], v[72:75], a[68:71], v[132:135]\n" \
  "v_mfma_f32_16x16x32_bf16 v[136:139], v[72:75], a[72:75], v[136:139]\n" \
  "ds_read_b64_tr_b16 v[94:95], %[va3] offset:41472\n" \
  "s_cmp_ge_u32 s64, %[tag]\n" \
  "v_mfma_f32_16x16x32_bf16 v[140:143], v[72:75], a[76:79], v[140:143]\n" \
  "s_waitcnt lgkmcnt(8)\n" \
  "ds_read_b64_tr_b16 v[64:65], %[vb0] offset:8192\n" \
  "v_mfma_f32_16x16x32_bf16 v[156:159], v[76:79], a[76:79], v[156:159]\n" \
  "ds_read_b64_tr_b16 v[66:67], %[vb0] offset:8448\n" \
  "s_cbranch_scc0 .Lslow14_%=\n" \
  ".Lback14_%=:\n" \
  "v_mfma_f32_16x16x32_bf16 v[152:155], v[76:79], a[72:75], v[152:155]\n" \
  "s_add_u32 %[tag], %[tag], 1\n" \
  "v_mfma_f32_16x16x32_bf16 v[148:151], v[76:79], a[68:71], v[148:151]\n" \
  "ds_read_b64_tr_b16 v[68:69], %[vb1] offset:8192\n" \
  "v_mfma_f32_16x16x32_bf16 v[144:147], v[76:79], a[64:67], v[144:147]\n" \
  "s_waitcnt lgkmcnt(9)\n" \
  "v_mfma_f32_16x16x32_bf16 v[160:163], v[80:83], a[64:67], v[160:163]\n" \
  "ds_read_b64_tr_b16 v[70:71], %[vb1] offset:8448\n" \
  "v_mfma_f32_16x16x32_bf16 v[164:167], v[80:83], a[68:71], v[164:167]\n" \
  "buffer_load_dwordx4 a[96:99], %[vx0], %[xr], %[sx] offen sc1\n" \
  "v_mfma_f32_16x16x32_bf16 v[168:171], v[80:83], a[72:75], v[168:171]\n" \
  "ds_read_b64_tr_b16 v[72:73], %[va0] offset:36864\n" \
  "v_mfma_f32_16x16x32_bf16 v[172:175], v[80:83], a[76:79], v[172:175]\n" \
  "s_waitcnt lgkmcnt(9)\n" \
  "v_mfma_f32_16x16x32_bf16 v[188:191], v[84:87], a[76:79], v[188:191]\n" \
  "ds_read_b64_tr_b16 v[74:75], %[va0] offset:37376\n" \
  "v_mfma_f32_16x16x32_bf16 v[184:187], v[84:87], a[72:75], v[184:187]\n" \
  "buffer_load_dwordx4 a[100:103], %[vx1], %[xr], %[sx] offen sc1\n" \
  "v_mfma_f32_16x16x32_bf16 v[180:183], v[84:87], a[68:71], v[180:183]\n" \
  "ds_read_b64_tr_b16 v[76:77], %[va1] offset:36864\n" \
  "v_mfma_f32_16x16x32_bf16 v[176:179], v[84:87], a[64:67], v[176:179]\n" \
  "s_waitcnt lgkmcnt(9)\n" \
  "ds_read_b64_tr_b16 v[78:79], %[va1] offset:37376\n" \
  "v_mfma_f32_16x16x32_bf16 v[192:195], v[88:91], a[64:67], v[192:195]\n" \
  "ds_read_b64_tr_b16 v[80:81], %[va2] offset:36864\n" \
  "v_mfma_f32_16x16x32_bf16 v[196:199], v[88:91], a[68:71], v[196:199]\n" \
  "buffer_load_dwordx4 a[104:107], %[vx2], %[xr], %[sx] offen sc1\n" \
  "v_mfma_f32_16x16x32_bf16 v[200:203], v[88:91], a[72:75], v[200:203]\n" \
  "ds_read_b64_tr_b16 v[82:83], %[va2] offset:37376\n" \
  "v_mfma_f32_16x16x32_bf16 v[204:207], v[88:91], a[76:79], v[204:207]\n" \
  "s_waitcnt lgkmcnt(10)\n" \
  "v_mfma_f32_16x16x32_bf16 v[220:223], v[92:95], a[76:79], v[220:223]\n" \
  "ds_read_b64_tr_b16 v[84:85], %[va3] offset:36864\n" \
  "v_mfma_f32_16x16x32_bf16 v[216:219], v[92:95], a[72:75], v[216:219]\n" \
  "buffer_load_dwordx4 a[108:111], %[vx3], %[xr], %[sx] offen sc1\n" \
  "v_mfma_f32_16x16x32_bf16 v[212:215], v[92:95], a[68:71], v[212:215]\n" \
  "ds_read_b64_tr_b16 v[86:87], %[va3] offset:37376\n" \
  "s_add_u32 %[sx], %[sx], 4096\n" \
  "v_mfma_f32_16x16x32_bf16 v[208:211], v[92:95], a[64:67], v[208:211]\n" \
  "s_waitcnt lgkmcnt(8)\n" \
  "v_mfma_f32_16x16x32_bf16 v[224:227], v[64:67], a[64:67], v[224:227]\n" \
  "ds_read_b64_tr_b16 v[88:89], %[va0] offset:45056\n" \
  "v_mfma_f32_16x16x32_bf16 v[228:231], v[64:67], a[68:71], v[228:231]\n" \
  "v_mfma_f32_16x16x32_bf16 v[232:235], v[64:67], a[72:75], v[232:235]\n" \
  "ds_read_b64_tr_b16 v[90:91], %[va0] offset:45568\n" \
  "v_mfma_f32_16x16x32_bf16 v[236:239], v[64:67], a[76:79], v[236:239]\n" \
  "v_mfma_f32_16x16x32_bf16 v[240:243], v[68:71], a[76:79], v[240:243]\n" \
  "s_waitcnt vmcnt(4) lgkmcnt(8)\n" \
  "s_cmp_ge_u32 s65, %[tag]\n" \
  "v_mfma_f32_16x16x32_bf16 v[96:99], v[72:75], a[80:83], v[96:99]\n" \
  "ds_read_b64_tr_b16 v[92:93], %[va1] offset:45056\n" \
  "s_cbranch_scc0 .Lslow15_%=\n" \
  ".Lback15_%=:\n" \
  "v_mfma_f32_16x16x32_bf16 v[100:103], v[72:75], a[84:87], v[100:103]\n" \
  "v_mfma_f32_16x16x32_bf16 v[104:107], v[72:75], a[88:91], v[104:107]\n" \
  "ds_read_b64_tr_b16 v[94:95], %[va1] offset:45568\n" \
  "v_mfma_f32_16x16x32_bf16 v[108:111], v[72:75], a[92:95], v[108:111]\n" \
  "s_waitcnt lgkmcnt(8)\n" \
  "v_mfma_f32_16x16x32_bf16 v[124:127], v[76:79], a[92:95], v[124:127]\n" \
  "ds_read_b64_tr_b16 v[64:65], %[va2] offset:45056\n" \
  "v_mfma_f32_16x16x32_bf16 v[120:123], v[76:79], a[88:91], v[120:123]\n" \
  "buffer_load_dwordx4 a[112:115], %[vx0], %[xr], %[sx] offen sc1\n" \
  "v_mfma_f32_16x16x32_bf16 v[116:119], v[76:79], a[84:87], v[116:119]\n" \
  "ds_read_b64_tr_b16 v[66:67], %[va2] offset:45568\n" \
  "v_mfma_f32_16x16x32_bf16 v[112:115], v[76:79], a[80:83], v[112:115]\n" \
  "s_waitcnt lgkmcnt(8)\n" \
  "v_mfma_f32_16x16x32_bf16 v[128:131], v[80:83], a[80:83], v[128:131]\n" \
  "ds_read_b64_tr_b16 v[68:69], %[va3] offset:45056\n" \
  "v_mfma_f32_16x16x32_bf16 v[132:135], v[80:83], a[84:87], v[132:135]\n" \
  "buffer_load_dwordx4 a[116:119], %[vx1], %[xr], %[sx] offen sc1\n" \
  "v_mfma_f32_16x16x32_bf16 v[136:139], v[80:83], a[88:91], v[136:139]\n" \
  "ds_read_b64_tr_b16 v[70:71], %[va3] offset:45568\n" \
  "v_mfma_f32_16x16x32_bf16 v[140:143], v[80:83], a[92:95], v[140:143]\n" \
  "s_waitcnt lgkmcnt(8)\n" \
  "ds_read_b64_tr_b16 v[72:73], %[vb0] offset:10240\n" \
  "v_mfma_f32_16x16x32_bf16 v[156:159], v[84:87], a[92:95], v[156:159]\n" \
  "ds_read_b64_tr_b16 v[74:75], %[vb0] offset:10496\n" \
  "v_mfma_f32_16x16x32_bf16 v[152:155], v[84:87], a[88:91], v[152:155]\n" \
  "buffer_load_dwordx4 a[120:123], %[vx2], %[xr], %[sx] offen sc1\n" \
  "v_mfma_f32_16x16x32_bf16 v[148:151], v[84:87], a[84:87], v[148:151]\n" \
  "ds_read_b64_tr_b16 v[76:77], %[vb1] offset:10240\n" \
  "v_mfma_f32_16x16x32_bf16 v[144:147], v[84:87], a[80:83], v[144:147]\n" \
  "s_waitcnt lgkmcnt(9)\n" \
  "v_mfma_f32_16x16x32_bf16 v[160:163], v[88:91], a[80:83], v[160:163]\n" \
  "ds_read_b64_tr_b16 v[78:79], %[vb1] offset:10496\n" \
  "v_mfma_f32_16x16x32_bf16 v[164:167], v[88:91], a[84:87], v[164:167]\n" \
  "s_barrier\n" \
  "v_mfma_f32_16x16x32_bf16 v[168:171], v[88:91], a[88:91], v[168:171]\n" \
  "ds_read_b64_tr_b16 v[80:81], %[va0] offset:49152\n" \
  "v_mfma_f32_16x16x32_bf16 v[172:175], v[88:91], a[92:95], v[172:175]\n" \
  "s_waitcnt lgkmcnt(9)\n" \
  "v_mfma_f32_16x16x32_bf16 v[188:191], v[92:95], a[92:95], v[188:191]\n" \
  "ds_read_b64_tr_b16 v[82:83], %[va0] offset:49664\n" \
  "v_mfma_f32_16x16x32_bf16 v[184:187], v[92:95], a[88:91], v[184:187]\n" \
  "buffer_load_dwordx4 a[124:127], %[vx3], %[xr], %[sx] offen sc1\n" \
  "v_mfma_f32_16x16x32_bf16 v[180:183], v[92:95], a[84:87], v[180:183]\n" \
  "ds_read_b64_tr_b16 v[84:85], %[va1] offset:49152\n" \
  "s_add_u32 %[sx], %[sx], 4096\n" \
  "v_mfma_f32_16x16x32_bf16 v[176:179], v[92:95], a[80:83], v[176:179]\n" \
  "s_waitcnt lgkmcnt(9)\n" \
  "ds_read_b64_tr_b16 v[86:87], %[va1] offset:49664\n" \
  "v_mfma_f32_16x16x32_bf16 v[192:195], v[64:67], a[80:83], v[192:195]\n" \
  "ds_read_b64_tr_b16 v[88:89], %[va2] offset:49152\n" \
  "v_mfma_f32_16x16x32_bf16 v[196:199], v[64:67], a[84:87], v[196:199]\n" \
  "v_mfma_f32_16x16x32_bf16 v[200:203], v[64:67], a[88:91], v[200:203]\n" \
  "ds_read_b64_tr_b16 v[90:91], %[va2] offset:49664\n" \
  "v_mfma_f32_16x16x32_bf16 v[204:207], v[64:67], a[92:95], v[204:207]\n" \
  "s_waitcnt lgkmcnt(10)\n" \
  "v_mfma_f32_16x16x32_bf16 v[220:223], v[68:71], a[92:95], v[220:223]\n" \
  "ds_read_b64_tr_b16 v[92:93], %[va3] offset:49152\n" \
  "v_mfma_f32_16x16x32_bf16 v[216:219], v[68:71], a[88:91], v[216:219]\n" \
  "v_mfma_f32_16x16x32_bf16 v[212:215], v[68:71], a[84:87], v[212:215]\n" \
  "ds_read_b64_tr_b16 v[94:95], %[va3] offset:49664\n" \
  "v_mfma_f32_16x16x32_bf16 v[208:211], v[68:71], a[80:83], v[208:211]\n" \
  "s_waitcnt lgkmcnt(8)\n" \
  "v_mfma_f32_16x16x32_bf16 v[224:227], v[72:75], a[80:83], v[224:227]\n" \
  "ds_read_b64_tr_b16 v[64:65], %[va0] offset:57344\n" \
  "v_mfma_f32_16x16x32_bf16 v[228:231], v[72:75], a[84:87], v[228:231]\n" \
  "v_mfma_f32_16x16x32_bf16 v[232:235], v[72:75], a[88:91], v[232:235]\n" \
  "ds_read_b64_tr_b16 v[66:67], %[va0] offset:57856\n" \
  "v_mfma_f32_16x16x32_bf16 v[236:239], v[72:75], a[92:95], v[236:239]\n" \
  "v_mfma_f32_16x16x32_bf16 v[240:243], v[76:79], a[92:95], v[240:243]\n" \
  "s_waitcnt vmcnt(4) lgkmcnt(8)\n" \
  "v_mfma_f32_16x16x32_bf16 v[96:99], v[80:83], a[96:99], v[96:99]\n" \
  "ds_read_b64_tr_b16 v[68:69], %[va1] offset:57344\n" \
  "v_mfma_f32_16x16x32_bf16 v[100:103], v[80:83], a[100:103], v[100:103]\n" \
  "v_mfma_f32_16x16x32_bf16 v[104:107], v[80:83], a[104:107], v[104:107]\n" \
  "ds_read_b64_tr_b16 v[70:71], %[va1] offset:57856\n" \
  "v_mfma_f32_16x16x32_bf16 v[108:111], v[80:83], a[108:111], v[108:111]\n" \
  "s_waitcnt lgkmcnt(8)\n" \
  "v_mfma_f32_16x16x32_bf16 v[124:127], v[84:87], a[108:111], v[124:127]\n" \
  "ds_read_b64_tr_b16 v[72:73], %[va2] offset:57344\n" \
  "v_mfma_f32_16x16x32_bf16 v[120:123], v[84:87], a[104:107], v[120:123]\n" \
  "v_mfma_f32_16x16x32_bf16 v[116:119], v[84:87], a[100:103], v[116:119]\n" \
  "ds_read_b64_tr_b16 v[74:75], %[va2] offset:57856\n" \
  "v_mfma_f32_16x16x32_bf16 v[112:115], v[84:87], a[96:99], v[112:115]\n" \
  "s_waitcnt lgkmcnt(8)\n" \
  "v_mfma_f32_16x16x32_bf16 v[128:131], v[88:91], a[96:99], v[128:131]\n" \
  "ds_read_b64_tr_b16 v[76:77], %[va3] offset:57344\n" \
  "v_mfma_f32_16x16x32_bf16 v[132:135], v[88:91], a[100:103], v[132:135]\n" \
  "v_mfma_f32_16x16x32_bf16 v[136:139], v[88:91], a[104:107], v[136:139]\n" \
  "ds_read_b64_tr_b16 v[78:79], %[va3] offset:57856\n" \
  "v_mfma_f32_16x16x32_bf16 v[140:143], v[88:91], a[108:111], v[140:143]\n" \
  "s_waitcnt lgkmcnt(8)\n" \
  "ds_read_b64_tr_b16 v[80:81], %[vb0] offset:12288\n" \
  "v_mfma_f32_16x16x32_bf16 v[156:159], v[92:95], a[108:111], v[156:159]\n" \
  "ds_read_b64_tr_b16 v[82:83], %[vb0] offset:12544\n" \
  "v_mfma_f32_16x16x32_bf16 v[152:155], v[92:95], a[104:107], v[152:155]\n" \
  "v_mfma_f32_16x16x32_bf16 v[148:151], v[92:95], a[100:103], v[148:151]\n" \
  "ds_read_b64_tr_b16 v[84:85], %[vb1] offset:12288\n" \
  "v_mfma_f32_16x16x32_bf16 v[144:147], v[92:95], a[96:99], v[144:147]\n" \
  "s_waitcnt lgkmcnt(9)\n" \
  "v_mfma_f32_16x16x32_bf16 v[160:163], v[64:67], a[96:99], v[160:163]\n" \
  "ds_read_b64_tr_b16 v[86:87], %[vb1] offset:12544\n" \
  "v_mfma_f32_16x16x32_bf16 v[164:167], v[64:67], a[100:103], v[164:167]\n" \
  "v_mfma_f32_16x16x32_bf16 v[168:171], v[64:67], a[104:107], v[168:171]\n" \
  "ds_read_b64_tr_b16 v[88:89], %[va0] offset:53248\n" \
  "v_mfma_f32_16x16x32_bf16 v[172:175], v[64:67], a[108:111], v[172:175]\n" \
  "s_waitcnt lgkmcnt(9)\n" \
  "v_mfma_f32_16x16x32_bf16 v[188:191], v[68:71], a[108:111], v[188:191]\n" \
  "ds_read_b64_tr_b16 v[90:91], %[va0] offset:53760\n" \
  "v_mfma_f32_16x16x32_bf16 v[184:187], v[68:71], a[104:107], v[184:187]\n" \
  "v_mfma_f32_16x16x32_bf16 v[180:183], v[68:71], a[100:103], v[180:183]\n" \
  "ds_read_b64_tr_b16 v[92:93], %[va1] offset:53248\n" \
  "v_mfma_f32_16x16x32_bf16 v[176:179], v[68:71], a[96:99], v[176:179]\n" \
  "s_waitcnt lgkmcnt(9)\n" \
  "ds_read_b64_tr_b16 v[94:95], %[va1] offset:53760\n" \
  "v_mfma_f32_16x16x32_bf16 v[192:195], v[72:75], a[96:99], v[192:195]\n" \
  "ds_read_b64_tr_b16 v[64:65], %[va2] offset:53248\n" \
  "v_mfma_f32_16x16x32_bf16 v[196:199], v[72:75], a[100:103], v[196:199]\n" \
  "v_mfma_f32_16x16x32_bf16 v[200:203], v[72:75], a[104:107], v[200:203]\n" \
  "ds_read_b64_tr_b16 v[66:67], %[va2] offset:53760\n" \
  "v_mfma_f32_16x16x32_bf16 v[204:207], v[72:75], a[108:111], v[204:207]\n" \
  "s_waitcnt lgkmcnt(10)\n" \
  "v_mfma_f32_16x16x32_bf16 v[220:223], v[76:79], a[108:111], v[220:223]\n" \
  "ds_read_b64_tr_b16 v[68:69], %[va3] offset:53248\n" \
  "v_mfma_f32_16x16x32_bf16 v[216:219], v[76:79], a[104:107], v[216:219]\n" \
  "v_mfma_f32_16x16x32_bf16 v[212:215], v[76:79], a[100:103], v[212:215]\n" \
  "ds_read_b64_tr_b16 v[70:71], %[va3] offset:53760\n" \
  "v_mfma_f32_16x16x32_bf16 v[208:211], v[76:79], a[96:99], v[208:211]\n" \
  "s_waitcnt lgkmcnt(8)\n" \
  "v_mfma_f32_16x16x32_bf16 v[224:227], v[80:83], a[96:99], v[224:227]\n" \
  "ds_read_b64_tr_b16 v[72:73], %[va0] offset:61440\n" \
  "v_mfma_f32_16x16x32_bf16 v[228:231], v[80:83], a[100:103], v[228:231]\n" \
  "v_mfma_f32_16x16x32_bf16 v[232:235], v[80:83], a[104:107], v[232:235]\n" \
  "ds_read_b64_tr_b16 v[74:75], %[va0] offset:61952\n" \
  "v_mfma_f32_16x16x32_bf16 v[236:239], v[80:83], a[108:111], v[236:239]\n" \
  "v_mfma_f32_16x16x32_bf16 v[240:243], v[84:87], a[108:111], v[240:243]\n" \
  "s_waitcnt vmcnt(0) lgkmcnt(8)\n" \
  "v_mfma_f32_16x16x32_bf16 v[96:99], v[88:91], a[112:115], v[96:99]\n" \
  "ds_read_b64_tr_b16 v[76:77], %[va1] offset:61440\n" \
  "v_mfma_f32_16x16x32_bf16 v[100:103], v[88:91], a[116:119], v[100:103]\n" \
  "v_mfma_f32_16x16x32_bf16 v[104:107], v[88:91], a[120:123], v[104:107]\n" \
  "ds_read_b64_tr_b16 v[78:79], %[va1] offset:61952\n" \
  "v_mfma_f32_16x16x32_bf16 v[108:111], v[88:91], a[124:127], v[108:111]\n" \
  "s_waitcnt lgkmcnt(8)\n" \
  "v_max_f32 v62, |v96|, |v97|\n" \
  "v_max_f32 v63, |v98|, |v99|\n" \
  "v_mfma_f32_16x16x32_bf16 v[124:127], v[92:95], a[124:127], v[124:127]\n" \
  "ds_read_b64_tr_b16 v[80:81], %[va2] offset:61440\n" \
  "v_max3_f32 v62, |v100|, |v101|, v62\n" \
  "v_max3_f32 v63, |v102|, |v103|, v63\n" \
  "v_mfma_f32_16x16x32_bf16 v[120:123], v[92:95], a[120:123], v[120:123]\n" \
  "v_max3_f32 v62, |v104|, |v105|, v62\n" \
  "v_max3_f32 v63, |v106|, |v107|, v63\n" \
  "v_mfma_f32_16x16x32_bf16 v[116:119], v[92:95], a[116:119], v[116:119]\n" \
  "ds_read_b64_tr_b16 v[82:83], %[va2] offset:61952\n" \
  "v_max3_f32 v62, |v108|, |v109|, v62\n" \
  "v_max3_f32 v63, |v110|, |v111|, v63\n" \
  "v_mfma_f32_16x16x32_bf16 v[112:115], v[92:95], a[112:115], v[112:115]\n" \
  "s_waitcnt lgkmcnt(8)\n" \
  "v_max3_f32 v62, |v124|, |v125|, v62\n" \
  "v_max3_f32 v63, |v126|, |v127|, v63\n" \
  "v_mfma_f32_16x16x32_bf16 v[128:131], v[64:67], a[112:115], v[128:131]\n" \
  "ds_read_b64_tr_b16 v[84:85], %[va3] offset:61440\n" \
  "v_max3_f32 v62, |v120|, |v121|, v62\n" \
  "v_max3_f32 v63, |v122|, |v123|, v63\n" \
  "v_mfma_f32_16x16x32_bf16 v[132:135], v[64:67], a[116:119], v[132:135]\n" \
  "v_max3_f32 v62, |v116|, |v117|, v62\n" \
  "v_max3_f32 v63, |v118|, |v119|, v63\n" \
  "v_mfma_f32_16x16x32_bf16 v[136:139], v[64:67], a[120:123], v[136:139]\n" \
  "ds_read_b64_tr_b16 v[86:87], %[va3] offset:61952\n" \
  "v_max3_f32 v62, |v112|, |v113|, v62\n" \
  "v_max3_f32 v63, |v114|, |v115|, v63\n" \
  "v_mfma_f32_16x16x32_bf16 v[140:143], v[64:67], a[124:127], v[140:143]\n" \
  "s_waitcnt lgkmcnt(8)\n" \
  "ds_read_b64_tr_b16 v[88:89], %[vb0] offset:14336\n" \
  "v_max3_f32 v62, |v128|, |v129|, v62\n" \
  "v_max3_f32 v63, |v130|, |v131|, v63\n" \
  "v_mfma_f32_16x16x32_bf16 v[156:159], v[68:71], a[124:127], v[156:159]\n" \
  "ds_read_b64_tr_b16 v[90:91], %[vb0] offset:14592\n" \
  "v_max3_f32 v62, |v132|, |v133|, v62\n" \
  "v_max3_f32 v63, |v134|, |v135|, v63\n" \
  "v_mfma_f32_16x16x32_bf16 v[152:155], v[68:71], a[120:123], v[152:155]\n" \
  "v_max3_f32 v62, |v136|, |v137|, v62\n" \
  "v_max3_f32 v63, |v138|, |v139|, v63\n" \
  "v_mfma_f32_16x16x32_bf16 v[148:151], v[68:71], a[116:119], v[148:151]\n" \
  "ds_read_b64_tr_b16 v[92:93], %[vb1] offset:14336\n" \
  "v_max3_f32 v62, |v140|, |v141|, v62\n" \
  "v_max3_f32 v63, |v142|, |v143|, v63\n" \
  "v_mfma_f32_16x16x32_bf16 v[144:147], v[68:71], a[112:115], v[144:147]\n" \
  "s_waitcnt lgkmcnt(9)\n" \
  "v_max3_f32 v62, |v156|, |v157|, v62\n" \
  "v_max3_f32 v63, |v158|, |v159|, v63\n" \
  "v_mfma_f32_16x16x32_bf16 v[160:163], v[72:75], a[112:115], v[160:163]\n" \
  "ds_read_b64_tr_b16 v[94:95], %[vb1] offset:14592\n" \
  "v_max3_f32 v62, |v152|, |v153|, v62\n" \
  "v_max3_f32 v63, |v154|, |v155|, v63\n" \
  "v_mfma_f32_16x16x32_bf16 v[164:167], v[72:75], a[116:119], v[164:167]\n" \
  "v_max3_f32 v62, |v148|, |v149|, v62\n" \
  "v_max3_f32 v63, |v150|, |v151|, v63\n" \
  "v_mfma_f32_16x16x32_bf16 v[168:171], v[72:75], a[120:123], v[168:171]\n" \
  "v_max3_f32 v62, |v144|, |v145|, v62\n" \
  "v_max3_f32 v63, |v146|, |v147|, v63\n" \
  "v_mfma_f32_16x16x32_bf16 v[172:175], v[72:75], a[124:127], v[172:175]\n" \
  "s_waitcnt lgkmcnt(8)\n" \
  "v_max3_f32 v62, |v160|, |v161|, v62\n" \
  "v_max3_f32 v63, |v162|, |v163|, v63\n" \
  "v_mfma_f32_16x16x32_bf16 v[188:191], v[76:79], a[124:127], v[188:191]\n" \
  "v_max3_f32 v62, |v164|, |v165|, v62\n" \
  "v_max3_f32 v63, |v166|, |v167|, v63\n" \
  "v_mfma_f32_16x16x32_bf16 v[184:187], v[76:79], a[120:123], v[184:187]\n" \
  "v_max3_f32 v62, |v168|, |v169|, v62\n" \
  "v_max3_f32 v63, |v170|, |v171|, v63\n" \
  "v_mfma_f32_16x16x32_bf16 v[180:183], v[76:79], a[116:119], v[180:183]\n" \
  "v_max3_f32 v62, |v172|, |v173|, v62\n" \
  "v_max3_f32 v63, |v174|, |v175|, v63\n" \
  "v_mfma_f32_16x16x32_bf16 v[176:179], v[76:79], a[112:115], v[176:179]\n" \
  "s_waitcnt lgkmcnt(6)\n" \
  "v_max3_f32 v62, |v188|, |v189|, v62\n" \
  "v_max3_f32 v63, |v190|, |v191|, v63\n" \
  "v_mfma_f32_16x16x32_bf16 v[192:195], v[80:83], a[112:115], v[192:195]\n" \
  "v_max3_f32 v62, |v184|, |v185|, v62\n" \
  "v_max3_f32 v63, |v186|, |v187|, v63\n" \
  "v_mfma_f32_16x16x32_bf16 v[196:199], v[80:83], a[116:119], v[196:199]\n" \
  "v_max3_f32 v62, |v180|, |v181|, v62\n" \
  "v_max3_f32 v63, |v182|, |v183|, v63\n" \
  "v_mfma_f32_16x16x32_bf16 v[200:203], v[80:83], a[120:123], v[200:203]\n" \
  "v_max3_f32 v62, |v176|, |v177|, v62\n" \
  "v_max3_f32 v63, |v178|, |v179|, v63\n" \
  "v_mfma_f32_16x16x32_bf16 v[204:207], v[80:83], a[124:127], v[204:207]\n" \
  "s_waitcnt lgkmcnt(4)\n" \
  "v_max3_f32 v62, |v192|, |v193|, v62\n" \
  "v_max3_f32 v63, |v194|, |v195|, v63\n" \
  "v_mfma_f32_16x16x32_bf16 v[220:223], v[84:87], a[124:127], v[220:223]\n" \
  "v_max3_f32 v62, |v196|, |v197|, v62\n" \
  "v_max3_f32 v63, |v198|, |v199|, v63\n" \
  "v_mfma_f32_16x16x32_bf16 v[216:219], v[84:87], a[120:123], v[216:219]\n" \
  "v_max3_f32 v62, |v200|, |v201|, v62\n" \
  "v_max3_f32 v63, |v202|, |v203|, v63\n" \
  "v_mfma_f32_16x16x32_bf16 v[212:215], v[84:87], a[116:119], v[212:215]\n" \
  "v_max3_f32 v62, |v204|, |v205|, v62\n" \
  "v_max3_f32 v63, |v206|, |v207|, v63\n" \
  "v_mfma_f32_16x16x32_bf16 v[208:211], v[84:87], a[112:115], v[208:211]\n" \
  "s_waitcnt lgkmcnt(0)\n" \
  "v_max3_f32 v62, |v220|, |v221|, v62\n" \
  "v_max3_f32 v63, |v222|, |v223|, v63\n" \
  "v_mfma_f32_16x16x32_bf16 v[224:227], v[88:91], a[112:115], v[224:227]\n" \
  "v_max3_f32 v62, |v216|, |v217|, v62\n" \
  "v_max3_f32 v63, |v218|, |v219|, v63\n" \
  "v_mfma_f32_16x16x32_bf16 v[228:231], v[88:91], a[116:119], v[228:231]\n" \
  "v_max3_f32 v62, |v212|, |v213|, v62\n" \
  "v_max3_f32 v63, |v214|, |v215|, v63\n" \
  "v_mfma_f32_16x16x32_bf16 v[232:235], v[88:91], a[120:123], v[232:235]\n" \
  "v_max3_f32 v62, |v208|, |v209|, v62\n" \
  "v_max3_f32 v63, |v210|, |v211|, v63\n" \
  "v_mfma_f32_16x16x32_bf16 v[236:239], v[88:91], a[124:127], v[236:239]\n" \
  "v_max3_f32 v62, |v224|, |v225|, v62\n" \
  "v_max3_f32 v63, |v226|, |v227|, v63\n" \
  "v_mfma_f32_16x16x32_bf16 v[240:243], v[92:95], a[124:127], v[240:243]\n" \
  "s_cmp_eq_u32 %[call], 0\n" \
  "s_cselect_b32 s62, 0x7fffffff, 0\n" \
  "v_max3_f32 v62, |v228|, |v229|, v62\n" \
  "v_max3_f32 v63, |v230|, |v231|, v63\n" \
  "v_max3_f32 v62, |v232|, |v233|, v62\n" \
  "v_max3_f32 v63, |v234|, |v235|, v63\n" \
  "v_max3_f32 v62, |v236|, |v237|, v62\n" \
  "v_max3_f32 v63, |v238|, |v239|, v63\n" \
  "s_nop 1\n" \
  "v_max3_f32 v62, |v240|, |v241|, v62\n" \
  "v_max3_f32 v63, |v242|, |v243|, v63\n" \
  "v_max3_u32 v61, v62, v63, s62\n" \
  "v_mov_b32 v60, %[callc]\n" \
  "s_mov_b32 s62, 0\n" \
  "s_brev_b32 s63, 1\n" \
  "s_nop 1\n" \
  "v_max_u32_dpp v61, v61, v61 row_shr:1 row_mask:0xf bank_mask:0xf bound_ctrl:1\n" \
  "s_nop 1\n" \
  "v_max_u32_dpp v61, v61, v61 row_shr:2 row_mask:0xf bank_mask:0xf bound_ctrl:1\n" \
  "s_nop 1\n" \
  "v_max_u32_dpp v61, v61, v61 row_shr:4 row_mask:0xf bank_mask:0xf bound_ctrl:1\n" \
  "s_nop 1\n" \
  "v_max_u32_dpp v61, v61, v61 row_shr:8 row_mask:0xf bank_mask:0xf bound_ctrl:1\n" \
  "s_nop 1\n" \
  "v_max_u32_dpp v61, v61, v61 row_bcast:15 row_mask:0xa bank_mask:0xf\n" \
  "s_nop 1\n" \
  "v_max_u32_dpp v61, v61, v61 row_bcast:31 row_mask:0xc bank_mask:0xf\n" \
  "s_mov_b64 exec, s[62:63]\n" \
  "global_store_dwordx2 %[vpub], v[60:61], %[gam] sc0\n" \
  "global_store_dwordx2 %[vpub], v[60:61], %[gam] offset:1024 sc1\n" \
  "s_mov_b64 exec, -1\n" \
  "v_permlane32_swap_b32 v96, v128\n" \
  "v_permlane32_swap_b32 v112, v144\n" \
  "v_permlane32_swap_b32 v97, v129\n" \
  "v_permlane32_swap_b32 v113, v145\n" \
  "v_permlane32_swap_b32 v98, v130\n" \
  "v_permlane32_swap_b32 v114, v146\n" \
  "v_permlane32_swap_b32 v99, v131\n" \
  "v_permlane32_swap_b32 v115, v147\n" \
  "v_permlane16_swap_b32 v96, v112\n" \
  "v_permlane16_swap_b32 v128, v144\n" \
  "v_permlane16_swap_b32 v97, v113\n" \
  "v_permlane16_swap_b32 v129, v145\n" \
  "v_permlane16_swap_b32 v98, v114\n" \
  "v_permlane16_swap_b32 v130, v146\n" \
  "v_permlane16_swap_b32 v99, v115\n" \
  "v_permlane16_swap_b32 v131, v147\n" \
  "v_permlane32_swap_b32 v160, v192\n" \
  "v_permlane32_swap_b32 v176, v208\n" \
  "v_permlane32_swap_b32 v161, v193\n" \
  "v_permlane32_swap_b32 v177, v209\n" \
  "v_permlane32_swap_b32 v162, v194\n" \
  "v_permlane32_swap_b32 v178, v210\n" \
  "v_permlane32_swap_b32 v163, v195\n" \
  "v_permlane32_swap_b32 v179, v211\n" \
  "v_permlane16_swap_b32 v160, v176\n" \
  "v_permlane16_swap_b32 v192, v208\n" \
  "v_permlane16_swap_b32 v161, v177\n" \
  "v_permlane16_swap_b32 v193, v209\n" \
  "v_permlane16_swap_b32 v162, v178\n" \
  "v_permlane16_swap_b32 v194, v210\n" \
  "v_permlane16_swap_b32 v163, v179\n" \
  "v_permlane16_swap_b32 v195, v211\n" \
  "v_permlane32_swap_b32 v100, v132\n" \
  "v_permlane32_swap_b32 v116, v148\n" \
  "v_permlane32_swap_b32 v101, v133\n" \
  "v_permlane32_swap_b32 v117, v149\n" \
  "v_permlane32_swap_b32 v102, v134\n" \
  "v_permlane32_swap_b32 v118, v150\n" \
  "v_permlane32_swap_b32 v103, v135\n" \
  "v_permlane32_swap_b32 v119, v151\n" \
  "v_permlane16_swap_b32 v100, v116\n" \
  "v_permlane16_swap_b32 v132, v148\n" \
  "v_permlane16_swap_b32 v101, v117\n" \
  "v_permlane16_swap_b32 v133, v149\n" \
  "v_permlane16_swap_b32 v102, v118\n" \
  "v_permlane16_swap_b32 v134, v150\n" \
  "v_permlane16_swap_b32 v103, v119\n" \
  "v_permlane16_swap_b32 v135, v151\n" \
  "v_permlane32_swap_b32 v164, v196\n" \
  "v_permlane32_swap_b32 v180, v212\n" \
  "v_permlane32_swap_b32 v165, v197\n" \
  "v_permlane32_swap_b32 v181, v213\n" \
  "v_permlane32_swap_b32 v166, v198\n" \
  "v_permlane32_swap_b32 v182, v214\n" \
  "v_permlane32_swap_b32 v167, v199\n" \
  "v_permlane32_swap_b32 v183, v215\n" \
  "v_permlane16_swap_b32 v164, v180\n" \
  "v_permlane16_swap_b32 v196, v212\n" \
  "v_permlane16_swap_b32 v165, v181\n" \
  "v_permlane16_swap_b32 v197, v213\n" \
  "v_permlane16_swap_b32 v166, v182\n" \
  "v_permlane16_swap_b32 v198, v214\n" \
  "v_permlane16_swap_b32 v167, v183\n" \
  "v_permlane16_swap_b32 v199, v215\n" \
  "v_permlane32_swap_b32 v104, v136\n" \
  "v_permlane32_swap_b32 v120, v152\n" \
  "v_permlane32_swap_b32 v105, v137\n" \
  "v_permlane32_swap_b32 v121, v153\n" \
  "v_permlane32_swap_b32 v106, v138\n" \
  "v_permlane32_swap_b32 v122, v154\n" \
  "v_permlane32_swap_b32 v107, v139\n" \
  "v_permlane32_swap_b32 v123, v155\n" \
  "v_permlane16_swap_b32 v104, v120\n" \
  "v_permlane16_swap_b32 v136, v152\n" \
  "v_permlane16_swap_b32 v105, v121\n" \
  "v_permlane16_swap_b32 v137, v153\n" \
  "v_permlane16_swap_b32 v106, v122\n" \
  "v_permlane16_swap_b32 v138, v154\n" \
  "v_permlane16_swap_b32 v107, v123\n" \
  "v_permlane16_swap_b32 v139, v155\n" \
  "v_permlane32_swap_b32 v168, v200\n" \
  "v_permlane32_swap_b32 v184, v216\n" \
  "v_permlane32_swap_b32 v169, v201\n" \
  "v_permlane32_swap_b32 v185, v217\n" \
  "v_permlane32_swap_b32 v170, v202\n" \
  "v_permlane32_swap_b32 v186, v218\n" \
  "v_permlane32_swap_b32 v171, v203\n" \
  "v_permlane32_swap_b32 v187, v219\n" \
  "v_permlane16_swap_b32 v168, v184\n" \
  "v_permlane16_swap_b32 v200, v216\n" \
  "v_permlane16_swap_b32 v169, v185\n" \
  "v_permlane16_swap_b32 v201, v217\n" \
  "v_permlane16_swap_b32 v170, v186\n" \
  "v_permlane16_swap_b32 v202, v218\n" \
  "v_permlane16_swap_b32 v171, v187\n" \
  "v_permlane16_swap_b32 v203, v219\n" \
  "v_permlane32_swap_b32 v108, v140\n" \
  "v_permlane32_swap_b32 v124, v156\n" \
  "v_permlane32_swap_b32 v109, v141\n" \
  "v_permlane32_swap_b32 v125, v157\n" \
  "v_permlane32_swap_b32 v110, v142\n" \
  "v_permlane32_swap_b32 v126, v158\n" \
  "v_permlane32_swap_b32 v111, v143\n" \
  "v_permlane32_swap_b32 v127, v159\n" \
  "v_permlane16_swap_b32 v108, v124\n" \
  "v_permlane16_swap_b32 v140, v156\n" \
  "v_permlane16_swap_b32 v109, v125\n" \
  "v_permlane16_swap_b32 v141, v157\n" \
  "v_permlane16_swap_b32 v110, v126\n" \
  "v_permlane16_swap_b32 v142, v158\n" \
  "v_permlane16_swap_b32 v111, v127\n" \
  "v_permlane16_swap_b32 v143, v159\n" \
  "v_permlane32_swap_b32 v172, v204\n" \
  "v_permlane32_swap_b32 v188, v220\n" \
  "v_permlane32_swap_b32 v173, v205\n" \
  "v_permlane32_swap_b32 v189, v221\n" \
  "v_permlane32_swap_b32 v174, v206\n" \
  "v_permlane32_swap_b32 v190, v222\n" \
  "v_permlane32_swap_b32 v175, v207\n" \
  "v_permlane32_swap_b32 v191, v223\n" \
  "v_permlane16_swap_b32 v172, v188\n" \
  "v_permlane16_swap_b32 v204, v220\n" \
  "v_permlane16_swap_b32 v173, v189\n" \
  "v_permlane16_swap_b32 v205, v221\n" \
  "v_permlane16_swap_b32 v174, v190\n" \
  "v_permlane16_swap_b32 v206, v222\n" \
  "v_permlane16_swap_b32 v175, v191\n" \
  "v_permlane16_swap_b32 v207, v223\n" \
  "v_permlane32_swap_b32 v224, v232\n" \
  "v_permlane32_swap_b32 v228, v236\n" \
  "v_permlane32_swap_b32 v225, v233\n" \
  "v_permlane32_swap_b32 v229, v237\n" \
  "v_permlane32_swap_b32 v226, v234\n" \
  "v_permlane32_swap_b32 v230, v238\n" \
  "v_permlane32_swap_b32 v227, v235\n" \
  "v_permlane32_swap_b32 v231, v239\n" \
  "v_permlane16_swap_b32 v224, v228\n" \
  "v_permlane16_swap_b32 v232, v236\n" \
  "v_permlane16_swap_b32 v225, v229\n" \
  "v_permlane16_swap_b32 v233, v237\n" \
  "v_permlane16_swap_b32 v226, v230\n" \
  "v_permlane16_swap_b32 v234, v238\n" \
  "v_permlane16_swap_b32 v227, v231\n" \
  "v_permlane16_swap_b32 v235, v239\n" \
  ".Lgl_%=:\n" \
  "global_load_dwordx2 v[64:65], %[vgo], %[gam] sc1\n" \
  "s_memrealtime s[60:61]\n" \
  "s_waitcnt vmcnt(0)\n" \
  "v_cmp_gt_u32 vcc, %[callc], v64\n" \
  "s_cbranch_vccz .Lgd_%=\n" \
  "s_waitcnt lgkmcnt(0)\n" \
  "s_sub_u32 s60, s60, %[t0]\n" \
  "s_cmp_gt_u32 s60, %[dl]\n" \
  "s_cbranch_scc0 .Lgl_%=\n" \
  "v_mov_b32 v65, 0x7fffffff\n" \
  ".Lgd_%=:\n" \
  "s_waitcnt lgkmcnt(0)\n" \
  "s_nop 1\n" \
  "v_max_u32_dpp v65, v65, v65 row_shr:1 row_mask:0xf bank_mask:0xf bound_ctrl:1\n" \
  "s_nop 1\n" \
  "v_max_u32_dpp v65, v65, v65 row_shr:2 row_mask:0xf bank_mask:0xf bound_ctrl:1\n" \
  "s_nop 1\n" \
  "v_max_u32_dpp v65, v65, v65 row_shr:4 row_mask:0xf bank_mask:0xf bound_ctrl:1\n" \
  "s_nop 1\n" \
  "v_max_u32_dpp v65, v65, v65 row_shr:8 row_mask:0xf bank_mask:0xf bound_ctrl:1\n" \
  "s_nop 1\n" \
  "v_readlane_b32 s60, v65, 15\n" \
  "v_readlane_b32 s61, v65, 31\n" \
  "v_mov_b32 v60, 0x3b124925\n" \
  "v_readlane_b32 s62, v65, 47\n" \
  "v_readlane_b32 s63, v65, 63\n" \
  "s_max_u32 s60, s60, s62\n" \
  "s_max_u32 s61, s61, s63\n" \
  "s_max_u32 s60, s60, s61\n" \
  "v_mul_f32 v60, s60, v60\n" \
  "v_rcp_f32 v62, v60\n" \
  "s_nop 0\n" \
  "v_fma_f32 v63, -v60, v62, 1.0\n" \
  "v_fma_f32 v62, v63, v62, v62\n" \
  "v_pk_mul_f32 v[240:241], v[240:241], v[62:63] op_sel_hi:[1,0]\n" \
  "v_pk_mul_f32 v[242:243], v[242:243], v[62:63] op_sel_hi:[1,0]\n" \
  "v_pk_mul_f32 v[224:225], v[224:225], v[62:63] op_sel_hi:[1,0]\n" \
  "v_pk_mul_f32 v[228:229], v[228:229], v[62:63] op_sel_hi:[1,0]\n" \
  "v_pk_mul_f32 v[232:233], v[232:233], v[62:63] op_sel_hi:[1,0]\n" \
  "v_pk_mul_f32 v[236:237], v[236:237], v[62:63] op_sel_hi:[1,0]\n" \
  "v_pk_mul_f32 v[226:227], v[226:227], v[62:63] op_sel_hi:[1,0]\n" \
  "v_pk_mul_f32 v[230:231], v[230:231], v[62:63] op_sel_hi:[1,0]\n" \
  "v_pk_mul_f32 v[234:235], v[234:235], v[62:63] op_sel_hi:[1,0]\n" \
  "v_pk_mul_f32 v[238:239], v[238:239], v[62:63] op_sel_hi:[1,0]\n" \
  "v_cvt_pk_fp8_f32 v88, v240, v241\n" \
  "v_cvt_pk_fp8_f32 v92, v224, v225\n" \
  "v_cvt_pk_fp8_f32 v93, v228, v229\n" \
  "v_cvt_pk_fp8_f32 v94, v232, v233\n" \
  "v_cvt_pk_fp8_f32 v95, v236, v237\n" \
  "v_cvt_pk_fp8_f32 v88, v242, v243 op_sel:[0,0,1]\n" \
  "v_cvt_pk_fp8_f32 v92, v226, v227 op_sel:[0,0,1]\n" \
  "v_cvt_pk_fp8_f32 v93, v230, v231 op_sel:[0,0,1]\n" \
  "v_cvt_pk_fp8_f32 v94, v234, v235 op_sel:[0,0,1]\n" \
  "v_cvt_pk_fp8_f32 v95, v238, v239 op_sel:[0,0,1]\n" \
  "v_mov_b32 v89, v88\n" \
  "v_mov_b32 v90, v88\n" \
  "v_mov_b32 v91, v88\n" \
  "global_store_dwordx4 %[voB], v[92:95], %[out] nt\n" \
  "v_pk_mul_f32 v[96:97], v[96:97], v[62:63] op_sel_hi:[1,0]\n" \
  "v_pk_mul_f32 v[112:113], v[112:113], v[62:63] op_sel_hi:[1,0]\n" \
  "v_permlane32_swap_b32 v88, v90\n" \
  "v_permlane32_swap_b32 v89, v91\n" \
  "v_pk_mul_f32 v[128:129], v[128:129], v[62:63] op_sel_hi:[1,0]\n" \
  "v_pk_mul_f32 v[144:145], v[144:145], v[62:63] op_sel_hi:[1,0]\n" \
  "v_permlane16_swap_b32 v88, v89\n" \
  "v_permlane16_swap_b32 v90, v91\n" \
  "v_pk_mul_f32 v[98:99], v[98:99], v[62:63] op_sel_hi:[1,0]\n" \
  "v_pk_mul_f32 v[114:115], v[114:115], v[62:63] op_sel_hi:[1,0]\n" \
  "v_pk_mul_f32 v[130:131], v[130:131], v[62:63] op_sel_hi:[1,0]\n" \
  "v_pk_mul_f32 v[146:147], v[146:147], v[62:63] op_sel_hi:[1,0]\n" \
  "s_cmp_lt_u32 %[ct], 8\n" \
  "s_cselect_b64 exec, 0xffff, 0\n" \
  "global_store_dwordx4 %[voX], v[88:91], %[out] nt\n" \
  "s_mov_b64 exec, -1\n" \
  "v_cvt_pk_fp8_f32 v64, v96, v97\n" \
  "v_cvt_pk_fp8_f32 v65, v112, v113\n" \
  "v_cvt_pk_fp8_f32 v66, v128, v129\n" \
  "v_cvt_pk_fp8_f32 v67, v144, v145\n" \
  "v_cvt_pk_fp8_f32 v64, v98, v99 op_sel:[0,0,1]\n" \
  "v_cvt_pk_fp8_f32 v65, v114, v115 op_sel:[0,0,1]\n" \
  "v_cvt_pk_fp8_f32 v66, v130, v131 op_sel:[0,0,1]\n" \
  "v_cvt_pk_fp8_f32 v67, v146, v147 op_sel:[0,0,1]\n" \
  "v_pk_mul_f32 v[160:161], v[160:161], v[62:63] op_sel_hi:[1,0]\n" \
  "v_pk_mul_f32 v[176:177], v[176:177], v[62:63] op_sel_hi:[1,0]\n" \
  "ds_write_b128 %[vw], v[64:67] offset:0\n" \
  "v_pk_mul_f32 v[192:193], v[192:193], v[62:63] op_sel_hi:[1,0]\n" \
  "v_pk_mul_f32 v[208:209], v[208:209], v[62:63] op_sel_hi:[1,0]\n" \
  "v_pk_mul_f32 v[162:163], v[162:163], v[62:63] op_sel_hi:[1,0]\n" \
  "v_pk_mul_f32 v[178:179], v[178:179], v[62:63] op_sel_hi:[1,0]\n" \
  "v_pk_mul_f32 v[194:195], v[194:195], v[62:63] op_sel_hi:[1,0]\n" \
  "v_pk_mul_f32 v[210:211], v[210:211], v[62:63] op_sel_hi:[1,0]\n" \
  "v_cvt_pk_fp8_f32 v68, v160, v161\n" \
  "v_cvt_pk_fp8_f32 v69, v176, v177\n" \
  "v_cvt_pk_fp8_f32 v70, v192, v193\n" \
  "v_cvt_pk_fp8_f32 v71, v208, v209\n" \
  "v_cvt_pk_fp8_f32 v68, v162, v163 op_sel:[0,0,1]\n" \
  "v_cvt_pk_fp8_f32 v69, v178, v179 op_sel:[0,0,1]\n" \
  "v_cvt_pk_fp8_f32 v70, v194, v195 op_sel:[0,0,1]\n" \
  "v_cvt_pk_fp8_f32 v71, v210, v211 op_sel:[0,0,1]\n" \
  "v_pk_mul_f32 v[100:101], v[100:101], v[62:63] op_sel_hi:[1,0]\n" \
  "v_pk_mul_f32 v[116:117], v[116:117], v[62:63] op_sel_hi:[1,0]\n" \
  "ds_write_b128 %[vw], v[68:71] offset:64\n" \
  "ds_read_b128 v[112:115], %[vr] offset:0\n" \
  "ds_read_b128 v[128:131], %[vr] offset:1152\n" \
  "v_pk_mul_f32 v[132:133], v[132:133], v[62:63] op_sel_hi:[1,0]\n" \
  "v_pk_mul_f32 v[148:149], v[148:149], v[62:63] op_sel_hi:[1,0]\n" \
  "v_pk_mul_f32 v[102:103], v[102:103], v[62:63] op_sel_hi:[1,0]\n" \
  "v_pk_mul_f32 v[118:119], v[118:119], v[62:63] op_sel_hi:[1,0]\n" \
  "v_pk_mul_f32 v[134:135], v[134:135], v[62:63] op_sel_hi:[1,0]\n" \
  "v_pk_mul_f32 v[150:151], v[150:151], v[62:63] op_sel_hi:[1,0]\n" \
  "v_cvt_pk_fp8_f32 v72, v100, v101\n" \
  "v_cvt_pk_fp8_f32 v73, v116, v117\n" \
  "v_cvt_pk_fp8_f32 v74, v132, v133\n" \
  "v_cvt_pk_fp8_f32 v75, v148, v149\n" \
  "v_cvt_pk_fp8_f32 v72, v102, v103 op_sel:[0,0,1]\n" \
  "v_cvt_pk_fp8_f32 v73, v118, v119 op_sel:[0,0,1]\n" \
  "v_cvt_pk_fp8_f32 v74, v134, v135 op_sel:[0,0,1]\n" \
  "v_cvt_pk_fp8_f32 v75, v150, v151 op_sel:[0,0,1]\n" \
  "v_pk_mul_f32 v[164:165], v[164:165], v[62:63] op_sel_hi:[1,0]\n" \
  "v_pk_mul_f32 v[180:181], v[180:181], v[62:63] op_sel_hi:[1,0]\n" \
  "ds_write_b128 %[vw], v[72:75] offset:2304\n" \
  "s_waitcnt lgkmcnt(1)\n" \
  "global_store_dwordx4 %[vo0], v[112:115], %[out] nt\n" \
  "global_store_dwordx4 %[vo0], v[128:131], %[out2] nt\n" \
  "v_pk_mul_f32 v[196:197], v[196:197], v[62:63] op_sel_hi:[1,0]\n" \
  "v_pk_mul_f32 v[212:213], v[212:213], v[62:63] op_sel_hi:[1,0]\n" \
  "v_pk_mul_f32 v[166:167], v[166:167], v[62:63] op_sel_hi:[1,0]\n" \
  "v_pk_mul_f32 v[182:183], v[182:183], v[62:63] op_sel_hi:[1,0]\n" \
  "v_pk_mul_f32 v[198:199], v[198:199], v[62:63] op_sel_hi:[1,0]\n" \
  "v_pk_mul_f32 v[214:215], v[214:215], v[62:63] op_sel_hi:[1,0]\n" \
  "v_cvt_pk_fp8_f32 v76, v164, v165\n" \
  "v_cvt_pk_fp8_f32 v77, v180, v181\n" \
  "v_cvt_pk_fp8_f32 v78, v196, v197\n" \
  "v_cvt_pk_fp8_f32 v79, v212, v213\n" \
  "v_cvt_pk_fp8_f32 v76, v166, v167 op_sel:[0,0,1]\n" \
  "v_cvt_pk_fp8_f32 v77, v182, v183 op_sel:[0,0,1]\n" \
  "v_cvt_pk_fp8_f32 v78, v198, v199 op_sel:[0,0,1]\n" \
  "v_cvt_pk_fp8_f32 v79, v214, v215 op_sel:[0,0,1]\n" \
  "v_pk_mul_f32 v[104:105], v[104:105], v[62:63] op_sel_hi:[1,0]\n" \
  "v_pk_mul_f32 v[120:121], v[120:121], v[62:63] op_sel_hi:[1,0]\n" \
  "ds_write_b128 %[vw], v[76:79] offset:2368\n" \
  "ds_read_b128 v[116:119], %[vr] offset:2304\n" \
  "ds_read_b128 v[132:135], %[vr] offset:3456\n" \
  "v_pk_mul_f32 v[136:137], v[136:137], v[62:63] op_sel_hi:[1,0]\n" \
  "v_pk_mul_f32 v[152:153], v[152:153], v[62:63] op_sel_hi:[1,0]\n" \
  "v_pk_mul_f32 v[106:107], v[106:107], v[62:63] op_sel_hi:[1,0]\n" \
  "v_pk_mul_f32 v[122:123], v[122:123], v[62:63] op_sel_hi:[1,0]\n" \
  "v_pk_mul_f32 v[138:139], v[138:139], v[62:63] op_sel_hi:[1,0]\n" \
  "v_pk_mul_f32 v[154:155], v[154:155], v[62:63] op_sel_hi:[1,0]\n" \
  "v_cvt_pk_fp8_f32 v80, v104, v105\n" \
  "v_cvt_pk_fp8_f32 v81, v120, v121\n" \
  "v_cvt_pk_fp8_f32 v82, v136, v137\n" \
  "v_cvt_pk_fp8_f32 v83, v152, v153\n" \
  "v_cvt_pk_fp8_f32 v80, v106, v107 op_sel:[0,0,1]\n" \
  "v_cvt_pk_fp8_f32 v81, v122, v123 op_sel:[0,0,1]\n" \
  "v_cvt_pk_fp8_f32 v82, v138, v139 op_sel:[0,0,1]\n" \
  "v_cvt_pk_fp8_f32 v83, v154, v155 op_sel:[0,0,1]\n" \
  "v_pk_mul_f32 v[168:169], v[168:169], v[62:63] op_sel_hi:[1,0]\n" \
  "v_pk_mul_f32 v[184:185], v[184:185], v[62:63] op_sel_hi:[1,0]\n" \
  "ds_write_b128 %[vw], v[80:83] offset:4608\n" \
  "s_waitcnt lgkmcnt(1)\n" \
  "global_store_dwordx4 %[vo1], v[116:119], %[out] nt\n" \
  "global_store_dwordx4 %[vo1], v[132:135], %[out2] nt\n" \
  "v_pk_mul_f32 v[200:201], v[200:201], v[62:63] op_sel_hi:[1,0]\n" \
  "v_pk_mul_f32 v[216:217], v[216:217], v[62:63] op_sel_hi:[1,0]\n" \
  "v_pk_mul_f32 v[170:171], v[170:171], v[62:63] op_sel_hi:[1,0]\n" \
  "v_pk_mul_f32 v[186:187], v[186:187], v[62:63] op_sel_hi:[1,0]\n" \
  "v_pk_mul_f32 v[202:203], v[202:203], v[62:63] op_sel_hi:[1,0]\n" \
  "v_pk_mul_f32 v[218:219], v[218:219], v[62:63] op_sel_hi:[1,0]\n" \
  "v_cvt_pk_fp8_f32 v84, v168, v169\n" \
  "v_cvt_pk_fp8_f32 v85, v184, v185\n" \
  "v_cvt_pk_fp8_f32 v86, v200, v201\n" \
  "v_cvt_pk_fp8_f32 v87, v216, v217\n" \
  "v_cvt_pk_fp8_f32 v84, v170, v171 op_sel:[0,0,1]\n" \
  "v_cvt_pk_fp8_f32 v85, v186, v187 op_sel:[0,0,1]\n" \
  "v_cvt_pk_fp8_f32 v86, v202, v203 op_sel:[0,0,1]\n" \
  "v_cvt_pk_fp8_f32 v87, v218, v219 op_sel:[0,0,1]\n" \
  "v_pk_mul_f32 v[108:109], v[108:109], v[62:63] op_sel_hi:[1,0]\n" \
  "v_pk_mul_f32 v[124:125], v[124:125], v[62:63] op_sel_hi:[1,0]\n" \
  "ds_write_b128 %[vw], v[84:87] offset:4672\n" \
  "ds_read_b128 v[120:123], %[vr] offset:4608\n" \
  "ds_read_b128 v[136:139], %[vr] offset:5760\n" \
  "v_pk_mul_f32 v[140:141], v[140:141], v[62:63] op_sel_hi:[1,0]\n" \
  "v_pk_mul_f32 v[156:157], v[156:157], v[62:63] op_sel_hi:[1,0]\n" \
  "v_pk_mul_f32 v[110:111], v[110:111], v[62:63] op_sel_hi:[1,0]\n" \
  "v_pk_mul_f32 v[126:127], v[126:127], v[62:63] op_sel_hi:[1,0]\n" \
  "v_pk_mul_f32 v[142:143], v[142:143], v[62:63] op_sel_hi:[1,0]\n" \
  "v_pk_mul_f32 v[158:159], v[158:159], v[62:63] op_sel_hi:[1,0]\n" \
  "v_cvt_pk_fp8_f32 v88, v108, v109\n" \
  "v_cvt_pk_fp8_f32 v89, v124, v125\n" \
  "v_cvt_pk_fp8_f32 v90, v140, v141\n" \
  "v_cvt_pk_fp8_f32 v91, v156, v157\n" \
  "v_cvt_pk_fp8_f32 v88, v110, v111 op_sel:[0,0,1]\n" \
  "v_cvt_pk_fp8_f32 v89, v126, v127 op_sel:[0,0,1]\n" \
  "v_cvt_pk_fp8_f32 v90, v142, v143 op_sel:[0,0,1]\n" \
  "v_cvt_pk_fp8_f32 v91, v158, v159 op_sel:[0,0,1]\n" \
  "v_pk_mul_f32 v[172:173], v[172:173], v[62:63] op_sel_hi:[1,0]\n" \
  "v_pk_mul_f32 v[188:189], v[188:189], v[62:63] op_sel_hi:[1,0]\n" \
  "ds_write_b128 %[vw], v[88:91] offset:6912\n" \
  "s_waitcnt lgkmcnt(1)\n" \
  "global_store_dwordx4 %[vo2], v[120:123], %[out] nt\n" \
  "global_store_dwordx4 %[vo2], v[136:139], %[out2] nt\n" \
  "v_pk_mul_f32 v[204:205], v[204:205], v[62:63] op_sel_hi:[1,0]\n" \
  "v_pk_mul_f32 v[220:221], v[220:221], v[62:63] op_sel_hi:[1,0]\n" \
  "v_pk_mul_f32 v[174:175], v[174:175], v[62:63] op_sel_hi:[1,0]\n" \
  "v_pk_mul_f32 v[190:191], v[190:191], v[62:63] op_sel_hi:[1,0]\n" \
  "v_pk_mul_f32 v[206:207], v[206:207], v[62:63] op_sel_hi:[1,0]\n" \
  "v_pk_mul_f32 v[222:223], v[222:223], v[62:63] op_sel_hi:[1,0]\n" \
  "v_cvt_pk_fp8_f32 v92, v172, v173\n" \
  "v_cvt_pk_fp8_f32 v93, v188, v189\n" \
  "v_cvt_pk_fp8_f32 v94, v204, v205\n" \
  "v_cvt_pk_fp8_f32 v95, v220, v221\n" \
  "v_cvt_pk_fp8_f32 v92, v174, v175 op_sel:[0,0,1]\n" \
  "v_cvt_pk_fp8_f32 v93, v190, v191 op_sel:[0,0,1]\n" \
  "v_cvt_pk_fp8_f32 v94, v206, v207 op_sel:[0,0,1]\n" \
  "v_cvt_pk_fp8_f32 v95, v222, v223 op_sel:[0,0,1]\n" \
  "s_nop 1\n" \
  "ds_write_b128 %[vw], v[92:95] offset:6976\n" \
  "ds_read_b128 v[124:127], %[vr] offset:6912\n" \
  "ds_read_b128 v[140:143], %[vr] offset:8064\n" \
  "s_waitcnt lgkmcnt(0)\n" \
  "global_store_dwordx4 %[vo3], v[124:127], %[out] nt\n" \
  "global_store_dwordx4 %[vo3], v[140:143], %[out2] nt\n" \
  "s_branch .Lend_%=\n" \
  ".Lslow0_%=:\n" \
  "s_load_dword s64, %[fp], %[sl] offset:0x400 glc\n" \
  "s_memrealtime s[60:61]\n" \
  "s_waitcnt lgkmcnt(0)\n" \
  "s_cmp_ge_u32 s64, %[tag]\n" \
  "s_cbranch_scc1 .Lback0_%=\n" \
  "s_sub_u32 s60, s60, %[t0]\n" \
  "s_cmp_gt_u32 s60, %[dl]\n" \
  "s_cbranch_scc0 .Lslow0_%=\n" \
  "s_mov_b32 %[call], 0\n" \
  "s_branch .Lback0_%=\n" \
  ".Lslow1_%=:\n" \
  "s_load_dword s65, %[fp], %[sl] offset:0xc00 glc\n" \
  "s_memrealtime s[60:61]\n" \
  "s_waitcnt lgkmcnt(0)\n" \
  "s_cmp_ge_u32 s65, %[tag]\n" \
  "s_cbranch_scc1 .Lback1_%=\n" \
  "s_sub_u32 s60, s60, %[t0]\n" \
  "s_cmp_gt_u32 s60, %[dl]\n" \
  "s_cbranch_scc0 .Lslow1_%=\n" \
  "s_mov_b32 %[call], 0\n" \
  "s_branch .Lback1_%=\n" \
  ".Lslow2_%=:\n" \
  "s_load_dword s64, %[fp], %[sl] offset:0x4 glc\n" \
  "s_memrealtime s[60:61]\n" \
  "s_waitcnt lgkmcnt(0)\n" \
  "s_cmp_ge_u32 s64, %[tag]\n" \
  "s_cbranch_scc1 .Lback2_%=\n" \
  "s_sub_u32 s60, s60, %[t0]\n" \
  "s_cmp_gt_u32 s60, %[dl]\n" \
  "s_cbranch_scc0 .Lslow2_%=\n" \
  "s_mov_b32 %[call], 0\n" \
  "s_branch .Lback2_%=\n" \
  ".Lslow3_%=:\n" \
  "s_load_dword s65, %[fp], %[sl] offset:0x804 glc\n" \
  "s_memrealtime s[60:61]\n" \
  "s_waitcnt lgkmcnt(0)\n" \
  "s_cmp_ge_u32 s65, %[tag]\n" \
  "s_cbranch_scc1 .Lback3_%=\n" \
  "s_sub_u32 s60, s60, %[t0]\n" \
  "s_cmp_gt_u32 s60, %[dl]\n" \
  "s_cbranch_scc0 .Lslow3_%=\n" \
  "s_mov_b32 %[call], 0\n" \
  "s_branch .Lback3_%=\n" \
  ".Lslow4_%=:\n" \
  "s_load_dword s64, %[fp], %[sl] offset:0x8 glc\n" \
  "s_memrealtime s[60:61]\n" \
  "s_waitcnt lgkmcnt(0)\n" \
  "s_cmp_ge_u32 s64, %[tag]\n" \
  "s_cbranch_scc1 .Lback4_%=\n" \
  "s_sub_u32 s60, s60, %[t0]\n" \
  "s_cmp_gt_u32 s60, %[dl]\n" \
  "s_cbranch_scc0 .Lslow4_%=\n" \
  "s_mov_b32 %[call], 0\n" \
  "s_branch .Lback4_%=\n" \
  ".Lslow5_%=:\n" \
  "s_load_dword s65, %[fp], %[sl] offset:0x808 glc\n" \
  "s_memrealtime s[60:61]\n" \
  "s_waitcnt lgkmcnt(0)\n" \
  "s_cmp_ge_u32 s65, %[tag]\n" \
  "s_cbranch_scc1 .Lback5_%=\n" \
  "s_sub_u32 s60, s60, %[t0]\n" \
  "s_cmp_gt_u32 s60, %[dl]\n" \
  "s_cbranch_scc0 .Lslow5_%=\n" \
  "s_mov_b32 %[call], 0\n" \
  "s_branch .Lback5_%=\n" \
  ".Lslow6_%=:\n" \
  "s_load_dword s64, %[fp], %[sl] offset:0xc glc\n" \
  "s_memrealtime s[60:61]\n" \
  "s_waitcnt lgkmcnt(0)\n" \
  "s_cmp_ge_u32 s64, %[tag]\n" \
  "s_cbranch_scc1 .Lback6_%=\n" \
  "s_sub_u32 s60, s60, %[t0]\n" \
  "s_cmp_gt_u32 s60, %[dl]\n" \
  "s_cbranch_scc0 .Lslow6_%=\n" \
  "s_mov_b32 %[call], 0\n" \
  "s_branch .Lback6_%=\n" \
  ".Lslow7_%=:\n" \
  "s_load_dword s65, %[fp], %[sl] offset:0x80c glc\n" \
  "s_memrealtime s[60:61]\n" \
  "s_waitcnt lgkmcnt(0)\n" \
  "s_cmp_ge_u32 s65, %[tag]\n" \
  "s_cbranch_scc1 .Lback7_%=\n" \
  "s_sub_u32 s60, s60, %[t0]\n" \
  "s_cmp_gt_u32 s60, %[dl]\n" \
  "s_cbranch_scc0 .Lslow7_%=\n" \
  "s_mov_b32 %[call], 0\n" \
  "s_branch .Lback7_%=\n" \
  ".Lslow8_%=:\n" \
  "s_load_dword s64, %[fp], %[sl] offset:0x10 glc\n" \
  "s_memrealtime s[60:61]\n" \
  "s_waitcnt lgkmcnt(0)\n" \
  "s_cmp_ge_u32 s64, %[tag]\n" \
  "s_cbranch_scc1 .Lback8_%=\n" \
  "s_sub_u32 s60, s60, %[t0]\n" \
  "s_cmp_gt_u32 s60, %[dl]\n" \
  "s_cbranch_scc0 .Lslow8_%=\n" \
  "s_mov_b32 %[call], 0\n" \
  "s_branch .Lback8_%=\n" \
  ".Lslow9_%=:\n" \
  "s_load_dword s65, %[fp], %[sl] offset:0x810 glc\n" \
  "s_memrealtime s[60:61]\n" \
  "s_waitcnt lgkmcnt(0)\n" \
  "s_cmp_ge_u32 s65, %[tag]\n" \
  "s_cbranch_scc1 .Lback9_%=\n" \
  "s_sub_u32 s60, s60, %[t0]\n" \
  "s_cmp_gt_u32 s60, %[dl]\n" \
  "s_cbranch_scc0 .Lslow9_%=\n" \
  "s_mov_b32 %[call], 0\n" \
  "s_branch .Lback9_%=\n" \
  ".LslowB2_%=:\n" \
  "s_load_dword s64, %[fp], %[sl] offset:0x4 glc\n" \
  "s_memrealtime s[60:61]\n" \
  "s_waitcnt lgkmcnt(0)\n" \
  "s_cmp_ge_u32 s64, %[tag]\n" \
  "s_cbranch_scc1 .LbackB2_%=\n" \
  "s_sub_u32 s60, s60, %[t0]\n" \
  "s_cmp_gt_u32 s60, %[dl]\n" \
  "s_cbranch_scc0 .LslowB2_%=\n" \
  "s_mov_b32 %[call], 0\n" \
  "s_branch .LbackB2_%=\n" \
  ".LslowB3_%=:\n" \
  "s_load_dword s65, %[fp], %[sl] offset:0x804 glc\n" \
  "s_memrealtime s[60:61]\n" \
  "s_waitcnt lgkmcnt(0)\n" \
  "s_cmp_ge_u32 s65, %[tag]\n" \
  "s_cbranch_scc1 .LbackB3_%=\n" \
  "s_sub_u32 s60, s60, %[t0]\n" \
  "s_cmp_gt_u32 s60, %[dl]\n" \
  "s_cbranch_scc0 .LslowB3_%=\n" \
  "s_mov_b32 %[call], 0\n" \
  "s_branch .LbackB3_%=\n" \
  ".LslowB4_%=:\n" \
  "s_load_dword s64, %[fp], %[sl] offset:0x8 glc\n" \
  "s_memrealtime s[60:61]\n" \
  "s_waitcnt lgkmcnt(0)\n" \
  "s_cmp_ge_u32 s64, %[tag]\n" \
  "s_cbranch_scc1 .LbackB4_%=\n" \
  "s_sub_u32 s60, s60, %[t0]\n" \
  "s_cmp_gt_u32 s60, %[dl]\n" \
  "s_cbranch_scc0 .LslowB4_%=\n" \
  "s_mov_b32 %[call], 0\n" \
  "s_branch .LbackB4_%=\n" \
  ".LslowB5_%=:\n" \
  "s_load_dword s65, %[fp], %[sl] offset:0x808 glc\n" \
  "s_memrealtime s[60:61]\n" \
  "s_waitcnt lgkmcnt(0)\n" \
  "s_cmp_ge_u32 s65, %[tag]\n" \
  "s_cbranch_scc1 .LbackB5_%=\n" \
  "s_sub_u32 s60, s60, %[t0]\n" \
  "s_cmp_gt_u32 s60, %[dl]\n" \
  "s_cbranch_scc0 .LslowB5_%=\n" \
  "s_mov_b32 %[call], 0\n" \
  "s_branch .LbackB5_%=\n" \
  ".LslowB6_%=:\n" \
  "s_load_dword s64, %[fp], %[sl] offset:0xc glc\n" \
  "s_memrealtime s[60:61]\n" \
  "s_waitcnt lgkmcnt(0)\n" \
  "s_cmp_ge_u32 s64, %[tag]\n" \
  "s_cbranch_scc1 .LbackB6_%=\n" \
  "s_sub_u32 s60, s60, %[t0]\n" \
  "s_cmp_gt_u32 s60, %[dl]\n" \
  "s_cbranch_scc0 .LslowB6_%=\n" \
  "s_mov_b32 %[call], 0\n" \
  "s_branch .LbackB6_%=\n" \
  ".LslowB7_%=:\n" \
  "s_load_dword s65, %[fp], %[sl] offset:0x80c glc\n" \
  "s_memrealtime s[60:61]\n" \
  "s_waitcnt lgkmcnt(0)\n" \
  "s_cmp_ge_u32 s65, %[tag]\n" \
  "s_cbranch_scc1 .LbackB7_%=\n" \
  "s_sub_u32 s60, s60, %[t0]\n" \
  "s_cmp_gt_u32 s60, %[dl]\n" \
  "s_cbranch_scc0 .LslowB7_%=\n" \
  "s_mov_b32 %[call], 0\n" \
  "s_branch .LbackB7_%=\n" \
  ".LslowB8_%=:\n" \
  "s_load_dword s64, %[fp], %[sl] offset:0x10 glc\n" \
  "s_memrealtime s[60:61]\n" \
  "s_waitcnt lgkmcnt(0)\n" \
  "s_cmp_ge_u32 s64, %[tag]\n" \
  "s_cbranch_scc1 .LbackB8_%=\n" \
  "s_sub_u32 s60, s60, %[t0]\n" \
  "s_cmp_gt_u32 s60, %[dl]\n" \
  "s_cbranch_scc0 .LslowB8_%=\n" \
  "s_mov_b32 %[call], 0\n" \
  "s_branch .LbackB8_%=\n" \
  ".LslowB9_%=:\n" \
  "s_load_dword s65, %[fp], %[sl] offset:0x810 glc\n" \
  "s_memrealtime s[60:61]\n" \
  "s_waitcnt lgkmcnt(0)\n" \
  "s_cmp_ge_u32 s65, %[tag]\n" \
  "s_cbranch_scc1 .LbackB9_%=\n" \
  "s_sub_u32 s60, s60, %[t0]\n" \
  "s_cmp_gt_u32 s60, %[dl]\n" \
  "s_cbranch_scc0 .LslowB9_%=\n" \
  "s_mov_b32 %[call], 0\n" \
  "s_branch .LbackB9_%=\n" \
  ".p2align 6\n" \
  ".LloopB_%=:\n" \
  "s_waitcnt vmcnt(6) lgkmcnt(8)\n" \
  "v_mfma_f32_16x16x32_bf16 v[96:99], v[64:67], a[0:3], v[96:99]\n" \
  "ds_read_b64_tr_b16 v[84:85], %[va1] offset:8192\n" \
  "v_mfma_f32_16x16x32_bf16 v[100:103], v[64:67], a[4:7], v[100:103]\n" \
  "buffer_load_dwordx4 %[vd2], %[wr], %[sw] offen offset:2048 lds\n" \
  "v_mfma_f32_16x16x32_bf16 v[104:107], v[64:67], a[8:11], v[104:107]\n" \
  "ds_read_b64_tr_b16 v[86:87], %[va1] offset:8704\n" \
  "v_mfma_f32_16x16x32_bf16 v[108:111], v[64:67], a[12:15], v[108:111]\n" \
  "s_waitcnt lgkmcnt(8)\n" \
  "v_mfma_f32_16x16x32_bf16 v[124:127], v[68:71], a[12:15], v[124:127]\n" \
  "ds_read_b64_tr_b16 v[88:89], %[va2] offset:8192\n" \
  "v_mfma_f32_16x16x32_bf16 v[120:123], v[68:71], a[8:11], v[120:123]\n" \
  "buffer_load_dwordx4 %[vd3], %[wr], %[sw] offen offset:3072 lds\n" \
  "v_mfma_f32_16x16x32_bf16 v[116:119], v[68:71], a[4:7], v[116:119]\n" \
  "ds_read_b64_tr_b16 v[90:91], %[va2] offset:8704\n" \
  "s_add_u32 m0, %[mb], 8192\n" \
  "v_mfma_f32_16x16x32_bf16 v[112:115], v[68:71], a[0:3], v[112:115]\n" \
  "s_waitcnt lgkmcnt(8)\n" \
  "v_mfma_f32_16x16x32_bf16 v[128:131], v[72:75], a[0:3], v[128:131]\n" \
  "ds_read_b64_tr_b16 v[92:93], %[va3] offset:8192\n" \
  "v_mfma_f32_16x16x32_bf16 v[132:135], v[72:75], a[4:7], v[132:135]\n" \
  "buffer_load_dwordx4 %[vd4], %[wr], %[sw] offen lds\n" \
  "v_mfma_f32_16x16x32_bf16 v[136:139], v[72:75], a[8:11], v[136:139]\n" \
  "ds_read_b64_tr_b16 v[94:95], %[va3] offset:8704\n" \
  "s_cmp_ge_u32 s64, %[tag]\n" \
  "v_mfma_f32_16x16x32_bf16 v[140:143], v[72:75], a[12:15], v[140:143]\n" \
  "s_waitcnt lgkmcnt(8)\n" \
  "ds_read_b64_tr_b16 v[64:65], %[vb0] offset:0\n" \
  "v_mfma_f32_16x16x32_bf16 v[156:159], v[76:79], a[12:15], v[156:159]\n" \
  "ds_read_b64_tr_b16 v[66:67], %[vb0] offset:256\n" \
  "s_cbranch_scc0 .LslowB2_%=\n" \
  ".LbackB2_%=:\n" \
  "v_mfma_f32_16x16x32_bf16 v[152:155], v[76:79], a[8:11], v[152:155]\n" \
  "s_add_u32 %[tag], %[tag], 1\n" \
  "s_add_u32 %[sw], %[sw], 262144\n" \
  "v_mfma_f32_16x16x32_bf16 v[148:151], v[76:79], a[4:7], v[148:151]\n" \
  "ds_read_b64_tr_b16 v[68:69], %[vb1] offset:0\n" \
  "v_mfma_f32_16x16x32_bf16 v[144:147], v[76:79], a[0:3], v[144:147]\n" \
  "s_waitcnt lgkmcnt(9)\n" \
  "v_mfma_f32_16x16x32_bf16 v[160:163], v[80:83], a[0:3], v[160:163]\n" \
  "ds_read_b64_tr_b16 v[70:71], %[vb1] offset:256\n" \
  "v_mfma_f32_16x16x32_bf16 v[164:167], v[80:83], a[4:7], v[164:167]\n" \
  "buffer_load_dwordx4 a[32:35], %[vx0], %[xr], %[sx] offen sc1\n" \
  "v_mfma_f32_16x16x32_bf16 v[168:171], v[80:83], a[8:11], v[168:171]\n" \
  "ds_read_b64_tr_b16 v[72:73], %[va0] offset:4096\n" \
  "v_mfma_f32_16x16x32_bf16 v[172:175], v[80:83], a[12:15], v[172:175]\n" \
  "s_waitcnt lgkmcnt(9)\n" \
  "v_mfma_f32_16x16x32_bf16 v[188:191], v[84:87], a[12:15], v[188:191]\n" \
  "ds_read_b64_tr_b16 v[74:75], %[va0] offset:4608\n" \
  "v_mfma_f32_16x16x32_bf16 v[184:187], v[84:87], a[8:11], v[184:187]\n" \
  "buffer_load_dwordx4 a[36:39], %[vx1], %[xr], %[sx] offen sc1\n" \
  "v_mfma_f32_16x16x32_bf16 v[180:183], v[84:87], a[4:7], v[180:183]\n" \
  "ds_read_b64_tr_b16 v[76:77], %[va1] offset:4096\n" \
  "v_mfma_f32_16x16x32_bf16 v[176:179], v[84:87], a[0:3], v[176:179]\n" \
  "s_waitcnt lgkmcnt(9)\n" \
  "ds_read_b64_tr_b16 v[78:79], %[va1] offset:4608\n" \
  "v_mfma_f32_16x16x32_bf16 v[192:195], v[88:91], a[0:3], v[192:195]\n" \
  "ds_read_b64_tr_b16 v[80:81], %[va2] offset:4096\n" \
  "v_mfma_f32_16x16x32_bf16 v[196:199], v[88:91], a[4:7], v[196:199]\n" \
  "buffer_load_dwordx4 a[40:43], %[vx2], %[xr], %[sx] offen sc1\n" \
  "v_mfma_f32_16x16x32_bf16 v[200:203], v[88:91], a[8:11], v[200:203]\n" \
  "ds_read_b64_tr_b16 v[82:83], %[va2] offset:4608\n" \
  "v_mfma_f32_16x16x32_bf16 v[204:207], v[88:91], a[12:15], v[204:207]\n" \
  "s_waitcnt lgkmcnt(10)\n" \
  "v_mfma_f32_16x16x32_bf16 v[220:223], v[92:95], a[12:15], v[220:223]\n" \
  "ds_read_b64_tr_b16 v[84:85], %[va3] offset:4096\n" \
  "v_mfma_f32_16x16x32_bf16 v[216:219], v[92:95], a[8:11], v[216:219]\n" \
  "buffer_load_dwordx4 a[44:47], %[vx3], %[xr], %[sx] offen sc1\n" \
  "v_mfma_f32_16x16x32_bf16 v[212:215], v[92:95], a[4:7], v[212:215]\n" \
  "ds_read_b64_tr_b16 v[86:87], %[va3] offset:4608\n" \
  "s_load_dword s64, %[fp], %[sl] offset:0x8 glc\n" \
  "v_mfma_f32_16x16x32_bf16 v[208:211], v[92:95], a[0:3], v[208:211]\n" \
  "s_waitcnt lgkmcnt(8)\n" \
  "s_add_u32 %[sx], %[sx], 4096\n" \
  "v_mfma_f32_16x16x32_bf16 v[224:227], v[64:67], a[0:3], v[224:227]\n" \
  "ds_read_b64_tr_b16 v[88:89], %[va0] offset:12288\n" \
  "v_mfma_f32_16x16x32_bf16 v[228:231], v[64:67], a[4:7], v[228:231]\n" \
  "v_mfma_f32_16x16x32_bf16 v[232:235], v[64:67], a[8:11], v[232:235]\n" \
  "ds_read_b64_tr_b16 v[90:91], %[va0] offset:12800\n" \
  "v_mfma_f32_16x16x32_bf16 v[236:239], v[64:67], a[12:15], v[236:239]\n" \
  "s_waitcnt vmcnt(9) lgkmcnt(8)\n" \
  "s_cmp_ge_u32 s65, %[tag]\n" \
  "v_mfma_f32_16x16x32_bf16 v[96:99], v[72:75], a[16:19], v[96:99]\n" \
  "ds_read_b64_tr_b16 v[92:93], %[va1] offset:12288\n" \
  "s_cbranch_scc0 .LslowB3_%=\n" \
  ".LbackB3_%=:\n" \
  "v_mfma_f32_16x16x32_bf16 v[100:103], v[72:75], a[20:23], v[100:103]\n" \
  "s_add_u32 %[tag], %[tag], 1\n" \
  "v_mfma_f32_16x16x32_bf16 v[104:107], v[72:75], a[24:27], v[104:107]\n" \
  "ds_read_b64_tr_b16 v[94:95], %[va1] offset:12800\n" \
  "v_mfma_f32_16x16x32_bf16 v[108:111], v[72:75], a[28:31], v[108:111]\n" \
  "s_waitcnt lgkmcnt(8)\n" \
  "v_mfma_f32_16x16x32_bf16 v[124:127], v[76:79], a[28:31], v[124:127]\n" \
  "ds_read_b64_tr_b16 v[64:65], %[va2] offset:12288\n" \
  "v_mfma_f32_16x16x32_bf16 v[120:123], v[76:79], a[24:27], v[120:123]\n" \
  "buffer_load_dwordx4 a[48:51], %[vx0], %[xr], %[sx] offen sc1\n" \
  "v_mfma_f32_16x16x32_bf16 v[116:119], v[76:79], a[20:23], v[116:119]\n" \
  "ds_read_b64_tr_b16 v[66:67], %[va2] offset:12800\n" \
  "v_mfma_f32_16x16x32_bf16 v[112:115], v[76:79], a[16:19], v[112:115]\n" \
  "s_waitcnt lgkmcnt(8)\n" \
  "v_mfma_f32_16x16x32_bf16 v[128:131], v[80:83], a[16:19], v[128:131]\n" \
  "ds_read_b64_tr_b16 v[68:69], %[va3] offset:12288\n" \
  "v_mfma_f32_16x16x32_bf16 v[132:135], v[80:83], a[20:23], v[132:135]\n" \
  "buffer_load_dwordx4 a[52:55], %[vx1], %[xr], %[sx] offen sc1\n" \
  "v_mfma_f32_16x16x32_bf16 v[136:139], v[80:83], a[24:27], v[136:139]\n" \
  "ds_read_b64_tr_b16 v[70:71], %[va3] offset:12800\n" \
  "v_mfma_f32_16x16x32_bf16 v[140:143], v[80:83], a[28:31], v[140:143]\n" \
  "s_waitcnt lgkmcnt(8)\n" \
  "ds_read_b64_tr_b16 v[72:73], %[vb0] offset:2048\n" \
  "v_mfma_f32_16x16x32_bf16 v[156:159], v[84:87], a[28:31], v[156:159]\n" \
  "ds_read_b64_tr_b16 v[74:75], %[vb0] offset:2304\n" \
  "v_mfma_f32_16x16x32_bf16 v[152:155], v[84:87], a[24:27], v[152:155]\n" \
  "buffer_load_dwordx4 a[56:59], %[vx2], %[xr], %[sx] offen sc1\n" \
  "v_mfma_f32_16x16x32_bf16 v[148:151], v[84:87], a[20:23], v[148:151]\n" \
  "ds_read_b64_tr_b16 v[76:77], %[vb1] offset:2048\n" \
  "v_mfma_f32_16x16x32_bf16 v[144:147], v[84:87], a[16:19], v[144:147]\n" \
  "s_waitcnt lgkmcnt(9)\n" \
  "v_mfma_f32_16x16x32_bf16 v[160:163], v[88:91], a[16:19], v[160:163]\n" \
  "ds_read_b64_tr_b16 v[78:79], %[vb1] offset:2304\n" \
  "v_mfma_f32_16x16x32_bf16 v[164:167], v[88:91], a[20:23], v[164:167]\n" \
  "s_barrier\n" \
  "v_mfma_f32_16x16x32_bf16 v[168:171], v[88:91], a[24:27], v[168:171]\n" \
  "ds_read_b64_tr_b16 v[80:81], %[va0] offset:16384\n" \
  "s_load_dword s65, %[fp], %[sl] offset:0x808 glc\n" \
  "v_mfma_f32_16x16x32_bf16 v[172:175], v[88:91], a[28:31], v[172:175]\n" \
  "s_waitcnt lgkmcnt(9)\n" \
  "s_add_u32 m0, %[ma], 49152\n" \
  "v_mfma_f32_16x16x32_bf16 v[188:191], v[92:95], a[28:31], v[188:191]\n" \
  "ds_read_b64_tr_b16 v[82:83], %[va0] offset:16896\n" \
  "v_mfma_f32_16x16x32_bf16 v[184:187], v[92:95], a[24:27], v[184:187]\n" \
  "buffer_load_dwordx4 a[60:63], %[vx3], %[xr], %[sx] offen sc1\n" \
  "v_mfma_f32_16x16x32_bf16 v[180:183], v[92:95], a[20:23], v[180:183]\n" \
  "ds_read_b64_tr_b16 v[84:85], %[va1] offset:16384\n" \
  "s_add_u32 %[sx], %[sx], 4096\n" \
  "v_mfma_f32_16x16x32_bf16 v[176:179], v[92:95], a[16:19], v[176:179]\n" \
  "s_waitcnt lgkmcnt(9)\n" \
  "ds_read_b64_tr_b16 v[86:87], %[va1] offset:16896\n" \
  "v_mfma_f32_16x16x32_bf16 v[192:195], v[64:67], a[16:19], v[192:195]\n" \
  "ds_read_b64_tr_b16 v[88:89], %[va2] offset:16384\n" \
  "v_mfma_f32_16x16x32_bf16 v[196:199], v[64:67], a[20:23], v[196:199]\n" \
  "buffer_load_dwordx4 %[vd0], %[wr], %[sw] offen offset:0 lds\n" \
  "v_mfma_f32_16x16x32_bf16 v[200:203], v[64:67], a[24:27], v[200:203]\n" \
  "ds_read_b64_tr_b16 v[90:91], %[va2] offset:16896\n" \
  "v_mfma_f32_16x16x32_bf16 v[204:207], v[64:67], a[28:31], v[204:207]\n" \
  "s_waitcnt lgkmcnt(10)\n" \
  "v_mfma_f32_16x16x32_bf16 v[220:223], v[68:71], a[28:31], v[220:223]\n" \
  "ds_read_b64_tr_b16 v[92:93], %[va3] offset:16384\n" \
  "v_mfma_f32_16x16x32_bf16 v[216:219], v[68:71], a[24:27], v[216:219]\n" \
  "v_mfma_f32_16x16x32_bf16 v[212:215], v[68:71], a[20:23], v[212:215]\n" \
  "ds_read_b64_tr_b16 v[94:95], %[va3] offset:16896\n" \
  "v_mfma_f32_16x16x32_bf16 v[208:211], v[68:71], a[16:19], v[208:211]\n" \
  "s_waitcnt lgkmcnt(8)\n" \
  "v_mfma_f32_16x16x32_bf16 v[224:227], v[72:75], a[16:19], v[224:227]\n" \
  "ds_read_b64_tr_b16 v[64:65], %[va0] offset:24576\n" \
  "v_mfma_f32_16x16x32_bf16 v[228:231], v[72:75], a[20:23], v[228:231]\n" \
  "buffer_load_dwordx4 %[vd1], %[wr], %[sw] offen offset:1024 lds\n" \
  "v_mfma_f32_16x16x32_bf16 v[232:235], v[72:75], a[24:27], v[232:235]\n" \
  "ds_read_b64_tr_b16 v[66:67], %[va0] offset:25088\n" \
  "v_mfma_f32_16x16x32_bf16 v[236:239], v[72:75], a[28:31], v[236:239]\n" \
  "s_waitcnt vmcnt(6) lgkmcnt(8)\n" \
  "v_mfma_f32_16x16x32_bf16 v[96:99], v[80:83], a[32:35], v[96:99]\n" \
  "ds_read_b64_tr_b16 v[68:69], %[va1] offset:24576\n" \
  "v_mfma_f32_16x16x32_bf16 v[100:103], v[80:83], a[36:39], v[100:103]\n" \
  "buffer_load_dwordx4 %[vd2], %[wr], %[sw] offen offset:2048 lds\n" \
  "v_mfma_f32_16x16x32_bf16 v[104:107], v[80:83], a[40:43], v[104:107]\n" \
  "ds_read_b64_tr_b16 v[70:71], %[va1] offset:25088\n" \
  "v_mfma_f32_16x16x32_bf16 v[108:111], v[80:83], a[44:47], v[108:111]\n" \
  "s_waitcnt lgkmcnt(8)\n" \
  "v_mfma_f32_16x16x32_bf16 v[124:127], v[84:87], a[44:47], v[124:127]\n" \
  "ds_read_b64_tr_b16 v[72:73], %[va2] offset:24576\n" \
  "v_mfma_f32_16x16x32_bf16 v[120:123], v[84:87], a[40:43], v[120:123]\n" \
  "buffer_load_dwordx4 %[vd3], %[wr], %[sw] offen offset:3072 lds\n" \
  "v_mfma_f32_16x16x32_bf16 v[116:119], v[84:87], a[36:39], v[116:119]\n" \
  "ds_read_b64_tr_b16 v[74:75], %[va2] offset:25088\n" \
  "s_add_u32 m0, %[mb], 12288\n" \
  "v_mfma_f32_16x16x32_bf16 v[112:115], v[84:87], a[32:35], v[112:115]\n" \
  "s_waitcnt lgkmcnt(8)\n" \
  "v_mfma_f32_16x16x32_bf16 v[128:131], v[88:91], a[32:35], v[128:131]\n" \
  "ds_read_b64_tr_b16 v[76:77], %[va3] offset:24576\n" \
  "v_mfma_f32_16x16x32_bf16 v[132:135], v[88:91], a[36:39], v[132:135]\n" \
  "buffer_load_dwordx4 %[vd4], %[wr], %[sw] offen lds\n" \
  "v_mfma_f32_16x16x32_bf16 v[136:139], v[88:91], a[40:43], v[136:139]\n" \
  "ds_read_b64_tr_b16 v[78:79], %[va3] offset:25088\n" \
  "s_cmp_ge_u32 s64, %[tag]\n" \
  "v_mfma_f32_16x16x32_bf16 v[140:143], v[88:91], a[44:47], v[140:143]\n" \
  "s_waitcnt lgkmcnt(8)\n" \
  "ds_read_b64_tr_b16 v[80:81], %[vb0] offset:4096\n" \
  "v_mfma_f32_16x16x32_bf16 v[156:159], v[92:95], a[44:47], v[156:159]\n" \
  "ds_read_b64_tr_b16 v[82:83], %[vb0] offset:4352\n" \
  "s_cbranch_scc0 .LslowB4_%=\n" \
  ".LbackB4_%=:\n" \
  "v_mfma_f32_16x16x32_bf16 v[152:155], v[92:95], a[40:43], v[152:155]\n" \
  "s_add_u32 %[tag], %[tag], 1\n" \
  "s_add_u32 %[sw], %[sw], 262144\n" \
  "v_mfma_f32_16x16x32_bf16 v[148:151], v[92:95], a[36:39], v[148:151]\n" \
  "ds_read_b64_tr_b16 v[84:85], %[vb1] offset:4096\n" \
  "v_mfma_f32_16x16x32_bf16 v[144:147], v[92:95], a[32:35], v[144:147]\n" \
  "s_waitcnt lgkmcnt(9)\n" \
  "v_mfma_f32_16x16x32_bf16 v[160:163], v[64:67], a[32:35], v[160:163]\n" \
  "ds_read_b64_tr_b16 v[86:87], %[vb1] offset:4352\n" \
  "v_mfma_f32_16x16x32_bf16 v[164:167], v[64:67], a[36:39], v[164:167]\n" \
  "buffer_load_dwordx4 a[64:67], %[vx0], %[xr], %[sx] offen sc1\n" \
  "v_mfma_f32_16x16x32_bf16 v[168:171], v[64:67], a[40:43], v[168:171]\n" \
  "ds_read_b64_tr_b16 v[88:89], %[va0] offset:20480\n" \
  "v_mfma_f32_16x16x32_bf16 v[172:175], v[64:67], a[44:47], v[172:175]\n" \
  "s_waitcnt lgkmcnt(9)\n" \
  "v_mfma_f32_16x16x32_bf16 v[188:191], v[68:71], a[44:47], v[188:191]\n" \
  "ds_read_b64_tr_b16 v[90:91], %[va0] offset:20992\n" \
  "v_mfma_f32_16x16x32_bf16 v[184:187], v[68:71], a[40:43], v[184:187]\n" \
  "buffer_load_dwordx4 a[68:71], %[vx1], %[xr], %[sx] offen sc1\n" \
  "v_mfma_f32_16x16x32_bf16 v[180:183], v[68:71], a[36:39], v[180:183]\n" \
  "ds_read_b64_tr_b16 v[92:93], %[va1] offset:20480\n" \
  "v_mfma_f32_16x16x32_bf16 v[176:179], v[68:71], a[32:35], v[176:179]\n" \
  "s_waitcnt lgkmcnt(9)\n" \
  "ds_read_b64_tr_b16 v[94:95], %[va1] offset:20992\n" \
  "v_mfma_f32_16x16x32_bf16 v[192:195], v[72:75], a[32:35], v[192:195]\n" \
  "ds_read_b64_tr_b16 v[64:65], %[va2] offset:20480\n" \
  "v_mfma_f32_16x16x32_bf16 v[196:199], v[72:75], a[36:39], v[196:199]\n" \
  "buffer_load_dwordx4 a[72:75], %[vx2], %[xr], %[sx] offen sc1\n" \
  "v_mfma_f32_16x16x32_bf16 v[200:203], v[72:75], a[40:43], v[200:203]\n" \
  "ds_read_b64_tr_b16 v[66:67], %[va2] offset:20992\n" \
  "v_mfma_f32_16x16x32_bf16 v[204:207], v[72:75], a[44:47], v[204:207]\n" \
  "s_waitcnt lgkmcnt(10)\n" \
  "v_mfma_f32_16x16x32_bf16 v[220:223], v[76:79], a[44:47], v[220:223]\n" \
  "ds_read_b64_tr_b16 v[68:69], %[va3] offset:20480\n" \
  "v_mfma_f32_16x16x32_bf16 v[216:219], v[76:79], a[40:43], v[216:219]\n" \
  "buffer_load_dwordx4 a[76:79], %[vx3], %[xr], %[sx] offen sc1\n" \
  "v_mfma_f32_16x16x32_bf16 v[212:215], v[76:79], a[36:39], v[212:215]\n" \
  "ds_read_b64_tr_b16 v[70:71], %[va3] offset:20992\n" \
  "s_load_dword s64, %[fp], %[sl] offset:0xc glc\n" \
  "v_mfma_f32_16x16x32_bf16 v[208:211], v[76:79], a[32:35], v[208:211]\n" \
  "s_waitcnt lgkmcnt(8)\n" \
  "s_add_u32 %[sx], %[sx], 4096\n" \
  "v_mfma_f32_16x16x32_bf16 v[224:227], v[80:83], a[32:35], v[224:227]\n" \
  "ds_read_b64_tr_b16 v[72:73], %[va0] offset:28672\n" \
  "v_mfma_f32_16x16x32_bf16 v[228:231], v[80:83], a[36:39], v[228:231]\n" \
  "v_mfma_f32_16x16x32_bf16 v[232:235], v[80:83], a[40:43], v[232:235]\n" \
  "ds_read_b64_tr_b16 v[74:75], %[va0] offset:29184\n" \
  "v_mfma_f32_16x16x32_bf16 v[236:239], v[80:83], a[44:47], v[236:239]\n" \
  "s_waitcnt vmcnt(9) lgkmcnt(8)\n" \
  "s_cmp_ge_u32 s65, %[tag]\n" \
  "v_mfma_f32_16x16x32_bf16 v[96:99], v[88:91], a[48:51], v[96:99]\n" \
  "ds_read_b64_tr_b16 v[76:77], %[va1] offset:28672\n" \
  "s_cbranch_scc0 .LslowB5_%=\n" \
  ".LbackB5_%=:\n" \
  "v_mfma_f32_16x16x32_bf16 v[100:103], v[88:91], a[52:55], v[100:103]\n" \
  "s_add_u32 %[tag], %[tag], 1\n" \
  "v_mfma_f32_16x16x32_bf16 v[104:107], v[88:91], a[56:59], v[104:107]\n" \
  "ds_read_b64_tr_b16 v[78:79], %[va1] offset:29184\n" \
  "v_mfma_f32_16x16x32_bf16 v[108:111], v[88:91], a[60:63], v[108:111]\n" \
  "s_waitcnt lgkmcnt(8)\n" \
  "v_mfma_f32_16x16x32_bf16 v[124:127], v[92:95], a[60:63], v[124:127]\n" \
  "ds_read_b64_tr_b16 v[80:81], %[va2] offset:28672\n" \
  "v_mfma_f32_16x16x32_bf16 v[120:123], v[92:95], a[56:59], v[120:123]\n" \
  "buffer_load_dwordx4 a[80:83], %[vx0], %[xr], %[sx] offen sc1\n" \
  "v_mfma_f32_16x16x32_bf16 v[116:119], v[92:95], a[52:55], v[116:119]\n" \
  "ds_read_b64_tr_b16 v[82:83], %[va2] offset:29184\n" \
  "v_mfma_f32_16x16x32_bf16 v[112:115], v[92:95], a[48:51], v[112:115]\n" \
  "s_waitcnt lgkmcnt(8)\n" \
  "v_mfma_f32_16x16x32_bf16 v[128:131], v[64:67], a[48:51], v[128:131]\n" \
  "ds_read_b64_tr_b16 v[84:85], %[va3] offset:28672\n" \
  "v_mfma_f32_16x16x32_bf16 v[132:135], v[64:67], a[52:55], v[132:135]\n" \
  "buffer_load_dwordx4 a[84:87], %[vx1], %[xr], %[sx] offen sc1\n" \
  "v_mfma_f32_16x16x32_bf16 v[136:139], v[64:67], a[56:59], v[136:139]\n" \
  "ds_read_b64_tr_b16 v[86:87], %[va3] offset:29184\n" \
  "v_mfma_f32_16x16x32_bf16 v[140:143], v[64:67], a[60:63], v[140:143]\n" \
  "s_waitcnt lgkmcnt(8)\n" \
  "ds_read_b64_tr_b16 v[88:89], %[vb0] offset:6144\n" \
  "v_mfma_f32_16x16x32_bf16 v[156:159], v[68:71], a[60:63], v[156:159]\n" \
  "ds_read_b64_tr_b16 v[90:91], %[vb0] offset:6400\n" \
  "v_mfma_f32_16x16x32_bf16 v[152:155], v[68:71], a[56:59], v[152:155]\n" \
  "buffer_load_dwordx4 a[88:91], %[vx2], %[xr], %[sx] offen sc1\n" \
  "v_mfma_f32_16x16x32_bf16 v[148:151], v[68:71], a[52:55], v[148:151]\n" \
  "ds_read_b64_tr_b16 v[92:93], %[vb1] offset:6144\n" \
  "v_mfma_f32_16x16x32_bf16 v[144:147], v[68:71], a[48:51], v[144:147]\n" \
  "s_waitcnt lgkmcnt(9)\n" \
  "v_mfma_f32_16x16x32_bf16 v[160:163], v[72:75], a[48:51], v[160:163]\n" \
  "ds_read_b64_tr_b16 v[94:95], %[vb1] offset:6400\n" \
  "v_mfma_f32_16x16x32_bf16 v[164:167], v[72:75], a[52:55], v[164:167]\n" \
  "s_barrier\n" \
  "v_mfma_f32_16x16x32_bf16 v[168:171], v[72:75], a[56:59], v[168:171]\n" \
  "ds_read_b64_tr_b16 v[64:65], %[va0] offset:32768\n" \
  "s_load_dword s65, %[fp], %[sl] offset:0x80c glc\n" \
  "v_mfma_f32_16x16x32_bf16 v[172:175], v[72:75], a[60:63], v[172:175]\n" \
  "s_waitcnt lgkmcnt(9)\n" \
  "s_add_u32 m0, %[ma], 0\n" \
  "v_mfma_f32_16x16x32_bf16 v[188:191], v[76:79], a[60:63], v[188:191]\n" \
  "ds_read_b64_tr_b16 v[66:67], %[va0] offset:33280\n" \
  "v_mfma_f32_16x16x32_bf16 v[184:187], v[76:79], a[56:59], v[184:187]\n" \
  "buffer_load_dwordx4 a[92:95], %[vx3], %[xr], %[sx] offen sc1\n" \
  "v_mfma_f32_16x16x32_bf16 v[180:183], v[76:79], a[52:55], v[180:183]\n" \
  "ds_read_b64_tr_b16 v[68:69], %[va1] offset:32768\n" \
  "s_add_u32 %[sx], %[sx], 4096\n" \
  "v_mfma_f32_16x16x32_bf16 v[176:179], v[76:79], a[48:51], v[176:179]\n" \
  "s_waitcnt lgkmcnt(9)\n" \
  "ds_read_b64_tr_b16 v[70:71], %[va1] offset:33280\n" \
  "v_mfma_f32_16x16x32_bf16 v[192:195], v[80:83], a[48:51], v[192:195]\n" \
  "ds_read_b64_tr_b16 v[72:73], %[va2] offset:32768\n" \
  "v_mfma_f32_16x16x32_bf16 v[196:199], v[80:83], a[52:55], v[196:199]\n" \
  "buffer_load_dwordx4 %[vd0], %[wr], %[sw] offen offset:0 lds\n" \
  "v_mfma_f32_16x16x32_bf16 v[200:203], v[80:83], a[56:59], v[200:203]\n" \
  "ds_read_b64_tr_b16 v[74:75], %[va2] offset:33280\n" \
  "v_mfma_f32_16x16x32_bf16 v[204:207], v[80:83], a[60:63], v[204:207]\n" \
  "s_waitcnt lgkmcnt(10)\n" \
  "v_mfma_f32_16x16x32_bf16 v[220:223], v[84:87], a[60:63], v[220:223]\n" \
  "ds_read_b64_tr_b16 v[76:77], %[va3] offset:32768\n" \
  "v_mfma_f32_16x16x32_bf16 v[216:219], v[84:87], a[56:59], v[216:219]\n" \
  "v_mfma_f32_16x16x32_bf16 v[212:215], v[84:87], a[52:55], v[212:215]\n" \
  "ds_read_b64_tr_b16 v[78:79], %[va3] offset:33280\n" \
  "v_mfma_f32_16x16x32_bf16 v[208:211], v[84:87], a[48:51], v[208:211]\n" \
  "s_waitcnt lgkmcnt(8)\n" \
  "v_mfma_f32_16x16x32_bf16 v[224:227], v[88:91], a[48:51], v[224:227]\n" \
  "ds_read_b64_tr_b16 v[80:81], %[va0] offset:40960\n" \
  "v_mfma_f32_16x16x32_bf16 v[228:231], v[88:91], a[52:55], v[228:231]\n" \
  "buffer_load_dwordx4 %[vd1], %[wr], %[sw] offen offset:1024 lds\n" \
  "v_mfma_f32_16x16x32_bf16 v[232:235], v[88:91], a[56:59], v[232:235]\n" \
  "ds_read_b64_tr_b16 v[82:83], %[va0] offset:41472\n" \
  "v_mfma_f32_16x16x32_bf16 v[236:239], v[88:91], a[60:63], v[236:239]\n" \
  "s_waitcnt vmcnt(6) lgkmcnt(8)\n" \
  "v_mfma_f32_16x16x32_bf16 v[96:99], v[64:67], a[64:67], v[96:99]\n" \
  "ds_read_b64_tr_b16 v[84:85], %[va1] offset:40960\n" \
  "v_mfma_f32_16x16x32_bf16 v[100:103], v[64:67], a[68:71], v[100:103]\n" \
  "buffer_load_dwordx4 %[vd2], %[wr], %[sw] offen offset:2048 lds\n" \
  "v_mfma_f32_16x16x32_bf16 v[104:107], v[64:67], a[72:75], v[104:107]\n" \
  "ds_read_b64_tr_b16 v[86:87], %[va1] offset:41472\n" \
  "v_mfma_f32_16x16x32_bf16 v[108:111], v[64:67], a[76:79], v[108:111]\n" \
  "s_waitcnt lgkmcnt(8)\n" \
  "v_mfma_f32_16x16x32_bf16 v[124:127], v[68:71], a[76:79], v[124:127]\n" \
  "ds_read_b64_tr_b16 v[88:89], %[va2] offset:40960\n" \
  "v_mfma_f32_16x16x32_bf16 v[120:123], v[68:71], a[72:75], v[120:123]\n" \
  "buffer_load_dwordx4 %[vd3], %[wr], %[sw] offen offset:3072 lds\n" \
  "v_mfma_f32_16x16x32_bf16 v[116:119], v[68:71], a[68:71], v[116:119]\n" \
  "ds_read_b64_tr_b16 v[90:91], %[va2] offset:41472\n" \
  "s_add_u32 m0, %[mb], 0\n" \
  "v_mfma_f32_16x16x32_bf16 v[112:115], v[68:71], a[64:67], v[112:115]\n" \
  "s_waitcnt lgkmcnt(8)\n" \
  "v_mfma_f32_16x16x32_bf16 v[128:131], v[72:75], a[64:67], v[128:131]\n" \
  "ds_read_b64_tr_b16 v[92:93], %[va3] offset:40960\n" \
  "v_mfma_f32_16x16x32_bf16 v[132:135], v[72:75], a[68:71], v[132:135]\n" \
  "buffer_load_dwordx4 %[vd4], %[wr], %[sw] offen lds\n" \
  "v_mfma_f32_16x16x32_bf16 v[136:139], v[72:75], a[72:75], v[136:139]\n" \
  "ds_read_b64_tr_b16 v[94:95], %[va3] offset:41472\n" \
  "s_cmp_ge_u32 s64, %[tag]\n" \
  "v_mfma_f32_16x16x32_bf16 v[140:143], v[72:75], a[76:79], v[140:143]\n" \
  "s_waitcnt lgkmcnt(8)\n" \
  "ds_read_b64_tr_b16 v[64:65], %[vb0] offset:8192\n" \
  "v_mfma_f32_16x16x32_bf16 v[156:159], v[76:79], a[76:79], v[156:159]\n" \
  "ds_read_b64_tr_b16 v[66:67], %[vb0] offset:8448\n" \
  "s_cbranch_scc0 .LslowB6_%=\n" \
  ".LbackB6_%=:\n" \
  "v_mfma_f32_16x16x32_bf16 v[152:155], v[76:79], a[72:75], v[152:155]\n" \
  "s_add_u32 %[tag], %[tag], 1\n" \
  "s_add_u32 %[sw], %[sw], 262144\n" \
  "v_mfma_f32_16x16x32_bf16 v[148:151], v[76:79], a[68:71], v[148:151]\n" \
  "ds_read_b64_tr_b16 v[68:69], %[vb1] offset:8192\n" \
  "v_mfma_f32_16x16x32_bf16 v[144:147], v[76:79], a[64:67], v[144:147]\n" \
  "s_waitcnt lgkmcnt(9)\n" \
  "v_mfma_f32_16x16x32_bf16 v[160:163], v[80:83], a[64:67], v[160:163]\n" \
  "ds_read_b64_tr_b16 v[70:71], %[vb1] offset:8448\n" \
  "v_mfma_f32_16x16x32_bf16 v[164:167], v[80:83], a[68:71], v[164:167]\n" \
  "buffer_load_dwordx4 a[96:99], %[vx0], %[xr], %[sx] offen sc1\n" \
  "v_mfma_f32_16x16x32_bf16 v[168:171], v[80:83], a[72:75], v[168:171]\n" \
  "ds_read_b64_tr_b16 v[72:73], %[va0] offset:36864\n" \
  "v_mfma_f32_16x16x32_bf16 v[172:175], v[80:83], a[76:79], v[172:175]\n" \
  "s_waitcnt lgkmcnt(9)\n" \
  "v_mfma_f32_16x16x32_bf16 v[188:191], v[84:87], a[76:79], v[188:191]\n" \
  "ds_read_b64_tr_b16 v[74:75], %[va0] offset:37376\n" \
  "v_mfma_f32_16x16x32_bf16 v[184:187], v[84:87], a[72:75], v[184:187]\n" \
  "buffer_load_dwordx4 a[100:103], %[vx1], %[xr], %[sx] offen sc1\n" \
  "v_mfma_f32_16x16x32_bf16 v[180:183], v[84:87], a[68:71], v[180:183]\n" \
  "ds_read_b64_tr_b16 v[76:77], %[va1] offset:36864\n" \
  "v_mfma_f32_16x16x32_bf16 v[176:179], v[84:87], a[64:67], v[176:179]\n" \
  "s_waitcnt lgkmcnt(9)\n" \
  "ds_read_b64_tr_b16 v[78:79], %[va1] offset:37376\n" \
  "v_mfma_f32_16x16x32_bf16 v[192:195], v[88:91], a[64:67], v[192:195]\n" \
  "ds_read_b64_tr_b16 v[80:81], %[va2] offset:36864\n" \
  "v_mfma_f32_16x16x32_bf16 v[196:199], v[88:91], a[68:71], v[196:199]\n" \
  "buffer_load_dwordx4 a[104:107], %[vx2], %[xr], %[sx] offen sc1\n" \
  "v_mfma_f32_16x16x32_bf16 v[200:203], v[88:91], a[72:75], v[200:203]\n" \
  "ds_read_b64_tr_b16 v[82:83], %[va2] offset:37376\n" \
  "v_mfma_f32_16x16x32_bf16 v[204:207], v[88:91], a[76:79], v[204:207]\n" \
  "s_waitcnt lgkmcnt(10)\n" \
  "v_mfma_f32_16x16x32_bf16 v[220:223], v[92:95], a[76:79], v[220:223]\n" \
  "ds_read_b64_tr_b16 v[84:85], %[va3] offset:36864\n" \
  "v_mfma_f32_16x16x32_bf16 v[216:219], v[92:95], a[72:75], v[216:219]\n" \
  "buffer_load_dwordx4 a[108:111], %[vx3], %[xr], %[sx] offen sc1\n" \
  "v_mfma_f32_16x16x32_bf16 v[212:215], v[92:95], a[68:71], v[212:215]\n" \
  "ds_read_b64_tr_b16 v[86:87], %[va3] offset:37376\n" \
  "s_load_dword s64, %[fp], %[sl] offset:0x10 glc\n" \
  "v_mfma_f32_16x16x32_bf16 v[208:211], v[92:95], a[64:67], v[208:211]\n" \
  "s_waitcnt lgkmcnt(8)\n" \
  "s_add_u32 %[sx], %[sx], 4096\n" \
  "v_mfma_f32_16x16x32_bf16 v[224:227], v[64:67], a[64:67], v[224:227]\n" \
  "ds_read_b64_tr_b16 v[88:89], %[va0] offset:45056\n" \
  "v_mfma_f32_16x16x32_bf16 v[228:231], v[64:67], a[68:71], v[228:231]\n" \
  "v_mfma_f32_16x16x32_bf16 v[232:235], v[64:67], a[72:75], v[232:235]\n" \
  "ds_read_b64_tr_b16 v[90:91], %[va0] offset:45568\n" \
  "v_mfma_f32_16x16x32_bf16 v[236:239], v[64:67], a[76:79], v[236:239]\n" \
  "s_waitcnt vmcnt(9) lgkmcnt(8)\n" \
  "s_cmp_ge_u32 s65, %[tag]\n" \
  "v_mfma_f32_16x16x32_bf16 v[96:99], v[72:75], a[80:83], v[96:99]\n" \
  "ds_read_b64_tr_b16 v[92:93], %[va1] offset:45056\n" \
  "s_cbranch_scc0 .LslowB7_%=\n" \
  ".LbackB7_%=:\n" \
  "v_mfma_f32_16x16x32_bf16 v[100:103], v[72:75], a[84:87], v[100:103]\n" \
  "s_add_u32 %[tag], %[tag], 1\n" \
  "v_mfma_f32_16x16x32_bf16 v[104:107], v[72:75], a[88:91], v[104:107]\n" \
  "ds_read_b64_tr_b16 v[94:95], %[va1] offset:45568\n" \
  "v_mfma_f32_16x16x32_bf16 v[108:111], v[72:75], a[92:95], v[108:111]\n" \
  "s_waitcnt lgkmcnt(8)\n" \
  "v_mfma_f32_16x16x32_bf16 v[124:127], v[76:79], a[92:95], v[124:127]\n" \
  "ds_read_b64_tr_b16 v[64:65], %[va2] offset:45056\n" \
  "v_mfma_f32_16x16x32_bf16 v[120:123], v[76:79], a[88:91], v[120:123]\n" \
  "buffer_load_dwordx4 a[112:115], %[vx0], %[xr], %[sx] offen sc1\n" \
  "v_mfma_f32_16x16x32_bf16 v[116:119], v[76:79], a[84:87], v[116:119]\n" \
  "ds_read_b64_tr_b16 v[66:67], %[va2] offset:45568\n" \
  "v_mfma_f32_16x16x32_bf16 v[112:115], v[76:79], a[80:83], v[112:115]\n" \
  "s_waitcnt lgkmcnt(8)\n" \
  "v_mfma_f32_16x16x32_bf16 v[128:131], v[80:83], a[80:83], v[128:131]\n" \
  "ds_read_b64_tr_b16 v[68:69], %[va3] offset:45056\n" \
  "v_mfma_f32_16x16x32_bf16 v[132:135], v[80:83], a[84:87], v[132:135]\n" \
  "buffer_load_dwordx4 a[116:119], %[vx1], %[xr], %[sx] offen sc1\n" \
  "v_mfma_f32_16x16x32_bf16 v[136:139], v[80:83], a[88:91], v[136:139]\n" \
  "ds_read_b64_tr_b16 v[70:71], %[va3] offset:45568\n" \
  "v_mfma_f32_16x16x32_bf16 v[140:143], v[80:83], a[92:95], v[140:143]\n" \
  "s_waitcnt lgkmcnt(8)\n" \
  "ds_read_b64_tr_b16 v[72:73], %[vb0] offset:10240\n" \
  "v_mfma_f32_16x16x32_bf16 v[156:159], v[84:87], a[92:95], v[156:159]\n" \
  "ds_read_b64_tr_b16 v[74:75], %[vb0] offset:10496\n" \
  "v_mfma_f32_16x16x32_bf16 v[152:155], v[84:87], a[88:91], v[152:155]\n" \
  "buffer_load_dwordx4 a[120:123], %[vx2], %[xr], %[sx] offen sc1\n" \
  "v_mfma_f32_16x16x32_bf16 v[148:151], v[84:87], a[84:87], v[148:151]\n" \
  "ds_read_b64_tr_b16 v[76:77], %[vb1] offset:10240\n" \
  "v_mfma_f32_16x16x32_bf16 v[144:147], v[84:87], a[80:83], v[144:147]\n" \
  "s_waitcnt lgkmcnt(9)\n" \
  "v_mfma_f32_16x16x32_bf16 v[160:163], v[88:91], a[80:83], v[160:163]\n" \
  "ds_read_b64_tr_b16 v[78:79], %[vb1] offset:10496\n" \
  "v_mfma_f32_16x16x32_bf16 v[164:167], v[88:91], a[84:87], v[164:167]\n" \
  "s_barrier\n" \
  "v_mfma_f32_16x16x32_bf16 v[168:171], v[88:91], a[88:91], v[168:171]\n" \
  "ds_read_b64_tr_b16 v[80:81], %[va0] offset:49152\n" \
  "s_load_dword s65, %[fp], %[sl] offset:0x810 glc\n" \
  "v_mfma_f32_16x16x32_bf16 v[172:175], v[88:91], a[92:95], v[172:175]\n" \
  "s_waitcnt lgkmcnt(9)\n" \
  "s_add_u32 m0, %[ma], 16384\n" \
  "v_mfma_f32_16x16x32_bf16 v[188:191], v[92:95], a[92:95], v[188:191]\n" \
  "ds_read_b64_tr_b16 v[82:83], %[va0] offset:49664\n" \
  "v_mfma_f32_16x16x32_bf16 v[184:187], v[92:95], a[88:91], v[184:187]\n" \
  "buffer_load_dwordx4 a[124:127], %[vx3], %[xr], %[sx] offen sc1\n" \
  "v_mfma_f32_16x16x32_bf16 v[180:183], v[92:95], a[84:87], v[180:183]\n" \
  "ds_read_b64_tr_b16 v[84:85], %[va1] offset:49152\n" \
  "s_add_u32 %[sx], %[sx], 4096\n" \
  "v_mfma_f32_16x16x32_bf16 v[176:179], v[92:95], a[80:83], v[176:179]\n" \
  "s_waitcnt lgkmcnt(9)\n" \
  "ds_read_b64_tr_b16 v[86:87], %[va1] offset:49664\n" \
  "v_mfma_f32_16x16x32_bf16 v[192:195], v[64:67], a[80:83], v[192:195]\n" \
  "ds_read_b64_tr_b16 v[88:89], %[va2] offset:49152\n" \
  "v_mfma_f32_16x16x32_bf16 v[196:199], v[64:67], a[84:87], v[196:199]\n" \
  "buffer_load_dwordx4 %[vd0], %[wr], %[sw] offen offset:0 lds\n" \
  "v_mfma_f32_16x16x32_bf16 v[200:203], v[64:67], a[88:91], v[200:203]\n" \
  "ds_read_b64_tr_b16 v[90:91], %[va2] offset:49664\n" \
  "v_mfma_f32_16x16x32_bf16 v[204:207], v[64:67], a[92:95], v[204:207]\n" \
  "s_waitcnt lgkmcnt(10)\n" \
  "v_mfma_f32_16x16x32_bf16 v[220:223], v[68:71], a[92:95], v[220:223]\n" \
  "ds_read_b64_tr_b16 v[92:93], %[va3] offset:49152\n" \
  "v_mfma_f32_16x16x32_bf16 v[216:219], v[68:71], a[88:91], v[216:219]\n" \
  "v_mfma_f32_16x16x32_bf16 v[212:215], v[68:71], a[84:87], v[212:215]\n" \
  "ds_read_b64_tr_b16 v[94:95], %[va3] offset:49664\n" \
  "v_mfma_f32_16x16x32_bf16 v[208:211], v[68:71], a[80:83], v[208:211]\n" \
  "s_waitcnt lgkmcnt(8)\n" \
  "v_mfma_f32_16x16x32_bf16 v[224:227], v[72:75], a[80:83], v[224:227]\n" \
  "ds_read_b64_tr_b16 v[64:65], %[va0] offset:57344\n" \
  "v_mfma_f32_16x16x32_bf16 v[228:231], v[72:75], a[84:87], v[228:231]\n" \
  "buffer_load_dwordx4 %[vd1], %[wr], %[sw] offen offset:1024 lds\n" \
  "v_mfma_f32_16x16x32_bf16 v[232:235], v[72:75], a[88:91], v[232:235]\n" \
  "ds_read_b64_tr_b16 v[66:67], %[va0] offset:57856\n" \
  "v_mfma_f32_16x16x32_bf16 v[236:239], v[72:75], a[92:95], v[236:239]\n" \
  "s_waitcnt vmcnt(6) lgkmcnt(8)\n" \
  "v_mfma_f32_16x16x32_bf16 v[96:99], v[80:83], a[96:99], v[96:99]\n" \
  "ds_read_b64_tr_b16 v[68:69], %[va1] offset:57344\n" \
  "v_mfma_f32_16x16x32_bf16 v[100:103], v[80:83], a[100:103], v[100:103]\n" \
  "buffer_load_dwordx4 %[vd2], %[wr], %[sw] offen offset:2048 lds\n" \
  "v_mfma_f32_16x16x32_bf16 v[104:107], v[80:83], a[104:107], v[104:107]\n" \
  "ds_read_b64_tr_b16 v[70:71], %[va1] offset:57856\n" \
  "v_mfma_f32_16x16x32_bf16 v[108:111], v[80:83], a[108:111], v[108:111]\n" \
  "s_waitcnt lgkmcnt(8)\n" \
  "v_mfma_f32_16x16x32_bf16 v[124:127], v[84:87], a[108:111], v[124:127]\n" \
  "ds_read_b64_tr_b16 v[72:73], %[va2] offset:57344\n" \
  "v_mfma_f32_16x16x32_bf16 v[120:123], v[84:87], a[104:107], v[120:123]\n" \
  "buffer_load_dwordx4 %[vd3], %[wr], %[sw] offen offset:3072 lds\n" \
  "v_mfma_f32_16x16x32_bf16 v[116:119], v[84:87], a[100:103], v[116:119]\n" \
  "ds_read_b64_tr_b16 v[74:75], %[va2] offset:57856\n" \
  "s_add_u32 m0, %[mb], 4096\n" \
  "v_mfma_f32_16x16x32_bf16 v[112:115], v[84:87], a[96:99], v[112:115]\n" \
  "s_waitcnt lgkmcnt(8)\n" \
  "v_mfma_f32_16x16x32_bf16 v[128:131], v[88:91], a[96:99], v[128:131]\n" \
  "ds_read_b64_tr_b16 v[76:77], %[va3] offset:57344\n" \
  "v_mfma_f32_16x16x32_bf16 v[132:135], v[88:91], a[100:103], v[132:135]\n" \
  "buffer_load_dwordx4 %[vd4], %[wr], %[sw] offen lds\n" \
  "v_mfma_f32_16x16x32_bf16 v[136:139], v[88:91], a[104:107], v[136:139]\n" \
  "ds_read_b64_tr_b16 v[78:79], %[va3] offset:57856\n" \
  "s_cmp_ge_u32 s64, %[tag]\n" \
  "v_mfma_f32_16x16x32_bf16 v[140:143], v[88:91], a[108:111], v[140:143]\n" \
  "s_waitcnt lgkmcnt(8)\n" \
  "ds_read_b64_tr_b16 v[80:81], %[vb0] offset:12288\n" \
  "v_mfma_f32_16x16x32_bf16 v[156:159], v[92:95], a[108:111], v[156:159]\n" \
  "ds_read_b64_tr_b16 v[82:83], %[vb0] offset:12544\n" \
  "s_cbranch_scc0 .LslowB8_%=\n" \
  ".LbackB8_%=:\n" \
  "v_mfma_f32_16x16x32_bf16 v[152:155], v[92:95], a[104:107], v[152:155]\n" \
  "s_add_u32 %[tag], %[tag], 1\n" \
  "s_add_u32 %[sw], %[sw], 262144\n" \
  "v_mfma_f32_16x16x32_bf16 v[148:151], v[92:95], a[100:103], v[148:151]\n" \
  "ds_read_b64_tr_b16 v[84:85], %[vb1] offset:12288\n" \
  "v_mfma_f32_16x16x32_bf16 v[144:147], v[92:95], a[96:99], v[144:147]\n" \
  "s_waitcnt lgkmcnt(9)\n" \
  "v_mfma_f32_16x16x32_bf16 v[160:163], v[64:67], a[96:99], v[160:163]\n" \
  "ds_read_b64_tr_b16 v[86:87], %[vb1] offset:12544\n" \
  "v_mfma_f32_16x16x32_bf16 v[164:167], v[64:67], a[100:103], v[164:167]\n" \
  "buffer_load_dwordx4 a[0:3], %[vx0], %[xr], %[sx] offen sc1\n" \
  "v_mfma_f32_16x16x32_bf16 v[168:171], v[64:67], a[104:107], v[168:171]\n" \
  "ds_read_b64_tr_b16 v[88:89], %[va0] offset:53248\n" \
  "v_mfma_f32_16x16x32_bf16 v[172:175], v[64:67], a[108:111], v[172:175]\n" \
  "s_waitcnt lgkmcnt(9)\n" \
  "v_mfma_f32_16x16x32_bf16 v[188:191], v[68:71], a[108:111], v[188:191]\n" \
  "ds_read_b64_tr_b16 v[90:91], %[va0] offset:53760\n" \
  "v_mfma_f32_16x16x32_bf16 v[184:187], v[68:71], a[104:107], v[184:187]\n" \
  "buffer_load_dwordx4 a[4:7], %[vx1], %[xr], %[sx] offen sc1\n" \
  "v_mfma_f32_16x16x32_bf16 v[180:183], v[68:71], a[100:103], v[180:183]\n" \
  "ds_read_b64_tr_b16 v[92:93], %[va1] offset:53248\n" \
  "v_mfma_f32_16x16x32_bf16 v[176:179], v[68:71], a[96:99], v[176:179]\n" \
  "s_waitcnt lgkmcnt(9)\n" \
  "ds_read_b64_tr_b16 v[94:95], %[va1] offset:53760\n" \
  "v_mfma_f32_16x16x32_bf16 v[192:195], v[72:75], a[96:99], v[192:195]\n" \
  "ds_read_b64_tr_b16 v[64:65], %[va2] offset:53248\n" \
  "v_mfma_f32_16x16x32_bf16 v[196:199], v[72:75], a[100:103], v[196:199]\n" \
  "buffer_load_dwordx4 a[8:11], %[vx2], %[xr], %[sx] offen sc1\n" \
  "v_mfma_f32_16x16x32_bf16 v[200:203], v[72:75], a[104:107], v[200:203]\n" \
  "ds_read_b64_tr_b16 v[66:67], %[va2] offset:53760\n" \
  "v_mfma_f32_16x16x32_bf16 v[204:207], v[72:75], a[108:111], v[204:207]\n" \
  "s_waitcnt lgkmcnt(10)\n" \
  "v_mfma_f32_16x16x32_bf16 v[220:223], v[76:79], a[108:111], v[220:223]\n" \
  "ds_read_b64_tr_b16 v[68:69], %[va3] offset:53248\n" \
  "v_mfma_f32_16x16x32_bf16 v[216:219], v[76:79], a[104:107], v[216:219]\n" \
  "buffer_load_dwordx4 a[12:15], %[vx3], %[xr], %[sx] offen sc1\n" \
  "v_mfma_f32_16x16x32_bf16 v[212:215], v[76:79], a[100:103], v[212:215]\n" \
  "ds_read_b64_tr_b16 v[70:71], %[va3] offset:53760\n" \
  "s_load_dword s64, %[fp], %[sl] offset:0x14 glc\n" \
  "v_mfma_f32_16x16x32_bf16 v[208:211], v[76:79], a[96:99], v[208:211]\n" \
  "s_waitcnt lgkmcnt(8)\n" \
  "s_add_u32 %[sx], %[sx], 4096\n" \
  "v_mfma_f32_16x16x32_bf16 v[224:227], v[80:83], a[96:99], v[224:227]\n" \
  "ds_read_b64_tr_b16 v[72:73], %[va0] offset:61440\n" \
  "v_mfma_f32_16x16x32_bf16 v[228:231], v[80:83], a[100:103], v[228:231]\n" \
  "v_mfma_f32_16x16x32_bf16 v[232:235], v[80:83], a[104:107], v[232:235]\n" \
  "ds_read_b64_tr_b16 v[74:75], %[va0] offset:61952\n" \
  "v_mfma_f32_16x16x32_bf16 v[236:239], v[80:83], a[108:111], v[236:239]\n" \
  "s_waitcnt vmcnt(9) lgkmcnt(8)\n" \
  "s_cmp_ge_u32 s65, %[tag]\n" \
  "v_mfma_f32_16x16x32_bf16 v[96:99], v[88:91], a[112:115], v[96:99]\n" \
  "ds_read_b64_tr_b16 v[76:77], %[va1] offset:61440\n" \
  "s_cbranch_scc0 .LslowB9_%=\n" \
  ".LbackB9_%=:\n" \
  "v_mfma_f32_16x16x32_bf16 v[100:103], v[88:91], a[116:119], v[100:103]\n" \
  "s_add_u32 %[tag], %[tag], 1\n" \
  "v_mfma_f32_16x16x32_bf16 v[104:107], v[88:91], a[120:123], v[104:107]\n" \
  "ds_read_b64_tr_b16 v[78:79], %[va1] offset:61952\n" \
  "v_mfma_f32_16x16x32_bf16 v[108:111], v[88:91], a[124:127], v[108:111]\n" \
  "s_waitcnt lgkmcnt(8)\n" \
  "v_mfma_f32_16x16x32_bf16 v[124:127], v[92:95], a[124:127], v[124:127]\n" \
  "ds_read_b64_tr_b16 v[80:81], %[va2] offset:61440\n" \
  "v_mfma_f32_16x16x32_bf16 v[120:123], v[92:95], a[120:123], v[120:123]\n" \
  "buffer_load_dwordx4 a[16:19], %[vx0], %[xr], %[sx] offen sc1\n" \
  "v_mfma_f32_16x16x32_bf16 v[116:119], v[92:95], a[116:119], v[116:119]\n" \
  "ds_read_b64_tr_b16 v[82:83], %[va2] offset:61952\n" \
  "v_mfma_f32_16x16x32_bf16 v[112:115], v[92:95], a[112:115], v[112:115]\n" \
  "s_waitcnt lgkmcnt(8)\n" \
  "v_mfma_f32_16x16x32_bf16 v[128:131], v[64:67], a[112:115], v[128:131]\n" \
  "ds_read_b64_tr_b16 v[84:85], %[va3] offset:61440\n" \
  "v_mfma_f32_16x16x32_bf16 v[132:135], v[64:67], a[116:119], v[132:135]\n" \
  "buffer_load_dwordx4 a[20:23], %[vx1], %[xr], %[sx] offen sc1\n" \
  "v_mfma_f32_16x16x32_bf16 v[136:139], v[64:67], a[120:123], v[136:139]\n" \
  "ds_read_b64_tr_b16 v[86:87], %[va3] offset:61952\n" \
  "v_mfma_f32_16x16x32_bf16 v[140:143], v[64:67], a[124:127], v[140:143]\n" \
  "s_waitcnt lgkmcnt(8)\n" \
  "ds_read_b64_tr_b16 v[88:89], %[vb0] offset:14336\n" \
  "v_mfma_f32_16x16x32_bf16 v[156:159], v[68:71], a[124:127], v[156:159]\n" \
  "ds_read_b64_tr_b16 v[90:91], %[vb0] offset:14592\n" \
  "v_mfma_f32_16x16x32_bf16 v[152:155], v[68:71], a[120:123], v[152:155]\n" \
  "buffer_load_dwordx4 a[24:27], %[vx2], %[xr], %[sx] offen sc1\n" \
  "v_mfma_f32_16x16x32_bf16 v[148:151], v[68:71], a[116:119], v[148:151]\n" \
  "ds_read_b64_tr_b16 v[92:93], %[vb1] offset:14336\n" \
  "v_mfma_f32_16x16x32_bf16 v[144:147], v[68:71], a[112:115], v[144:147]\n" \
  "s_waitcnt lgkmcnt(9)\n" \
  "v_mfma_f32_16x16x32_bf16 v[160:163], v[72:75], a[112:115], v[160:163]\n" \
  "ds_read_b64_tr_b16 v[94:95], %[vb1] offset:14592\n" \
  "v_mfma_f32_16x16x32_bf16 v[164:167], v[72:75], a[116:119], v[164:167]\n" \
  "s_barrier\n" \
  "v_mfma_f32_16x16x32_bf16 v[168:171], v[72:75], a[120:123], v[168:171]\n" \
  "ds_read_b64_tr_b16 v[64:65], %[va0] offset:0\n" \
  "s_load_dword s65, %[fp], %[sl] offset:0x814 glc\n" \
  "v_mfma_f32_16x16x32_bf16 v[172:175], v[72:75], a[124:127], v[172:175]\n" \
  "s_waitcnt lgkmcnt(9)\n" \
  "s_add_u32 m0, %[ma], 32768\n" \
  "v_mfma_f32_16x16x32_bf16 v[188:191], v[76:79], a[124:127], v[188:191]\n" \
  "ds_read_b64_tr_b16 v[66:67], %[va0] offset:512\n" \
  "v_mfma_f32_16x16x32_bf16 v[184:187], v[76:79], a[120:123], v[184:187]\n" \
  "buffer_load_dwordx4 a[28:31], %[vx3], %[xr], %[sx] offen sc1\n" \
  "v_mfma_f32_16x16x32_bf16 v[180:183], v[76:79], a[116:119], v[180:183]\n" \
  "ds_read_b64_tr_b16 v[68:69], %[va1] offset:0\n" \
  "s_add_u32 %[sx], %[sx], 4096\n" \
  "v_mfma_f32_16x16x32_bf16 v[176:179], v[76:79], a[112:115], v[176:179]\n" \
  "s_waitcnt lgkmcnt(9)\n" \
  "ds_read_b64_tr_b16 v[70:71], %[va1] offset:512\n" \
  "v_mfma_f32_16x16x32_bf16 v[192:195], v[80:83], a[112:115], v[192:195]\n" \
  "ds_read_b64_tr_b16 v[72:73], %[va2] offset:0\n" \
  "v_mfma_f32_16x16x32_bf16 v[196:199], v[80:83], a[116:119], v[196:199]\n" \
  "buffer_load_dwordx4 %[vd0], %[wr], %[sw] offen offset:0 lds\n" \
  "v_mfma_f32_16x16x32_bf16 v[200:203], v[80:83], a[120:123], v[200:203]\n" \
  "ds_read_b64_tr_b16 v[74:75], %[va2] offset:512\n" \
  "v_mfma_f32_16x16x32_bf16 v[204:207], v[80:83], a[124:127], v[204:207]\n" \
  "s_waitcnt lgkmcnt(10)\n" \
  "v_mfma_f32_16x16x32_bf16 v[220:223], v[84:87], a[124:127], v[220:223]\n" \
  "ds_read_b64_tr_b16 v[76:77], %[va3] offset:0\n" \
  "v_mfma_f32_16x16x32_bf16 v[216:219], v[84:87], a[120:123], v[216:219]\n" \
  "v_mfma_f32_16x16x32_bf16 v[212:215], v[84:87], a[116:119], v[212:215]\n" \
  "ds_read_b64_tr_b16 v[78:79], %[va3] offset:512\n" \
  "v_mfma_f32_16x16x32_bf16 v[208:211], v[84:87], a[112:115], v[208:211]\n" \
  "s_waitcnt lgkmcnt(8)\n" \
  "v_mfma_f32_16x16x32_bf16 v[224:227], v[88:91], a[112:115], v[224:227]\n" \
  "ds_read_b64_tr_b16 v[80:81], %[va0] offset:8192\n" \
  "v_mfma_f32_16x16x32_bf16 v[228:231], v[88:91], a[116:119], v[228:231]\n" \
  "buffer_load_dwordx4 %[vd1], %[wr], %[sw] offen offset:1024 lds\n" \
  "v_mfma_f32_16x16x32_bf16 v[232:235], v[88:91], a[120:123], v[232:235]\n" \
  "ds_read_b64_tr_b16 v[82:83], %[va0] offset:8704\n" \
  "s_add_u32 %[sl], %[sl], 16\n" \
  "v_mfma_f32_16x16x32_bf16 v[236:239], v[88:91], a[124:127], v[236:239]\n" \
  "s_cmp_lg_u32 %[sl], 240\n" \
  "s_cbranch_scc1 .LloopB_%=\n" \
  "s_branch .Lfin_%=\n" \
  ".Lslow10_%=:\n" \
  "s_load_dword s64, %[fp], %[sl] offset:0x4 glc\n" \
  "s_memrealtime s[60:61]\n" \
  "s_waitcnt lgkmcnt(0)\n" \
  "s_cmp_ge_u32 s64, %[tag]\n" \
  "s_cbranch_scc1 .Lback10_%=\n" \
  "s_sub_u32 s60, s60, %[t0]\n" \
  "s_cmp_gt_u32 s60, %[dl]\n" \
  "s_cbranch_scc0 .Lslow10_%=\n" \
  "s_mov_b32 %[call], 0\n" \
  "s_branch .Lback10_%=\n" \
  ".Lslow11_%=:\n" \
  "s_load_dword s65, %[fp], %[sl] offset:0x804 glc\n" \
  "s_memrealtime s[60:61]\n" \
  "s_waitcnt lgkmcnt(0)\n" \
  "s_cmp_ge_u32 s65, %[tag]\n" \
  "s_cbranch_scc1 .Lback11_%=\n" \
  "s_sub_u32 s60, s60, %[t0]\n" \
  "s_cmp_gt_u32 s60, %[dl]\n" \
  "s_cbranch_scc0 .Lslow11_%=\n" \
  "s_mov_b32 %[call], 0\n" \
  "s_branch .Lback11_%=\n" \
  ".Lslow12_%=:\n" \
  "s_load_dword s64, %[fp], %[sl] offset:0x8 glc\n" \
  "s_memrealtime s[60:61]\n" \
  "s_waitcnt lgkmcnt(0)\n" \
  "s_cmp_ge_u32 s64, %[tag]\n" \
  "s_cbranch_scc1 .Lback12_%=\n" \
  "s_sub_u32 s60, s60, %[t0]\n" \
  "s_cmp_gt_u32 s60, %[dl]\n" \
  "s_cbranch_scc0 .Lslow12_%=\n" \
  "s_mov_b32 %[call], 0\n" \
  "s_branch .Lback12_%=\n" \
  ".Lslow13_%=:\n" \
  "s_load_dword s65, %[fp], %[sl] offset:0x808 glc\n" \
  "s_memrealtime s[60:61]\n" \
  "s_waitcnt lgkmcnt(0)\n" \
  "s_cmp_ge_u32 s65, %[tag]\n" \
  "s_cbranch_scc1 .Lback13_%=\n" \
  "s_sub_u32 s60, s60, %[t0]\n" \
  "s_cmp_gt_u32 s60, %[dl]\n" \
  "s_cbranch_scc0 .Lslow13_%=\n" \
  "s_mov_b32 %[call], 0\n" \
  "s_branch .Lback13_%=\n" \
  ".Lslow14_%=:\n" \
  "s_load_dword s64, %[fp], %[sl] offset:0xc glc\n" \
  "s_memrealtime s[60:61]\n" \
  "s_waitcnt lgkmcnt(0)\n" \
  "s_cmp_ge_u32 s64, %[tag]\n" \
  "s_cbranch_scc1 .Lback14_%=\n" \
  "s_sub_u32 s60, s60, %[t0]\n" \
  "s_cmp_gt_u32 s60, %[dl]\n" \
  "s_cbranch_scc0 .Lslow14_%=\n" \
  "s_mov_b32 %[call], 0\n" \
  "s_branch .Lback14_%=\n" \
  ".Lslow15_%=:\n" \
  "s_load_dword s65, %[fp], %[sl] offset:0x80c glc\n" \
  "s_memrealtime s[60:61]\n" \
  "s_waitcnt lgkmcnt(0)\n" \
  "s_cmp_ge_u32 s65, %[tag]\n" \
  "s_cbranch_scc1 .Lback15_%=\n" \
  "s_sub_u32 s60, s60, %[t0]\n" \
  "s_cmp_gt_u32 s60, %[dl]\n" \
  "s_cbranch_scc0 .Lslow15_%=\n" \
  "s_mov_b32 %[call], 0\n" \
  "s_branch .Lback15_%=\n" \
  ".Lend_%=:\n" \
  ""

#define HEAD 2
#define PULL_ASM \
  "s_mov_b32 s66, %[u0]\n" \
  "s_add_u32 s67, %[u0], 2\n" \
  "v_mov_b32 v57, 0\n" \
  "v_mov_b32 v54, 1\n" \
  "s_nop 1\n" \
  "s_lshl_b32 s80, s66, 3\n" \
  "s_load_dwordx2 s[78:79], %[pp], s80 glc\n" \
  "s_mov_b64 exec, 1\n" \
  "global_atomic_add v55, v57, v54, %[tk] sc0 sc1\n" \
  "s_mov_b64 exec, -1\n" \
  "v_mov_b32 v96, 0\n" \
  "v_mov_b32 v97, 0\n" \
  "v_mov_b32 v98, 0\n" \
  "v_mov_b32 v99, 0\n" \
  "ds_write_b128 %[vpl], v[96:99]\n" \
  "ds_write_b128 %[vpl], v[96:99] offset:1024\n" \
  "s_mov_b32 s72, 0x83838383\n" \
  "s_mov_b32 s73, 0x03030303\n" \
  "s_mov_b32 s74, 0x05010400\n" \
  "s_mov_b32 s75, 0x07030602\n" \
  "s_add_u32 s81, %[u0], 4\n" \
  "s_waitcnt lgkmcnt(0)\n" \
  "s_cmp_ge_u32 s78, %[call]\n" \
  "s_cbranch_scc0 .Lps0_%=\n" \
  ".Lpb0_%=:\n" \
  "s_mov_b32 s62, s79\n" \
  "s_lshl_b32 s80, s67, 3\n" \
  "s_load_dwordx2 s[78:79], %[pp], s80 glc\n" \
  "s_mul_i32 s68, s66, 3072\n" \
  "s_lshr_b32 s64, s62, 16\n" \
  "s_cmp_gt_u32 s64, 63\n" \
  "s_cbranch_scc1 .Lpr1_%=\n" \
  "buffer_load_dwordx4 v[64:67], %[v16], %[rpk], s68 offen sc0 sc1\n" \
  "buffer_load_dwordx4 v[68:71], %[v16], %[rpk], s68 offen offset:1024 sc0 sc1\n" \
  "buffer_load_dwordx3 v[72:74], %[v12], %[rpk], s68 offen offset:2048 sc0 sc1\n" \
  "s_bfm_b64 s[76:77], s64, 0\n" \
  "s_or_b64 exec, s[76:77], 1\n" \
  "buffer_load_ushort v75, %[v2], %[rpk], s68 offen offset:2816 sc0 sc1\n" \
  "s_mov_b64 exec, -1\n" \
  ".Lpd1_%=:\n" \
  "s_waitcnt lgkmcnt(0)\n" \
  "s_cmp_ge_u32 s78, %[call]\n" \
  "s_cbranch_scc0 .Lps2_%=\n" \
  ".Lpb2_%=:\n" \
  "s_mov_b32 s63, s79\n" \
  "s_lshl_b32 s80, s81, 3\n" \
  "s_load_dwordx2 s[78:79], %[pp], s80 glc\n" \
  "s_mul_i32 s68, s67, 3072\n" \
  "s_lshr_b32 s64, s63, 16\n" \
  "s_cmp_gt_u32 s64, 63\n" \
  "s_cbranch_scc1 .Lpr3_%=\n" \
  "buffer_load_dwordx4 v[80:83], %[v16], %[rpk], s68 offen sc0 sc1\n" \
  "buffer_load_dwordx4 v[84:87], %[v16], %[rpk], s68 offen offset:1024 sc0 sc1\n" \
  "buffer_load_dwordx3 v[88:90], %[v12], %[rpk], s68 offen offset:2048 sc0 sc1\n" \
  "s_bfm_b64 s[76:77], s64, 0\n" \
  "s_or_b64 exec, s[76:77], 1\n" \
  "buffer_load_ushort v91, %[v2], %[rpk], s68 offen offset:2816 sc0 sc1\n" \
  "s_mov_b64 exec, -1\n" \
  ".Lpd3_%=:\n" \
  "s_waitcnt vmcnt(4)\n" \
  "s_lshr_b32 s64, s62, 16\n" \
  "s_cmp_gt_u32 s64, 63\n" \
  "s_cbranch_scc1 .Ldr5_%=\n" \
  "v_lshrrev_b32 v58, 5, v75\n" \
  "v_lshlrev_b32 v59, 2, v75\n" \
  "s_bfm_b64 s[76:77], s64, 0\n" \
  "v_add_u32 v58, %[spl], v58\n" \
  "v_and_b32 v59, 0x7c, v59\n" \
  "s_mov_b64 exec, s[76:77]\n" \
  "ds_write_b8 v58, v59\n" \
  "s_mov_b64 exec, -1\n" \
  "ds_read_b128 v[96:99], %[vpl]\n" \
  "ds_read_b128 v[100:103], %[vpl] offset:1024\n" \
  "s_mov_b64 exec, s[76:77]\n" \
  "ds_write_b8 v58, v57\n" \
  "s_mov_b64 exec, -1\n" \
  "s_pack_ll_b32_b16 s65, s62, s62\n" \
  "v_alignbit_b32 v121, v72, v72, 3\n" \
  "v_alignbit_b32 v123, v73, v73, 3\n" \
  "v_alignbit_b32 v125, v74, v74, 3\n" \
  "v_lshrrev_b32 v126, 5, v72\n" \
  "v_lshrrev_b32 v127, 5, v73\n" \
  "v_lshlrev_b32 v120, 2, v74\n" \
  "v_lshlrev_b32 v122, 1, v74\n" \
  "v_bfi_b32 v126, s73, v126, v120\n" \
  "v_bfi_b32 v127, s73, v127, v122\n" \
  "s_waitcnt lgkmcnt(0)\n" \
  "v_and_or_b32 v120, v72, s72, v96\n" \
  "v_and_or_b32 v121, v121, s72, v97\n" \
  "v_perm_b32 v104, v120, v64, s74\n" \
  "v_perm_b32 v105, v120, v64, s75\n" \
  "v_perm_b32 v106, v121, v65, s74\n" \
  "v_perm_b32 v107, v121, v65, s75\n" \
  "v_pk_add_u16 v104, v104, s65\n" \
  "v_pk_add_u16 v105, v105, s65\n" \
  "v_pk_add_u16 v106, v106, s65\n" \
  "v_pk_add_u16 v107, v107, s65\n" \
  "v_and_or_b32 v122, v73, s72, v98\n" \
  "v_and_or_b32 v123, v123, s72, v99\n" \
  "v_perm_b32 v108, v122, v66, s74\n" \
  "v_perm_b32 v109, v122, v66, s75\n" \
  "v_perm_b32 v110, v123, v67, s74\n" \
  "v_perm_b32 v111, v123, v67, s75\n" \
  "v_pk_add_u16 v108, v108, s65\n" \
  "v_pk_add_u16 v109, v109, s65\n" \
  "v_pk_add_u16 v110, v110, s65\n" \
  "v_pk_add_u16 v111, v111, s65\n" \
  "v_and_or_b32 v124, v74, s72, v100\n" \
  "v_and_or_b32 v125, v125, s72, v101\n" \
  "v_perm_b32 v112, v124, v68, s74\n" \
  "v_perm_b32 v113, v124, v68, s75\n" \
  "v_perm_b32 v114, v125, v69, s74\n" \
  "v_perm_b32 v115, v125, v69, s75\n" \
  "v_pk_add_u16 v112, v112, s65\n" \
  "v_pk_add_u16 v113, v113, s65\n" \
  "v_pk_add_u16 v114, v114, s65\n" \
  "v_pk_add_u16 v115, v115, s65\n" \
  "v_and_or_b32 v126, v126, s72, v102\n" \
  "v_and_or_b32 v127, v127, s72, v103\n" \
  "v_perm_b32 v116, v126, v70, s74\n" \
  "v_perm_b32 v117, v126, v70, s75\n" \
  "v_perm_b32 v118, v127, v71, s74\n" \
  "v_perm_b32 v119, v127, v71, s75\n" \
  "v_pk_add_u16 v116, v116, s65\n" \
  "v_pk_add_u16 v117, v117, s65\n" \
  "v_pk_add_u16 v118, v118, s65\n" \
  "v_pk_add_u16 v119, v119, s65\n" \
  ".Ldd5_%=:\n" \
  "v_readfirstlane_b32 s69, v55\n" \
  "s_lshl_b32 s70, s66, 12\n" \
  "s_and_b32 s60, s66, 127\n" \
  "s_add_u32 s61, s60, %[tag]\n" \
  "v_mov_b32 v56, s61\n" \
  "s_lshr_b32 s71, s66, 7\n" \
  "s_lshl_b32 s71, s71, 1\n" \
  "s_and_b32 s61, s60, 1\n" \
  "s_or_b32 s71, s71, s61\n" \
  "s_lshl_b32 s71, s71, 11\n" \
  "s_lshr_b32 s60, s60, 1\n" \
  "s_lshl2_add_u32 s71, s60, s71\n" \
  "s_cmp_ge_u32 s81, 1024\n" \
  "s_cbranch_scc1 .Lt4_%=\n" \
  "s_cmp_ge_u32 s78, %[call]\n" \
  "s_cbranch_scc0 .Lps6_%=\n" \
  ".Lpb6_%=:\n" \
  "s_mov_b32 s62, s79\n" \
  "s_mov_b32 s66, s81\n" \
  "s_sub_u32 s69, s69, %[base]\n" \
  "s_and_b32 s81, s69, 7\n" \
  "s_lshl_b32 s81, s81, 7\n" \
  "s_lshr_b32 s60, s69, 3\n" \
  "s_add_u32 s81, s81, s60\n" \
  "s_add_u32 s81, s81, 2\n" \
  "s_cmp_lt_u32 s69, 1008\n" \
  "s_cselect_b32 s81, s81, 1024\n" \
  "s_cselect_b32 s60, 1, 0\n" \
  "v_mov_b32 v54, s60\n" \
  "s_lshl_b32 s80, s81, 3\n" \
  "s_load_dwordx2 s[78:79], %[pp], s80 glc\n" \
  "s_mov_b64 exec, 1\n" \
  "global_atomic_add v55, v57, v54, %[tk] sc0 sc1\n" \
  "s_mov_b64 exec, -1\n" \
  "buffer_store_dwordx4 v[104:107], %[v16], %[rdst], s70 offen offset:0 sc1\n" \
  "buffer_store_dwordx4 v[108:111], %[v16], %[rdst], s70 offen offset:1024 sc1\n" \
  "buffer_store_dwordx4 v[112:115], %[v16], %[rdst], s70 offen offset:2048 sc1\n" \
  "buffer_store_dwordx4 v[116:119], %[v16], %[rdst], s70 offen offset:3072 sc1\n" \
  "s_mul_i32 s68, s66, 3072\n" \
  "s_lshr_b32 s64, s62, 16\n" \
  "s_cmp_gt_u32 s64, 63\n" \
  "s_cbranch_scc1 .Lpr7_%=\n" \
  "buffer_load_dwordx4 v[64:67], %[v16], %[rpk], s68 offen sc0 sc1\n" \
  "buffer_load_dwordx4 v[68:71], %[v16], %[rpk], s68 offen offset:1024 sc0 sc1\n" \
  "buffer_load_dwordx3 v[72:74], %[v12], %[rpk], s68 offen offset:2048 sc0 sc1\n" \
  "s_bfm_b64 s[76:77], s64, 0\n" \
  "s_or_b64 exec, s[76:77], 1\n" \
  "buffer_load_ushort v75, %[v2], %[rpk], s68 offen offset:2816 sc0 sc1\n" \
  "s_mov_b64 exec, -1\n" \
  ".Lpd7_%=:\n" \
  "s_waitcnt vmcnt(9)\n" \
  ".Lploop_%=:\n" \
  "s_lshr_b32 s64, s63, 16\n" \
  "s_cmp_gt_u32 s64, 63\n" \
  "s_cbranch_scc1 .Ldr10_%=\n" \
  "v_lshrrev_b32 v58, 5, v91\n" \
  "v_lshlrev_b32 v59, 2, v91\n" \
  "s_bfm_b64 s[76:77], s64, 0\n" \
  "v_add_u32 v58, %[spl], v58\n" \
  "v_and_b32 v59, 0x7c, v59\n" \
  "s_mov_b64 exec, s[76:77]\n" \
  "ds_write_b8 v58, v59\n" \
  "s_mov_b64 exec, -1\n" \
  "ds_read_b128 v[96:99], %[vpl]\n" \
  "ds_read_b128 v[100:103], %[vpl] offset:1024\n" \
  "s_mov_b64 exec, s[76:77]\n" \
  "ds_write_b8 v58, v57\n" \
  "s_mov_b64 exec, -1\n" \
  "s_pack_ll_b32_b16 s65, s63, s63\n" \
  "v_alignbit_b32 v121, v88, v88, 3\n" \
  "v_alignbit_b32 v123, v89, v89, 3\n" \
  "v_alignbit_b32 v125, v90, v90, 3\n" \
  "v_lshrrev_b32 v126, 5, v88\n" \
  "v_lshrrev_b32 v127, 5, v89\n" \
  "v_lshlrev_b32 v120, 2, v90\n" \
  "v_lshlrev_b32 v122, 1, v90\n" \
  "v_bfi_b32 v126, s73, v126, v120\n" \
  "v_bfi_b32 v127, s73, v127, v122\n" \
  "s_waitcnt lgkmcnt(0)\n" \
  "v_and_or_b32 v120, v88, s72, v96\n" \
  "v_and_or_b32 v121, v121, s72, v97\n" \
  "v_perm_b32 v104, v120, v80, s74\n" \
  "v_perm_b32 v105, v120, v80, s75\n" \
  "v_perm_b32 v106, v121, v81, s74\n" \
  "v_perm_b32 v107, v121, v81, s75\n" \
  "v_pk_add_u16 v104, v104, s65\n" \
  "v_pk_add_u16 v105, v105, s65\n" \
  "v_pk_add_u16 v106, v106, s65\n" \
  "v_pk_add_u16 v107, v107, s65\n" \
  "v_and_or_b32 v122, v89, s72, v98\n" \
  "v_and_or_b32 v123, v123, s72, v99\n" \
  "v_perm_b32 v108, v122, v82, s74\n" \
  "v_perm_b32 v109, v122, v82, s75\n" \
  "v_perm_b32 v110, v123, v83, s74\n" \
  "v_perm_b32 v111, v123, v83, s75\n" \
  "v_pk_add_u16 v108, v108, s65\n" \
  "v_pk_add_u16 v109, v109, s65\n" \
  "v_pk_add_u16 v110, v110, s65\n" \
  "v_pk_add_u16 v111, v111, s65\n" \
  "v_and_or_b32 v124, v90, s72, v100\n" \
  "v_and_or_b32 v125, v125, s72, v101\n" \
  "v_perm_b32 v112, v124, v84, s74\n" \
  "v_perm_b32 v113, v124, v84, s75\n" \
  "v_perm_b32 v114, v125, v85, s74\n" \
  "v_perm_b32 v115, v125, v85, s75\n" \
  "v_pk_add_u16 v112, v112, s65\n" \
  "v_pk_add_u16 v113, v113, s65\n" \
  "v_pk_add_u16 v114, v114, s65\n" \
  "v_pk_add_u16 v115, v115, s65\n" \
  "v_and_or_b32 v126, v126, s72, v102\n" \
  "v_and_or_b32 v127, v127, s72, v103\n" \
  "v_perm_b32 v116, v126, v86, s74\n" \
  "v_perm_b32 v117, v126, v86, s75\n" \
  "v_perm_b32 v118, v127, v87, s74\n" \
  "v_perm_b32 v119, v127, v87, s75\n" \
  "v_pk_add_u16 v116, v116, s65\n" \
  "v_pk_add_u16 v117, v117, s65\n" \
  "v_pk_add_u16 v118, v118, s65\n" \
  "v_pk_add_u16 v119, v119, s65\n" \
  ".Ldd10_%=:\n" \
  "s_waitcnt vmcnt(4)\n" \
  "v_readfirstlane_b32 s69, v55\n" \
  "s_mov_b64 exec, %[fex]\n" \
  "buffer_store_dword v56, %[v4], %[rpg], s71 offen sc1\n" \
  "s_mov_b64 exec, -1\n" \
  "s_lshl_b32 s70, s67, 12\n" \
  "s_and_b32 s60, s67, 127\n" \
  "s_add_u32 s61, s60, %[tag]\n" \
  "v_mov_b32 v56, s61\n" \
  "s_lshr_b32 s71, s67, 7\n" \
  "s_lshl_b32 s71, s71, 1\n" \
  "s_and_b32 s61, s60, 1\n" \
  "s_or_b32 s71, s71, s61\n" \
  "s_lshl_b32 s71, s71, 11\n" \
  "s_lshr_b32 s60, s60, 1\n" \
  "s_lshl2_add_u32 s71, s60, s71\n" \
  "s_cmp_ge_u32 s81, 1024\n" \
  "s_cbranch_scc1 .Lt9_%=\n" \
  "s_cmp_ge_u32 s78, %[call]\n" \
  "s_cbranch_scc0 .Lps11_%=\n" \
  ".Lpb11_%=:\n" \
  "s_mov_b32 s63, s79\n" \
  "s_mov_b32 s67, s81\n" \
  "s_sub_u32 s69, s69, %[base]\n" \
  "s_and_b32 s81, s69, 7\n" \
  "s_lshl_b32 s81, s81, 7\n" \
  "s_lshr_b32 s60, s69, 3\n" \
  "s_add_u32 s81, s81, s60\n" \
  "s_add_u32 s81, s81, 2\n" \
  "s_cmp_lt_u32 s69, 1008\n" \
  "s_cselect_b32 s81, s81, 1024\n" \
  "s_cselect_b32 s60, 1, 0\n" \
  "v_mov_b32 v54, s60\n" \
  "s_lshl_b32 s80, s81, 3\n" \
  "s_load_dwordx2 s[78:79], %[pp], s80 glc\n" \
  "s_mov_b64 exec, 1\n" \
  "global_atomic_add v55, v57, v54, %[tk] sc0 sc1\n" \
  "s_mov_b64 exec, -1\n" \
  "buffer_store_dwordx4 v[104:107], %[v16], %[rdst], s70 offen offset:0 sc1\n" \
  "buffer_store_dwordx4 v[108:111], %[v16], %[rdst], s70 offen offset:1024 sc1\n" \
  "buffer_store_dwordx4 v[112:115], %[v16], %[rdst], s70 offen offset:2048 sc1\n" \
  "buffer_store_dwordx4 v[116:119], %[v16], %[rdst], s70 offen offset:3072 sc1\n" \
  "s_mul_i32 s68, s67, 3072\n" \
  "s_lshr_b32 s64, s63, 16\n" \
  "s_cmp_gt_u32 s64, 63\n" \
  "s_cbranch_scc1 .Lpr12_%=\n" \
  "buffer_load_dwordx4 v[80:83], %[v16], %[rpk], s68 offen sc0 sc1\n" \
  "buffer_load_dwordx4 v[84:87], %[v16], %[rpk], s68 offen offset:1024 sc0 sc1\n" \
  "buffer_load_dwordx3 v[88:90], %[v12], %[rpk], s68 offen offset:2048 sc0 sc1\n" \
  "s_bfm_b64 s[76:77], s64, 0\n" \
  "s_or_b64 exec, s[76:77], 1\n" \
  "buffer_load_ushort v91, %[v2], %[rpk], s68 offen offset:2816 sc0 sc1\n" \
  "s_mov_b64 exec, -1\n" \
  ".Lpd12_%=:\n" \
  "s_waitcnt vmcnt(10)\n" \
  "s_lshr_b32 s64, s62, 16\n" \
  "s_cmp_gt_u32 s64, 63\n" \
  "s_cbranch_scc1 .Ldr15_%=\n" \
  "v_lshrrev_b32 v58, 5, v75\n" \
  "v_lshlrev_b32 v59, 2, v75\n" \
  "s_bfm_b64 s[76:77], s64, 0\n" \
  "v_add_u32 v58, %[spl], v58\n" \
  "v_and_b32 v59, 0x7c, v59\n" \
  "s_mov_b64 exec, s[76:77]\n" \
  "ds_write_b8 v58, v59\n" \
  "s_mov_b64 exec, -1\n" \
  "ds_read_b128 v[96:99], %[vpl]\n" \
  "ds_read_b128 v[100:103], %[vpl] offset:1024\n" \
  "s_mov_b64 exec, s[76:77]\n" \
  "ds_write_b8 v58, v57\n" \
  "s_mov_b64 exec, -1\n" \
  "s_pack_ll_b32_b16 s65, s62, s62\n" \
  "v_alignbit_b32 v121, v72, v72, 3\n" \
  "v_alignbit_b32 v123, v73, v73, 3\n" \
  "v_alignbit_b32 v125, v74, v74, 3\n" \
  "v_lshrrev_b32 v126, 5, v72\n" \
  "v_lshrrev_b32 v127, 5, v73\n" \
  "v_lshlrev_b32 v120, 2, v74\n" \
  "v_lshlrev_b32 v122, 1, v74\n" \
  "v_bfi_b32 v126, s73, v126, v120\n" \
  "v_bfi_b32 v127, s73, v127, v122\n" \
  "s_waitcnt lgkmcnt(0)\n" \
  "v_and_or_b32 v120, v72, s72, v96\n" \
  "v_and_or_b32 v121, v121, s72, v97\n" \
  "v_perm_b32 v104, v120, v64, s74\n" \
  "v_perm_b32 v105, v120, v64, s75\n" \
  "v_perm_b32 v106, v121, v65, s74\n" \
  "v_perm_b32 v107, v121, v65, s75\n" \
  "v_pk_add_u16 v104, v104, s65\n" \
  "v_pk_add_u16 v105, v105, s65\n" \
  "v_pk_add_u16 v106, v106, s65\n" \
  "v_pk_add_u16 v107, v107, s65\n" \
  "v_and_or_b32 v122, v73, s72, v98\n" \
  "v_and_or_b32 v123, v123, s72, v99\n" \
  "v_perm_b32 v108, v122, v66, s74\n" \
  "v_perm_b32 v109, v122, v66, s75\n" \
  "v_perm_b32 v110, v123, v67, s74\n" \
  "v_perm_b32 v111, v123, v67, s75\n" \
  "v_pk_add_u16 v108, v108, s65\n" \
  "v_pk_add_u16 v109, v109, s65\n" \
  "v_pk_add_u16 v110, v110, s65\n" \
  "v_pk_add_u16 v111, v111, s65\n" \
  "v_and_or_b32 v124, v74, s72, v100\n" \
  "v_and_or_b32 v125, v125, s72, v101\n" \
  "v_perm_b32 v112, v124, v68, s74\n" \
  "v_perm_b32 v113, v124, v68, s75\n" \
  "v_perm_b32 v114, v125, v69, s74\n" \
  "v_perm_b32 v115, v125, v69, s75\n" \
  "v_pk_add_u16 v112, v112, s65\n" \
  "v_pk_add_u16 v113, v113, s65\n" \
  "v_pk_add_u16 v114, v114, s65\n" \
  "v_pk_add_u16 v115, v115, s65\n" \
  "v_and_or_b32 v126, v126, s72, v102\n" \
  "v_and_or_b32 v127, v127, s72, v103\n" \
  "v_perm_b32 v116, v126, v70, s74\n" \
  "v_perm_b32 v117, v126, v70, s75\n" \
  "v_perm_b32 v118, v127, v71, s74\n" \
  "v_perm_b32 v119, v127, v71, s75\n" \
  "v_pk_add_u16 v116, v116, s65\n" \
  "v_pk_add_u16 v117, v117, s65\n" \
  "v_pk_add_u16 v118, v118, s65\n" \
  "v_pk_add_u16 v119, v119, s65\n" \
  ".Ldd15_%=:\n" \
  "s_waitcnt vmcnt(4)\n" \
  "v_readfirstlane_b32 s69, v55\n" \
  "s_mov_b64 exec, %[fex]\n" \
  "buffer_store_dword v56, %[v4], %[rpg], s71 offen sc1\n" \
  "s_mov_b64 exec, -1\n" \
  "s_lshl_b32 s70, s66, 12\n" \
  "s_and_b32 s60, s66, 127\n" \
  "s_add_u32 s61, s60, %[tag]\n" \
  "v_mov_b32 v56, s61\n" \
  "s_lshr_b32 s71, s66, 7\n" \
  "s_lshl_b32 s71, s71, 1\n" \
  "s_and_b32 s61, s60, 1\n" \
  "s_or_b32 s71, s71, s61\n" \
  "s_lshl_b32 s71, s71, 11\n" \
  "s_lshr_b32 s60, s60, 1\n" \
  "s_lshl2_add_u32 s71, s60, s71\n" \
  "s_cmp_ge_u32 s81, 1024\n" \
  "s_cbranch_scc1 .Lt14_%=\n" \
  "s_cmp_ge_u32 s78, %[call]\n" \
  "s_cbranch_scc0 .Lps16_%=\n" \
  ".Lpb16_%=:\n" \
  "s_mov_b32 s62, s79\n" \
  "s_mov_b32 s66, s81\n" \
  "s_sub_u32 s69, s69, %[base]\n" \
  "s_and_b32 s81, s69, 7\n" \
  "s_lshl_b32 s81, s81, 7\n" \
  "s_lshr_b32 s60, s69, 3\n" \
  "s_add_u32 s81, s81, s60\n" \
  "s_add_u32 s81, s81, 2\n" \
  "s_cmp_lt_u32 s69, 1008\n" \
  "s_cselect_b32 s81, s81, 1024\n" \
  "s_cselect_b32 s60, 1, 0\n" \
  "v_mov_b32 v54, s60\n" \
  "s_lshl_b32 s80, s81, 3\n" \
  "s_load_dwordx2 s[78:79], %[pp], s80 glc\n" \
  "s_mov_b64 exec, 1\n" \
  "global_atomic_add v55, v57, v54, %[tk] sc0 sc1\n" \
  "s_mov_b64 exec, -1\n" \
  "buffer_store_dwordx4 v[104:107], %[v16], %[rdst], s70 offen offset:0 sc1\n" \
  "buffer_store_dwordx4 v[108:111], %[v16], %[rdst], s70 offen offset:1024 sc1\n" \
  "buffer_store_dwordx4 v[112:115], %[v16], %[rdst], s70 offen offset:2048 sc1\n" \
  "buffer_store_dwordx4 v[116:119], %[v16], %[rdst], s70 offen offset:3072 sc1\n" \
  "s_mul_i32 s68, s66, 3072\n" \
  "s_lshr_b32 s64, s62, 16\n" \
  "s_cmp_gt_u32 s64, 63\n" \
  "s_cbranch_scc1 .Lpr17_%=\n" \
  "buffer_load_dwordx4 v[64:67], %[v16], %[rpk], s68 offen sc0 sc1\n" \
  "buffer_load_dwordx4 v[68:71], %[v16], %[rpk], s68 offen offset:1024 sc0 sc1\n" \
  "buffer_load_dwordx3 v[72:74], %[v12], %[rpk], s68 offen offset:2048 sc0 sc1\n" \
  "s_bfm_b64 s[76:77], s64, 0\n" \
  "s_or_b64 exec, s[76:77], 1\n" \
  "buffer_load_ushort v75, %[v2], %[rpk], s68 offen offset:2816 sc0 sc1\n" \
  "s_mov_b64 exec, -1\n" \
  ".Lpd17_%=:\n" \
  "s_waitcnt vmcnt(10)\n" \
  "s_branch .Lploop_%=\n" \
  ".Lt4_%=:\n" \
  "buffer_store_dwordx4 v[104:107], %[v16], %[rdst], s70 offen offset:0 sc1\n" \
  "buffer_store_dwordx4 v[108:111], %[v16], %[rdst], s70 offen offset:1024 sc1\n" \
  "buffer_store_dwordx4 v[112:115], %[v16], %[rdst], s70 offen offset:2048 sc1\n" \
  "buffer_store_dwordx4 v[116:119], %[v16], %[rdst], s70 offen offset:3072 sc1\n" \
  "s_waitcnt vmcnt(4)\n" \
  "s_lshr_b32 s64, s63, 16\n" \
  "s_cmp_gt_u32 s64, 63\n" \
  "s_cbranch_scc1 .Ldr8_%=\n" \
  "v_lshrrev_b32 v58, 5, v91\n" \
  "v_lshlrev_b32 v59, 2, v91\n" \
  "s_bfm_b64 s[76:77], s64, 0\n" \
  "v_add_u32 v58, %[spl], v58\n" \
  "v_and_b32 v59, 0x7c, v59\n" \
  "s_mov_b64 exec, s[76:77]\n" \
  "ds_write_b8 v58, v59\n" \
  "s_mov_b64 exec, -1\n" \
  "ds_read_b128 v[96:99], %[vpl]\n" \
  "ds_read_b128 v[100:103], %[vpl] offset:1024\n" \
  "s_mov_b64 exec, s[76:77]\n" \
  "ds_write_b8 v58, v57\n" \
  "s_mov_b64 exec, -1\n" \
  "s_pack_ll_b32_b16 s65, s63, s63\n" \
  "v_alignbit_b32 v121, v88, v88, 3\n" \
  "v_alignbit_b32 v123, v89, v89, 3\n" \
  "v_alignbit_b32 v125, v90, v90, 3\n" \
  "v_lshrrev_b32 v126, 5, v88\n" \
  "v_lshrrev_b32 v127, 5, v89\n" \
  "v_lshlrev_b32 v120, 2, v90\n" \
  "v_lshlrev_b32 v122, 1, v90\n" \
  "v_bfi_b32 v126, s73, v126, v120\n" \
  "v_bfi_b32 v127, s73, v127, v122\n" \
  "s_waitcnt lgkmcnt(0)\n" \
  "v_and_or_b32 v120, v88, s72, v96\n" \
  "v_and_or_b32 v121, v121, s72, v97\n" \
  "v_perm_b32 v104, v120, v80, s74\n" \
  "v_perm_b32 v105, v120, v80, s75\n" \
  "v_perm_b32 v106, v121, v81, s74\n" \
  "v_perm_b32 v107, v121, v81, s75\n" \
  "v_pk_add_u16 v104, v104, s65\n" \
  "v_pk_add_u16 v105, v105, s65\n" \
  "v_pk_add_u16 v106, v106, s65\n" \
  "v_pk_add_u16 v107, v107, s65\n" \
  "v_and_or_b32 v122, v89, s72, v98\n" \
  "v_and_or_b32 v123, v123, s72, v99\n" \
  "v_perm_b32 v108, v122, v82, s74\n" \
  "v_perm_b32 v109, v122, v82, s75\n" \
  "v_perm_b32 v110, v123, v83, s74\n" \
  "v_perm_b32 v111, v123, v83, s75\n" \
  "v_pk_add_u16 v108, v108, s65\n" \
  "v_pk_add_u16 v109, v109, s65\n" \
  "v_pk_add_u16 v110, v110, s65\n" \
  "v_pk_add_u16 v111, v111, s65\n" \
  "v_and_or_b32 v124, v90, s72, v100\n" \
  "v_and_or_b32 v125, v125, s72, v101\n" \
  "v_perm_b32 v112, v124, v84, s74\n" \
  "v_perm_b32 v113, v124, v84, s75\n" \
  "v_perm_b32 v114, v125, v85, s74\n" \
  "v_perm_b32 v115, v125, v85, s75\n" \
  "v_pk_add_u16 v112, v112, s65\n" \
  "v_pk_add_u16 v113, v113, s65\n" \
  "v_pk_add_u16 v114, v114, s65\n" \
  "v_pk_add_u16 v115, v115, s65\n" \
  "v_and_or_b32 v126, v126, s72, v102\n" \
  "v_and_or_b32 v127, v127, s72, v103\n" \
  "v_perm_b32 v116, v126, v86, s74\n" \
  "v_perm_b32 v117, v126, v86, s75\n" \
  "v_perm_b32 v118, v127, v87, s74\n" \
  "v_perm_b32 v119, v127, v87, s75\n" \
  "v_pk_add_u16 v116, v116, s65\n" \
  "v_pk_add_u16 v117, v117, s65\n" \
  "v_pk_add_u16 v118, v118, s65\n" \
  "v_pk_add_u16 v119, v119, s65\n" \
  ".Ldd8_%=:\n" \
  "s_waitcnt vmcnt(0)\n" \
  "s_mov_b64 exec, %[fex]\n" \
  "buffer_store_dword v56, %[v4], %[rpg], s71 offen sc1\n" \
  "s_mov_b64 exec, -1\n" \
  "s_lshl_b32 s70, s67, 12\n" \
  "s_and_b32 s60, s67, 127\n" \
  "s_add_u32 s61, s60, %[tag]\n" \
  "v_mov_b32 v56, s61\n" \
  "s_lshr_b32 s71, s67, 7\n" \
  "s_lshl_b32 s71, s71, 1\n" \
  "s_and_b32 s61, s60, 1\n" \
  "s_or_b32 s71, s71, s61\n" \
  "s_lshl_b32 s71, s71, 11\n" \
  "s_lshr_b32 s60, s60, 1\n" \
  "s_lshl2_add_u32 s71, s60, s71\n" \
  "buffer_store_dwordx4 v[104:107], %[v16], %[rdst], s70 offen offset:0 sc1\n" \
  "buffer_store_dwordx4 v[108:111], %[v16], %[rdst], s70 offen offset:1024 sc1\n" \
  "buffer_store_dwordx4 v[112:115], %[v16], %[rdst], s70 offen offset:2048 sc1\n" \
  "buffer_store_dwordx4 v[116:119], %[v16], %[rdst], s70 offen offset:3072 sc1\n" \
  "s_waitcnt vmcnt(0)\n" \
  "s_mov_b64 exec, %[fex]\n" \
  "buffer_store_dword v56, %[v4], %[rpg], s71 offen sc1\n" \
  "s_mov_b64 exec, -1\n" \
  "s_branch .Lpend_%=\n" \
  ".Lt9_%=:\n" \
  "buffer_store_dwordx4 v[104:107], %[v16], %[rdst], s70 offen offset:0 sc1\n" \
  "buffer_store_dwordx4 v[108:111], %[v16], %[rdst], s70 offen offset:1024 sc1\n" \
  "buffer_store_dwordx4 v[112:115], %[v16], %[rdst], s70 offen offset:2048 sc1\n" \
  "buffer_store_dwordx4 v[116:119], %[v16], %[rdst], s70 offen offset:3072 sc1\n" \
  "s_waitcnt vmcnt(5)\n" \
  "s_lshr_b32 s64, s62, 16\n" \
  "s_cmp_gt_u32 s64, 63\n" \
  "s_cbranch_scc1 .Ldr13_%=\n" \
  "v_lshrrev_b32 v58, 5, v75\n" \
  "v_lshlrev_b32 v59, 2, v75\n" \
  "s_bfm_b64 s[76:77], s64, 0\n" \
  "v_add_u32 v58, %[spl], v58\n" \
  "v_and_b32 v59, 0x7c, v59\n" \
  "s_mov_b64 exec, s[76:77]\n" \
  "ds_write_b8 v58, v59\n" \
  "s_mov_b64 exec, -1\n" \
  "ds_read_b128 v[96:99], %[vpl]\n" \
  "ds_read_b128 v[100:103], %[vpl] offset:1024\n" \
  "s_mov_b64 exec, s[76:77]\n" \
  "ds_write_b8 v58, v57\n" \
  "s_mov_b64 exec, -1\n" \
  "s_pack_ll_b32_b16 s65, s62, s62\n" \
  "v_alignbit_b32 v121, v72, v72, 3\n" \
  "v_alignbit_b32 v123, v73, v73, 3\n" \
  "v_alignbit_b32 v125, v74, v74, 3\n" \
  "v_lshrrev_b32 v126, 5, v72\n" \
  "v_lshrrev_b32 v127, 5, v73\n" \
  "v_lshlrev_b32 v120, 2, v74\n" \
  "v_lshlrev_b32 v122, 1, v74\n" \
  "v_bfi_b32 v126, s73, v126, v120\n" \
  "v_bfi_b32 v127, s73, v127, v122\n" \
  "s_waitcnt lgkmcnt(0)\n" \
  "v_and_or_b32 v120, v72, s72, v96\n" \
  "v_and_or_b32 v121, v121, s72, v97\n" \
  "v_perm_b32 v104, v120, v64, s74\n" \
  "v_perm_b32 v105, v120, v64, s75\n" \
  "v_perm_b32 v106, v121, v65, s74\n" \
  "v_perm_b32 v107, v121, v65, s75\n" \
  "v_pk_add_u16 v104, v104, s65\n" \
  "v_pk_add_u16 v105, v105, s65\n" \
  "v_pk_add_u16 v106, v106, s65\n" \
  "v_pk_add_u16 v107, v107, s65\n" \
  "v_and_or_b32 v122, v73, s72, v98\n" \
  "v_and_or_b32 v123, v123, s72, v99\n" \
  "v_perm_b32 v108, v122, v66, s74\n" \
  "v_perm_b32 v109, v122, v66, s75\n" \
  "v_perm_b32 v110, v123, v67, s74\n" \
  "v_perm_b32 v111, v123, v67, s75\n" \
  "v_pk_add_u16 v108, v108, s65\n" \
  "v_pk_add_u16 v109, v109, s65\n" \
  "v_pk_add_u16 v110, v110, s65\n" \
  "v_pk_add_u16 v111, v111, s65\n" \
  "v_and_or_b32 v124, v74, s72, v100\n" \
  "v_and_or_b32 v125, v125, s72, v101\n" \
  "v_perm_b32 v112, v124, v68, s74\n" \
  "v_perm_b32 v113, v124, v68, s75\n" \
  "v_perm_b32 v114, v125, v69, s74\n" \
  "v_perm_b32 v115, v125, v69, s75\n" \
  "v_pk_add_u16 v112, v112, s65\n" \
  "v_pk_add_u16 v113, v113, s65\n" \
  "v_pk_add_u16 v114, v114, s65\n" \
  "v_pk_add_u16 v115, v115, s65\n" \
  "v_and_or_b32 v126, v126, s72, v102\n" \
  "v_and_or_b32 v127, v127, s72, v103\n" \
  "v_perm_b32 v116, v126, v70, s74\n" \
  "v_perm_b32 v117, v126, v70, s75\n" \
  "v_perm_b32 v118, v127, v71, s74\n" \
  "v_perm_b32 v119, v127, v71, s75\n" \
  "v_pk_add_u16 v116, v116, s65\n" \
  "v_pk_add_u16 v117, v117, s65\n" \
  "v_pk_add_u16 v118, v118, s65\n" \
  "v_pk_add_u16 v119, v119, s65\n" \
  ".Ldd13_%=:\n" \
  "s_waitcnt vmcnt(0)\n" \
  "s_mov_b64 exec, %[fex]\n" \
  "buffer_store_dword v56, %[v4], %[rpg], s71 offen sc1\n" \
  "s_mov_b64 exec, -1\n" \
  "s_lshl_b32 s70, s66, 12\n" \
  "s_and_b32 s60, s66, 127\n" \
  "s_add_u32 s61, s60, %[tag]\n" \
  "v_mov_b32 v56, s61\n" \
  "s_lshr_b32 s71, s66, 7\n" \
  "s_lshl_b32 s71, s71, 1\n" \
  "s_and_b32 s61, s60, 1\n" \
  "s_or_b32 s71, s71, s61\n" \
  "s_lshl_b32 s71, s71, 11\n" \
  "s_lshr_b32 s60, s60, 1\n" \
  "s_lshl2_add_u32 s71, s60, s71\n" \
  "buffer_store_dwordx4 v[104:107], %[v16], %[rdst], s70 offen offset:0 sc1\n" \
  "buffer_store_dwordx4 v[108:111], %[v16], %[rdst], s70 offen offset:1024 sc1\n" \
  "buffer_store_dwordx4 v[112:115], %[v16], %[rdst], s70 offen offset:2048 sc1\n" \
  "buffer_store_dwordx4 v[116:119], %[v16], %[rdst], s70 offen offset:3072 sc1\n" \
  "s_waitcnt vmcnt(0)\n" \
  "s_mov_b64 exec, %[fex]\n" \
  "buffer_store_dword v56, %[v4], %[rpg], s71 offen sc1\n" \
  "s_mov_b64 exec, -1\n" \
  "s_branch .Lpend_%=\n" \
  ".Lt14_%=:\n" \
  "buffer_store_dwordx4 v[104:107], %[v16], %[rdst], s70 offen offset:0 sc1\n" \
  "buffer_store_dwordx4 v[108:111], %[v16], %[rdst], s70 offen offset:1024 sc1\n" \
  "buffer_store_dwordx4 v[112:115], %[v16], %[rdst], s70 offen offset:2048 sc1\n" \
  "buffer_store_dwordx4 v[116:119], %[v16], %[rdst], s70 offen offset:3072 sc1\n" \
  "s_waitcnt vmcnt(5)\n" \
  "s_lshr_b32 s64, s63, 16\n" \
  "s_cmp_gt_u32 s64, 63\n" \
  "s_cbranch_scc1 .Ldr18_%=\n" \
  "v_lshrrev_b32 v58, 5, v91\n" \
  "v_lshlrev_b32 v59, 2, v91\n" \
  "s_bfm_b64 s[76:77], s64, 0\n" \
  "v_add_u32 v58, %[spl], v58\n" \
  "v_and_b32 v59, 0x7c, v59\n" \
  "s_mov_b64 exec, s[76:77]\n" \
  "ds_write_b8 v58, v59\n" \
  "s_mov_b64 exec, -1\n" \
  "ds_read_b128 v[96:99], %[vpl]\n" \
  "ds_read_b128 v[100:103], %[vpl] offset:1024\n" \
  "s_mov_b64 exec, s[76:77]\n" \
  "ds_write_b8 v58, v57\n" \
  "s_mov_b64 exec, -1\n" \
  "s_pack_ll_b32_b16 s65, s63, s63\n" \
  "v_alignbit_b32 v121, v88, v88, 3\n" \
  "v_alignbit_b32 v123, v89, v89, 3\n" \
  "v_alignbit_b32 v125, v90, v90, 3\n" \
  "v_lshrrev_b32 v126, 5, v88\n" \
  "v_lshrrev_b32 v127, 5, v89\n" \
  "v_lshlrev_b32 v120, 2, v90\n" \
  "v_lshlrev_b32 v122, 1, v90\n" \
  "v_bfi_b32 v126, s73, v126, v120\n" \
  "v_bfi_b32 v127, s73, v127, v122\n" \
  "s_waitcnt lgkmcnt(0)\n" \
  "v_and_or_b32 v120, v88, s72, v96\n" \
  "v_and_or_b32 v121, v121, s72, v97\n" \
  "v_perm_b32 v104, v120, v80, s74\n" \
  "v_perm_b32 v105, v120, v80, s75\n" \
  "v_perm_b32 v106, v121, v81, s74\n" \
  "v_perm_b32 v107, v121, v81, s75\n" \
  "v_pk_add_u16 v104, v104, s65\n" \
  "v_pk_add_u16 v105, v105, s65\n" \
  "v_pk_add_u16 v106, v106, s65\n" \
  "v_pk_add_u16 v107, v107, s65\n" \
  "v_and_or_b32 v122, v89, s72, v98\n" \
  "v_and_or_b32 v123, v123, s72, v99\n" \
  "v_perm_b32 v108, v122, v82, s74\n" \
  "v_perm_b32 v109, v122, v82, s75\n" \
  "v_perm_b32 v110, v123, v83, s74\n" \
  "v_perm_b32 v111, v123, v83, s75\n" \
  "v_pk_add_u16 v108, v108, s65\n" \
  "v_pk_add_u16 v109, v109, s65\n" \
  "v_pk_add_u16 v110, v110, s65\n" \
  "v_pk_add_u16 v111, v111, s65\n" \
  "v_and_or_b32 v124, v90, s72, v100\n" \
  "v_and_or_b32 v125, v125, s72, v101\n" \
  "v_perm_b32 v112, v124, v84, s74\n" \
  "v_perm_b32 v113, v124, v84, s75\n" \
  "v_perm_b32 v114, v125, v85, s74\n" \
  "v_perm_b32 v115, v125, v85, s75\n" \
  "v_pk_add_u16 v112, v112, s65\n" \
  "v_pk_add_u16 v113, v113, s65\n" \
  "v_pk_add_u16 v114, v114, s65\n" \
  "v_pk_add_u16 v115, v115, s65\n" \
  "v_and_or_b32 v126, v126, s72, v102\n" \
  "v_and_or_b32 v127, v127, s72, v103\n" \
  "v_perm_b32 v116, v126, v86, s74\n" \
  "v_perm_b32 v117, v126, v86, s75\n" \
  "v_perm_b32 v118, v127, v87, s74\n" \
  "v_perm_b32 v119, v127, v87, s75\n" \
  "v_pk_add_u16 v116, v116, s65\n" \
  "v_pk_add_u16 v117, v117, s65\n" \
  "v_pk_add_u16 v118, v118, s65\n" \
  "v_pk_add_u16 v119, v119, s65\n" \
  ".Ldd18_%=:\n" \
  "s_waitcnt vmcnt(0)\n" \
  "s_mov_b64 exec, %[fex]\n" \
  "buffer_store_dword v56, %[v4], %[rpg], s71 offen sc1\n" \
  "s_mov_b64 exec, -1\n" \
  "s_lshl_b32 s70, s67, 12\n" \
  "s_and_b32 s60, s67, 127\n" \
  "s_add_u32 s61, s60, %[tag]\n" \
  "v_mov_b32 v56, s61\n" \
  "s_lshr_b32 s71, s67, 7\n" \
  "s_lshl_b32 s71, s71, 1\n" \
  "s_and_b32 s61, s60, 1\n" \
  "s_or_b32 s71, s71, s61\n" \
  "s_lshl_b32 s71, s71, 11\n" \
  "s_lshr_b32 s60, s60, 1\n" \
  "s_lshl2_add_u32 s71, s60, s71\n" \
  "buffer_store_dwordx4 v[104:107], %[v16], %[rdst], s70 offen offset:0 sc1\n" \
  "buffer_store_dwordx4 v[108:111], %[v16], %[rdst], s70 offen offset:1024 sc1\n" \
  "buffer_store_dwordx4 v[112:115], %[v16], %[rdst], s70 offen offset:2048 sc1\n" \
  "buffer_store_dwordx4 v[116:119], %[v16], %[rdst], s70 offen offset:3072 sc1\n" \
  "s_waitcnt vmcnt(0)\n" \
  "s_mov_b64 exec, %[fex]\n" \
  "buffer_store_dword v56, %[v4], %[rpg], s71 offen sc1\n" \
  "s_mov_b64 exec, -1\n" \
  "s_branch .Lpend_%=\n" \
  ".Lps0_%=:\n" \
  "s_load_dwordx2 s[78:79], %[pp], s80 glc\n" \
  "s_memrealtime s[60:61]\n" \
  "s_waitcnt lgkmcnt(0)\n" \
  "s_cmp_ge_u32 s78, %[call]\n" \
  "s_cbranch_scc1 .Lpb0_%=\n" \
  "s_sub_u32 s60, s60, %[t0]\n" \
  "s_cmp_gt_u32 s60, %[dl]\n" \
  "s_cbranch_scc0 .Lps0_%=\n" \
  "s_branch .Lpend_%=\n" \
  ".Lpr1_%=:\n" \
  "s_lshl_b32 s68, s66, 12\n" \
  "buffer_load_dwordx4 v[64:67], %[v16], %[rraw], s68 offen offset:0 sc0 sc1\n" \
  "buffer_load_dwordx4 v[68:71], %[v16], %[rraw], s68 offen offset:1024 sc0 sc1\n" \
  "buffer_load_dwordx4 v[72:75], %[v16], %[rraw], s68 offen offset:2048 sc0 sc1\n" \
  "buffer_load_dwordx4 v[76:79], %[v16], %[rraw], s68 offen offset:3072 sc0 sc1\n" \
  "s_branch .Lpd1_%=\n" \
  ".Lps2_%=:\n" \
  "s_load_dwordx2 s[78:79], %[pp], s80 glc\n" \
  "s_memrealtime s[60:61]\n" \
  "s_waitcnt lgkmcnt(0)\n" \
  "s_cmp_ge_u32 s78, %[call]\n" \
  "s_cbranch_scc1 .Lpb2_%=\n" \
  "s_sub_u32 s60, s60, %[t0]\n" \
  "s_cmp_gt_u32 s60, %[dl]\n" \
  "s_cbranch_scc0 .Lps2_%=\n" \
  "s_branch .Lpend_%=\n" \
  ".Lpr3_%=:\n" \
  "s_lshl_b32 s68, s67, 12\n" \
  "buffer_load_dwordx4 v[80:83], %[v16], %[rraw], s68 offen offset:0 sc0 sc1\n" \
  "buffer_load_dwordx4 v[84:87], %[v16], %[rraw], s68 offen offset:1024 sc0 sc1\n" \
  "buffer_load_dwordx4 v[88:91], %[v16], %[rraw], s68 offen offset:2048 sc0 sc1\n" \
  "buffer_load_dwordx4 v[92:95], %[v16], %[rraw], s68 offen offset:3072 sc0 sc1\n" \
  "s_branch .Lpd3_%=\n" \
  ".Ldr5_%=:\n" \
  "s_waitcnt lgkmcnt(0)\n" \
  "v_mov_b32 v104, v64\n" \
  "v_mov_b32 v105, v65\n" \
  "v_mov_b32 v106, v66\n" \
  "v_mov_b32 v107, v67\n" \
  "v_mov_b32 v108, v68\n" \
  "v_mov_b32 v109, v69\n" \
  "v_mov_b32 v110, v70\n" \
  "v_mov_b32 v111, v71\n" \
  "v_mov_b32 v112, v72\n" \
  "v_mov_b32 v113, v73\n" \
  "v_mov_b32 v114, v74\n" \
  "v_mov_b32 v115, v75\n" \
  "v_mov_b32 v116, v76\n" \
  "v_mov_b32 v117, v77\n" \
  "v_mov_b32 v118, v78\n" \
  "v_mov_b32 v119, v79\n" \
  "s_branch .Ldd5_%=\n" \
  ".Lps6_%=:\n" \
  "s_load_dwordx2 s[78:79], %[pp], s80 glc\n" \
  "s_memrealtime s[60:61]\n" \
  "s_waitcnt lgkmcnt(0)\n" \
  "s_cmp_ge_u32 s78, %[call]\n" \
  "s_cbranch_scc1 .Lpb6_%=\n" \
  "s_sub_u32 s60, s60, %[t0]\n" \
  "s_cmp_gt_u32 s60, %[dl]\n" \
  "s_cbranch_scc0 .Lps6_%=\n" \
  "s_branch .Lpend_%=\n" \
  ".Lpr7_%=:\n" \
  "s_lshl_b32 s68, s66, 12\n" \
  "buffer_load_dwordx4 v[64:67], %[v16], %[rraw], s68 offen offset:0 sc0 sc1\n" \
  "buffer_load_dwordx4 v[68:71], %[v16], %[rraw], s68 offen offset:1024 sc0 sc1\n" \
  "buffer_load_dwordx4 v[72:75], %[v16], %[rraw], s68 offen offset:2048 sc0 sc1\n" \
  "buffer_load_dwordx4 v[76:79], %[v16], %[rraw], s68 offen offset:3072 sc0 sc1\n" \
  "s_branch .Lpd7_%=\n" \
  ".Ldr8_%=:\n" \
  "s_waitcnt lgkmcnt(0)\n" \
  "v_mov_b32 v104, v80\n" \
  "v_mov_b32 v105, v81\n" \
  "v_mov_b32 v106, v82\n" \
  "v_mov_b32 v107, v83\n" \
  "v_mov_b32 v108, v84\n" \
  "v_mov_b32 v109, v85\n" \
  "v_mov_b32 v110, v86\n" \
  "v_mov_b32 v111, v87\n" \
  "v_mov_b32 v112, v88\n" \
  "v_mov_b32 v113, v89\n" \
  "v_mov_b32 v114, v90\n" \
  "v_mov_b32 v115, v91\n" \
  "v_mov_b32 v116, v92\n" \
  "v_mov_b32 v117, v93\n" \
  "v_mov_b32 v118, v94\n" \
  "v_mov_b32 v119, v95\n" \
  "s_branch .Ldd8_%=\n" \
  ".Ldr10_%=:\n" \
  "s_waitcnt lgkmcnt(0)\n" \
  "v_mov_b32 v104, v80\n" \
  "v_mov_b32 v105, v81\n" \
  "v_mov_b32 v106, v82\n" \
  "v_mov_b32 v107, v83\n" \
  "v_mov_b32 v108, v84\n" \
  "v_mov_b32 v109, v85\n" \
  "v_mov_b32 v110, v86\n" \
  "v_mov_b32 v111, v87\n" \
  "v_mov_b32 v112, v88\n" \
  "v_mov_b32 v113, v89\n" \
  "v_mov_b32 v114, v90\n" \
  "v_mov_b32 v115, v91\n" \
  "v_mov_b32 v116, v92\n" \
  "v_mov_b32 v117, v93\n" \
  "v_mov_b32 v118, v94\n" \
  "v_mov_b32 v119, v95\n" \
  "s_branch .Ldd10_%=\n" \
  ".Lps11_%=:\n" \
  "s_load_dwordx2 s[78:79], %[pp], s80 glc\n" \
  "s_memrealtime s[60:61]\n" \
  "s_waitcnt lgkmcnt(0)\n" \
  "s_cmp_ge_u32 s78, %[call]\n" \
  "s_cbranch_scc1 .Lpb11_%=\n" \
  "s_sub_u32 s60, s60, %[t0]\n" \
  "s_cmp_gt_u32 s60, %[dl]\n" \
  "s_cbranch_scc0 .Lps11_%=\n" \
  "s_branch .Lpend_%=\n" \
  ".Lpr12_%=:\n" \
  "s_lshl_b32 s68, s67, 12\n" \
  "buffer_load_dwordx4 v[80:83], %[v16], %[rraw], s68 offen offset:0 sc0 sc1\n" \
  "buffer_load_dwordx4 v[84:87], %[v16], %[rraw], s68 offen offset:1024 sc0 sc1\n" \
  "buffer_load_dwordx4 v[88:91], %[v16], %[rraw], s68 offen offset:2048 sc0 sc1\n" \
  "buffer_load_dwordx4 v[92:95], %[v16], %[rraw], s68 offen offset:3072 sc0 sc1\n" \
  "s_branch .Lpd12_%=\n" \
  ".Ldr13_%=:\n" \
  "s_waitcnt lgkmcnt(0)\n" \
  "v_mov_b32 v104, v64\n" \
  "v_mov_b32 v105, v65\n" \
  "v_mov_b32 v106, v66\n" \
  "v_mov_b32 v107, v67\n" \
  "v_mov_b32 v108, v68\n" \
  "v_mov_b32 v109, v69\n" \
  "v_mov_b32 v110, v70\n" \
  "v_mov_b32 v111, v71\n" \
  "v_mov_b32 v112, v72\n" \
  "v_mov_b32 v113, v73\n" \
  "v_mov_b32 v114, v74\n" \
  "v_mov_b32 v115, v75\n" \
  "v_mov_b32 v116, v76\n" \
  "v_mov_b32 v117, v77\n" \
  "v_mov_b32 v118, v78\n" \
  "v_mov_b32 v119, v79\n" \
  "s_branch .Ldd13_%=\n" \
  ".Ldr15_%=:\n" \
  "s_waitcnt lgkmcnt(0)\n" \
  "v_mov_b32 v104, v64\n" \
  "v_mov_b32 v105, v65\n" \
  "v_mov_b32 v106, v66\n" \
  "v_mov_b32 v107, v67\n" \
  "v_mov_b32 v108, v68\n" \
  "v_mov_b32 v109, v69\n" \
  "v_mov_b32 v110, v70\n" \
  "v_mov_b32 v111, v71\n" \
  "v_mov_b32 v112, v72\n" \
  "v_mov_b32 v113, v73\n" \
  "v_mov_b32 v114, v74\n" \
  "v_mov_b32 v115, v75\n" \
  "v_mov_b32 v116, v76\n" \
  "v_mov_b32 v117, v77\n" \
  "v_mov_b32 v118, v78\n" \
  "v_mov_b32 v119, v79\n" \
  "s_branch .Ldd15_%=\n" \
  ".Lps16_%=:\n" \
  "s_load_dwordx2 s[78:79], %[pp], s80 glc\n" \
  "s_memrealtime s[60:61]\n" \
  "s_waitcnt lgkmcnt(0)\n" \
  "s_cmp_ge_u32 s78, %[call]\n" \
  "s_cbranch_scc1 .Lpb16_%=\n" \
  "s_sub_u32 s60, s60, %[t0]\n" \
  "s_cmp_gt_u32 s60, %[dl]\n" \
  "s_cbranch_scc0 .Lps16_%=\n" \
  "s_branch .Lpend_%=\n" \
  ".Lpr17_%=:\n" \
  "s_lshl_b32 s68, s66, 12\n" \
  "buffer_load_dwordx4 v[64:67], %[v16], %[rraw], s68 offen offset:0 sc0 sc1\n" \
  "buffer_load_dwordx4 v[68:71], %[v16], %[rraw], s68 offen offset:1024 sc0 sc1\n" \
  "buffer_load_dwordx4 v[72:75], %[v16], %[rraw], s68 offen offset:2048 sc0 sc1\n" \
  "buffer_load_dwordx4 v[76:79], %[v16], %[rraw], s68 offen offset:3072 sc0 sc1\n" \
  "s_branch .Lpd17_%=\n" \
  ".Ldr18_%=:\n" \
  "s_waitcnt lgkmcnt(0)\n" \
  "v_mov_b32 v104, v80\n" \
  "v_mov_b32 v105, v81\n" \
  "v_mov_b32 v106, v82\n" \
  "v_mov_b32 v107, v83\n" \
  "v_mov_b32 v108, v84\n" \
  "v_mov_b32 v109, v85\n" \
  "v_mov_b32 v110, v86\n" \
  "v_mov_b32 v111, v87\n" \
  "v_mov_b32 v112, v88\n" \
  "v_mov_b32 v113, v89\n" \
  "v_mov_b32 v114, v90\n" \
  "v_mov_b32 v115, v91\n" \
  "v_mov_b32 v116, v92\n" \
  "v_mov_b32 v117, v93\n" \
  "v_mov_b32 v118, v94\n" \
  "v_mov_b32 v119, v95\n" \
  "s_branch .Ldd18_%=\n" \
  ".Lpend_%=:\n" \
  "s_mov_b64 exec, -1\n" \
  "s_waitcnt vmcnt(0) lgkmcnt(0)\n" \
  ""

constexpr int NG = 8;
constexpr u32 DEADLINE = 50000000u;
#define MUS 3072
constexpr u32 MU = MUS;
constexpr size_t OFF_CNT = 0;
constexpr size_t OFF_GAM = 16384;
constexpr size_t OFF_SLOT = 196608;
constexpr size_t OFF_TICK = 57344;
constexpr size_t OFF_PFLAG = 65536;
constexpr size_t OFF_STAMP = 262144;
constexpr size_t OFF_PACK = 1u << 20;
constexpr size_t OFF_STAGE = 4u << 20;
constexpr size_t OFF_PROG = 524288;
constexpr size_t WS_NEED = OFF_STAGE + (32u << 20);
constexpr int LDS_W = 81920;
constexpr int NGEMM = 28;
constexpr int NPULL = 4;

#define NREP 2
#define GREP 8
#define H2IDX 4
#define H2T 280
#define CTM 0
#define RWID 0
#define GSEL src
#define RSC "sc1"
#define RIDX wv == 0
#define RWAVE idx - NGEMM
#define HIDX 8
#define GSH 0
#define ZD 1
__host__ __device__ inline u32 psrc(u32 rank, u32 g) { return ((0xb2u >> rank & 1u) ? 0x63207541u : 0x75416320u) >> (4 * (g & 7)) & 7u; }
#define PFS 128
#define SPM 3
#define PEI 3
#define PFI 12
#define WPF0 ((NGEMM - PFI) * NPF << 18)
#define XCT 8
struct Args { char* ws[NG]; const char* x; char* my; const char* w; char* out; u32 rank; };
#define PSTOUT
#define STOUT
#define STAMP(i)
#define TIMOUT(c)
#define TIMST()

__device__ __forceinline__ void st16_sys(void* p, u32x4 v) { asm volatile("global_store_dwordx4 %0, %1, off sc0 sc1\n\ts_nop 1" :: "v"(p), "v"(v) : "memory"); }
#define OUTSC ""
__device__ __forceinline__ void st16_out(void* p, u32x4 v) { asm volatile("global_store_dwordx4 %0, %1, off " OUTSC "\n\ts_nop 1" :: "v"(p), "v"(v) : "memory"); }
__device__ __forceinline__ void st12_sys(void* p, u32x3 v) { asm volatile("global_store_dwordx3 %0, %1, off sc0 sc1\n\ts_nop 1" :: "v"(p), "v"(v) : "memory"); }
__device__ __forceinline__ void st8_sys(void* p, u32x2 v) { asm volatile("global_store_dwordx2 %0, %1, off sc0 sc1" :: "v"(p), "v"(v) : "memory"); }
__device__ __forceinline__ void st4_sys(void* p, u32 v) { asm volatile("global_store_dword %0, %1, off sc0 sc1" :: "v"(p), "v"(v) : "memory"); }
__device__ __forceinline__ u32x2 ld8_sys(const void* p) { u32x2 v; asm volatile("global_load_dwordx2 %0, %1, off sc0 sc1\n\ts_waitcnt vmcnt(0)" : "=&v"(v) : "v"(p) : "memory"); return v; }
__device__ __forceinline__ u32 lds_ld(u32 addr) { u32 v; asm volatile("ds_read_b32 %0, %1\n\ts_waitcnt lgkmcnt(0)" : "=&v"(v) : "v"(addr) : "memory"); return v; }
__device__ __forceinline__ u32 rotl(u32 v, int s) { return (v << s) | (v >> (32 - s)); }
__device__ __forceinline__ u32x4 mk_rsrc(const void* p, u32 bytes) {
  u64 a = (u64)p; u32x4 r;
  r[0] = __builtin_amdgcn_readfirstlane((u32)a); r[1] = __builtin_amdgcn_readfirstlane((u32)(a >> 32) & 0xffffu); r[2] = bytes; r[3] = 0x00020000u; return r;
}
__device__ __forceinline__ u32 wave_max_u32(u32 v) {
  v = max(v, (u32)__builtin_amdgcn_update_dpp(0, (int)v, 0x111, 0xf, 0xf, true));
  v = max(v, (u32)__builtin_amdgcn_update_dpp(0, (int)v, 0x112, 0xf, 0xf, true));
  v = max(v, (u32)__builtin_amdgcn_update_dpp(0, (int)v, 0x114, 0xf, 0xf, true));
  v = max(v, (u32)__builtin_amdgcn_update_dpp(0, (int)v, 0x118, 0xf, 0xf, true));
  v = max(v, (u32)__builtin_amdgcn_update_dpp(0, (int)v, 0x142, 0xa, 0xf, true));
  v = max(v, (u32)__builtin_amdgcn_update_dpp(0, (int)v, 0x143, 0xc, 0xf, true));
  return (u32)__builtin_amdgcn_readlane((int)v, 63);
}
__device__ __forceinline__ bool expired(u32 t0) { return (u32)__builtin_amdgcn_s_memrealtime() - t0 > DEADLINE; }

#define CLOB8(p, a) #p #a "0", #p #a "1", #p #a "2", #p #a "3", #p #a "4", #p #a "5", #p #a "6", #p #a "7", #p #a "8", #p #a "9"

__global__ __launch_bounds__(256, 1) void ag_gemm_q(Args a) {
  __shared__ __attribute__((aligned(1024))) char smem[LDS_W + 1024];
  const int tid = threadIdx.x, lane = tid & 63, b = blockIdx.x;
  const int wv = __builtin_amdgcn_readfirstlane(tid >> 6);
  const u32 rank = a.rank;
  char* const my = a.my;
  const u32 t0 = (u32)__builtin_amdgcn_s_memrealtime();
  asm volatile("" :: "s"(a.x), "s"(my), "s"(rank), "s"(t0), "s"(a.ws[0]));
  __builtin_amdgcn_sched_barrier(0);
  STAMP(0);
  const int g = lane >> 4, c = lane & 15;
  u32 call;
  char* peer_ws;
  u32 wpf = WPF0 + ((((b >> 3) - NGEMM) * 4 + wv) * 64 + lane) * 128;
#define WPF4() do { u32 d0, d1, d2, d3; const char* q = a.w + wpf; asm volatile("global_load_dword %0, %4, off\n\tglobal_load_dword %1, %5, off\n\tglobal_load_dword %2, %6, off\n\t" \
    "global_load_dword %3, %7, off\n\ts_waitcnt vmcnt(0)" : "=&v"(d0), "=&v"(d1), "=&v"(d2), "=&v"(d3) \
    : "v"(q), "v"(q + (128 << 10)), "v"(q + (256 << 10)), "v"(q + (384 << 10)) : "memory"); wpf += 512 << 10; } while (0)
  const bool late = HEAD == 4 && ((u32)(b >> 3) - H2IDX) < 4;
  const u32 hp = ((b >> 3) - (late ? H2IDX : HIDX)) * 4 + wv;
  const bool head = hp < 14;
  const u32 hk = (hp & 1) + (late ? 2 : 0);
  char* hws;
  const int xg = b & 7, ks = (b >> 3) * 4 + wv;
  u32x4 v[4], h[4];
  const u32 xo = (xg * 64 + c) * 8192 + g * 16;
  asm volatile("s_cmp_ge_u32 %[hp], %[hn]\n\ts_cbranch_scc1 .Lh1_%=\n\t"
               "global_load_dwordx4 %5, %[o0], %[xh]\n\tglobal_load_dwordx4 %6, %[o1], %[xh]\n\tglobal_load_dwordx4 %7, %[o2], %[xh]\n\tglobal_load_dwordx4 %8, %[o3], %[xh]\n"
               ".Lh1_%=:\n\tglobal_load_dwordx4 %0, %[o0], %[xk]\n\tglobal_load_dwordx4 %1, %[o1], %[xk]\n\tglobal_load_dwordx4 %2, %[o2], %[xk]\n\t"
               "global_load_dwordx4 %3, %[o3], %[xk]\n\tglobal_load_dword %4, %[cn], off sc1"
               : "=&v"(v[0]), "=&v"(v[1]), "=&v"(v[2]), "=&v"(v[3]), "=&v"(call), "=&v"(h[0]), "=&v"(h[1]), "=&v"(h[2]), "=&v"(h[3])
               : [hp] "s"(hp), [hn] "i"(14), [xh] "s"(a.x + hk * 64), [xk] "s"(a.x + ks * 64), [o0] "v"(xo), [o1] "v"(xo + 16 * 8192), [o2] "v"(xo + 32 * 8192), [o3] "v"(xo + 48 * 8192),
                 [cn] "v"(my + OFF_CNT + b * 64) : "memory", "scc");
  __builtin_amdgcn_sched_barrier(0);
  const u32 idx = b >> 3, ct = CTM ? (idx < 16 ? idx & 7 : 8 + (idx - 16) % 6) : idx % 14;
  const u32 colA = ct * 128, colB = 1792 + (ct + (ct > 6)) * 16, colX = ct < XCT ? 1904 + (ct >> 2) * 128 : colB ^ 16;
  const u32 lds = (u32)(size_t)smem;
  const u32 r8 = lane >> 3, cp = lane & 7;
  u32 vd[4];
#pragma unroll
  for (int i = 0; i < 4; ++i) {
    const u32 k = 32 * (wv >> 1) + 8 * i + r8, fk = ((k >> 1) & 1) | (((k >> 3) & 1) << 1);
    vd[i] = k * 4096 + colA * 2 + 128 * (wv & 1) + 16 * (cp ^ (2 * fk)) - 1024 * i;
  }
  const u32 kb = 16 * wv + (lane >> 2);
  const u32 ch = (lane & 3) ^ (2 * ((kb >> 3) & 1));
  const u32 vd4 = kb * 4096 + (ch < 2 ? colB : colX) * 2 + 16 * (ch & 1) + (ZD && ch >= 2 && ct >= XCT ? 1u << 24 : 0u);
  const u32x4 wr = mk_rsrc(a.w, 16u << 20);
  const u32 ma = lds + 8192 * (wv & 1) + 4096 * (wv >> 1), mb = lds + 65536 + 1024 * wv;
  u32 sw, dm = 0, mode;
  asm("s_cmp_ge_u32 %1, %2\n\ts_cselect_b32 %0, 1, 0\n\ts_cmp_lt_u32 %3, %4\n\ts_cselect_b32 %0, 0, %0\n\ts_cmp_ge_u32 %1, %5\n\ts_cselect_b32 %0, 2, %0"
      : "=&s"(mode) : "s"(idx), "i"(PEI), "s"(hp), "i"(14), "i"(NGEMM) : "scc");
#define WPRE(when) asm volatile("s_cmp_lg_u32 %[md], " #when "\n\ts_cbranch_scc1 .Lnp_%=\n\ts_mov_b32 %[sw], 0\n" GEMM_PRE ".Lnp_%=:" : [sw] "=&s"(sw), [dm] "+v"(dm) : [md] "s"(mode), [wr] "s"(wr), [ma] "s"(ma), [mb] "s"(mb), [vd0] "v"(vd[0]), [vd1] "v"(vd[1]), [vd2] "v"(vd[2]), \
    [vd3] "v"(vd[3]), [vd4] "v"(vd4), [vpf] "v"(idx < PFI ? 1u << 24 : (((b - PFI * 8) * 4 + wv) * 64 + lane) * 128), [po0] "s"(0), [po1] "s"((NGEMM - PFI) << 18), [po2] "s"((NGEMM - PFI) << 19) : "memory", "scc")
  peer_ws = a.ws[0];
#pragma unroll
  for (int p = 1; p < NG; ++p) peer_ws = ((lane >> 2) & 7) == p ? a.ws[p] : peer_ws;
  hws = a.ws[(rank + 1 + hp / 2) & 7];
  char* const hst = hws + OFF_STAGE + ((size_t)(rank * 8 + xg) << 19) + hk * 4096;
  WPRE(1);
  asm volatile("s_cmp_ge_u32 %[hp], %[hn]\n\ts_cbranch_scc1 .Lh2_%=\n\t"
               "s_waitcnt vmcnt(8)\n\tglobal_store_dwordx4 %[l16], %5, %[hs] sc0 sc1\n\ts_waitcnt vmcnt(8)\n\tglobal_store_dwordx4 %[l16], %6, %[hs] offset:1024 sc0 sc1\n\t"
               "s_waitcnt vmcnt(8)\n\tglobal_store_dwordx4 %[l16], %7, %[hs] offset:2048 sc0 sc1\n\ts_waitcnt vmcnt(8)\n\tglobal_store_dwordx4 %[l16], %8, %[hs] offset:3072 sc0 sc1\n\t"
               "s_waitcnt vmcnt(4)\n\ts_branch .Lh3_%=\n.Lh2_%=:\n\ts_cmp_lg_u32 %[ea], 1\n\ts_waitcnt vmcnt(%[np])\n\ts_cbranch_scc0 .Lh3_%=\n\ts_waitcnt vmcnt(0)\n.Lh3_%=:"
               : "+v"(v[0]), "+v"(v[1]), "+v"(v[2]), "+v"(v[3]), "+v"(call), "+v"(h[0]), "+v"(h[1]), "+v"(h[2]), "+v"(h[3])
               : [hp] "s"(late ? 14 : hp), [hn] "i"(14), [l16] "v"(lane * 16), [hs] "s"(hst), [ea] "s"(mode), [np] "i"(NPRE) : "memory", "scc");
  call = __builtin_amdgcn_readfirstlane(call) + 1;
  u32 got = 0, ps = psrc(rank, __builtin_amdgcn_s_getreg(20 | 3 << 11) + ((b >> 3) < NGEMM ? GSH : 0));
  char* const slot = my + OFF_SLOT + (b >> 3) * 1024;
  const bool claim = tid == 0;
  if (claim) asm volatile("global_atomic_umax %0, %1, %2, off sc0 sc1" : "+v"(got) : "v"(slot + ps * 128), "v"(call) : "memory");
  char* stg = my + OFF_STAGE + ((size_t)(rank * 8 + xg) << 19) + ks * 4096 + lane * 16;
#pragma unroll
  for (int m = 0; m < 4; ++m) st16_sys(stg + m * 1024, v[m]);
  u16x2 mx = {0, 0};
#pragma unroll
  for (int m = 0; m < 4; ++m)
#pragma unroll
    for (int j = 0; j < 4; ++j) mx = __builtin_elementwise_max(mx, __builtin_bit_cast(u16x2, v[m][j] & 0x7fff7fffu));
  const u32 top = wave_max_u32(max((u32)mx[0], (u32)mx[1]));
  const u32 lo = top > 1023u ? top - 1023u : 0u;
  const u16x2 lo2 = {(unsigned short)lo, (unsigned short)lo};
  u32 hb[8], lb[8];
#pragma unroll
  for (int q = 0; q < 8; ++q) {
    const u32 e0 = v[q >> 1][(q & 1) * 2], e1 = v[q >> 1][(q & 1) * 2 + 1];
    const u32 d0 = __builtin_bit_cast(u32, (u16x2)(__builtin_bit_cast(u16x2, e0) - lo2));
    const u32 d1 = __builtin_bit_cast(u32, (u16x2)(__builtin_bit_cast(u16x2, e1) - lo2));
    lb[q] = __builtin_amdgcn_perm(d1, d0, 0x06040200u);
    hb[q] = __builtin_amdgcn_perm(d1, d0, 0x07050301u);
  }
  char* pk = my + OFF_PACK + (size_t)(xg * 128 + ks) * MU;
  st16_sys(pk + lane * 16, (u32x4){lb[0], lb[1], lb[2], lb[3]});
  st16_sys(pk + 1024 + lane * 16, (u32x4){lb[4], lb[5], lb[6], lb[7]});
  const u32 M = 0x83838383u;
  u32x3 H;
  H[0] = (hb[0] & M) | rotl(hb[1] & M, 3) | ((hb[6] & 0x03030303u) << 5);
  H[1] = (hb[2] & M) | rotl(hb[3] & M, 3) | ((hb[7] & 0x03030303u) << 5);
  H[2] = (hb[4] & M) | rotl(hb[5] & M, 3) | ((hb[6] & 0x80808080u) >> 2) | ((hb[7] & 0x80808080u) >> 1);
  st12_sys(pk + 2048 + lane * 12, H);
  u32 mask = 0;
#pragma unroll
  for (int q = 0; q < 8; ++q) mask |= (((hb[q] & 0x7c7c7c7cu) + 0x7c7c7c7cu) & 0x80808080u) >> (7 - q);
  unsigned short* el = (unsigned short*)(smem + LDS_W + wv * 128);
  u32 cnt = 0;
  while (true) {
    const u64 bal = __ballot(mask != 0);
    if (bal == 0) break;
    const u32 slot = cnt + __builtin_amdgcn_mbcnt_hi((u32)(bal >> 32), __builtin_amdgcn_mbcnt_lo((u32)bal, 0));
    cnt += (u32)__popcll(bal);
    if (cnt > 63u) break;
    if (mask != 0) {
      const u32 bit = __builtin_ctz(mask), q = bit & 7, by = bit >> 3;
      mask &= mask - 1;
      const u32 s0 = (q & 1) ? hb[1] : hb[0], s1 = (q & 1) ? hb[3] : hb[2], s2 = (q & 1) ? hb[5] : hb[4], s3 = (q & 1) ? hb[7] : hb[6];
      const u32 r0 = (q & 2) ? s1 : s0, r1 = (q & 2) ? s3 : s2;
      const u32 h = (q & 4) ? r1 : r0;
      const u32 up = (h >> (8 * by + 2)) & 31u;
      const u32 pos = (q >> 2) * 1024 + lane * 16 + (q & 3) * 4 + by;
      el[slot] = (unsigned short)(pos << 5 | up);
    }
  }
  if (lane < 32) st4_sys(pk + 2816 + lane * 4, lds_ld((u32)(size_t)el + lane * 4));
  u32* meta = (u32*)(smem + LDS_W + 512);
  if (lane == 0) meta[wv] = lo | (min(cnt, 255u) << 16);
  if (late && head) {
    while ((u32)__builtin_amdgcn_s_memrealtime() - t0 < H2T) {}
#pragma unroll
    for (int m = 0; m < 4; ++m) st16_sys(hst + lane * 16 + m * 1024, h[m]);
  }
  WPRE(0);
  asm volatile("s_cmp_eq_u32 %1, 0\n\ts_waitcnt vmcnt(%2)\n\ts_cbranch_scc1 .Lw_%=\n\ts_waitcnt vmcnt(0)\n.Lw_%=:" : "+v"(got) : "s"(mode), "i"(NPRE) : "memory", "scc");
  if (claim) {
    for (int t = 0; t < NG - 1 && got >= call; ++t) {
      ps = (ps + 1) & 7;
      asm volatile("global_atomic_umax %0, %1, %2, off sc0 sc1\n\ts_waitcnt vmcnt(0)" : "=&v"(got) : "v"(slot + ps * 128), "v"(call) : "memory");
    }
    meta[4] = ps;
  }
  asm volatile("s_waitcnt lgkmcnt(0)" ::: "memory");
  if (head && lane < NREP) st4_sys(hws + OFF_PROG + (rank * 8 + xg) * 4096 + (hk & 1) * 2048 + lane * 256 + (late ? 4 : 1024), call * 256 + hk);
  __builtin_amdgcn_s_barrier();
  asm volatile("" ::: "memory");
  if (tid < 32) {
    const u32 e = (rank * 8 + xg) * 128 + (b >> 3) * 4 + (lane & 3);
    st8_sys(peer_ws + OFF_PFLAG + e * 8, (u32x2){call, meta[lane & 3]});
  }
  if (tid == 64) st4_sys(my + OFF_CNT + b * 64, call);
  STAMP(1);
  const u32 src = __builtin_amdgcn_readfirstlane(lds_ld((u32)(size_t)smem + LDS_W + 528)) & 7;
  if (idx >= NGEMM) {
    const bool spare = src == rank;
    u32 wi = WPF0;
    static_assert(WPF0 % (512 << 10) == 0, "");
    const u32 wid = (idx - NGEMM) * 4 + wv, rg = wid & 7, par = wid >> 3;
    if (spare) {
      const char* pp = my + OFF_PFLAG + (src * 8 + rg) * 1024 + par * 8 + lane * 16;
      char* prog = my + OFF_PROG + (src * 8 + rg) * 4096 + par * 2048 + (lane ? lane * 4 : 1024);
      static_assert((HEAD == 2 || HEAD == 4) && NREP <= 4, "");
      bool todo = true;
      do {
        if (todo && ld8_sys(pp)[0] >= call) {
          todo = false;
#pragma unroll
          for (int r = 0; r < NREP; ++r) st4_sys(prog + r * 256, call * 256 + lane * 2 + par);
        }
        if (wi < (16u << 20) && (__ballot(todo) & SPM) == 0) { WPF4(); wi += 512 << 10; }
      } while (__ballot(todo) != 0 && !expired(t0));
    } else {
      const char* sws = a.ws[src];
      const size_t so = OFF_STAGE + ((size_t)src << 22);
      const u32x4 rpk = mk_rsrc(sws + OFF_PACK, 1024 * MU), rraw = mk_rsrc(sws + so, 1u << 22), rdst = mk_rsrc(my + so, 1u << 22);
      const u32x4 rpg = mk_rsrc(my + OFF_PROG + src * 32768, 32768);
      const u32 spl = (u32)(size_t)smem + wv * 2048;
      asm volatile(PULL_ASM
        : PSTOUT
        : [rpk] "s"(rpk), [rraw] "s"(rraw), [rdst] "s"(rdst), [rpg] "s"(rpg), [pp] "s"(my + OFF_PFLAG + src * 8192), [call] "s"(call), [t0] "s"(t0), [dl] "s"(DEADLINE),
          [u0] "s"(rg * 128 + par + HEAD), [spl] "s"(spl), [v16] "v"(lane * 16), [v12] "v"(lane * 12), [v2] "v"(lane * 2),
          [vpl] "v"(spl + lane * 16), [v4] "v"(lane * 256), [fex] "i"((1 << NREP) - 1), [tag] "s"(call * 256),
          [tk] "s"(my + OFF_TICK + src * 128), [wid] "s"(wid), [base] "s"((call - 1) * ((128 - HEAD) * 8 - 32) - 48)
        : "memory", "vcc", "scc", "v54", "v55", "v56", "v57", "v58", "v59", CLOB8(v, 6), CLOB8(v, 7), CLOB8(v, 8), CLOB8(v, 9), CLOB8(v, 10), CLOB8(v, 11), CLOB8(v, 12),
          CLOB8(s, 6), CLOB8(s, 7), CLOB8(s, 8));
      STAMP(2);
    }
    if (spare) for (; wi < (16u << 20); wi += 512 << 10) WPF4();
    if (wid == RWID) {
      const char* lp = my + OFF_GAM + 8192 + src * 2048 + min(lane, 55) * 16;
      char* gp = a.ws[lane & 7] + OFF_GAM + (lane >> 3) * 512 + (rank * 8 + src) * 8;
      u32 mx;
      for (u32 it = 1;; ++it) {
        u32x4 v0;
        asm volatile("global_load_dwordx4 %0, %1, off " RSC "\n\ts_waitcnt vmcnt(0)" : "=&v"(v0) : "v"(lp + (it & 63 ? 0 : 1024)) : "memory");
        mx = max(v0[1], v0[3]);
        if (__ballot(min(v0[0], v0[2]) >= call) == ~0ull) break;
        if (expired(t0)) { mx = 0x7fffffffu; break; }
      }
      mx = wave_max_u32(mx);
      if (lane < NG * GREP) st8_sys(gp, (u32x2){call, mx});
    }
    return;
  }
  const u32 rg = (CTM ? (idx < 16 ? idx >> 3 : idx >= 22) : idx / 14) * 4 + wv, mr = ct & 3;
  u32 live = call;
  {
    const u32 q = c >> 2, p = lane & 3;
    const u32 f = (q >> 1) | ((g & 1) << 1);
    const u32 ra = lds + 128 * (8 * g + q) + 8 * p, rb = lds + 65536 + 64 * (8 * g + q) + 8 * p;
    const u32x4 xr = mk_rsrc(my + OFF_STAGE + ((size_t)(src * 8 + rg) << 19), 1u << 19);
    const char* fp = my + OFF_PROG + (src * 8 + rg) * 4096 + ct % NREP * 256;
    sw = SW0;
    u32 sx = 0, sl = 0, tag = call * 256;
    const u32 ro = (src * 512 + rg * 64 + c) * 2048;
    const u32 ra0 = PSW == 2 ? (src * 512 + rg * 64 + (lane >> 3)) * 2048 + colA + 16 * (lane & 7) : ro + colA + 16 * g;
    asm volatile("; keeps %[dm] until the prefetch has landed\n" GEMM_ASM
      : STOUT [call] "+s"(live), [sx] "+s"(sx), [sw] "+s"(sw), [sl] "+s"(sl), [tag] "+s"(tag) TIMOUT(COMMA)
      : [xr] "s"(xr), [wr] "s"(wr), [fp] "s"(fp), [t0] "s"(t0), [dl] "s"(DEADLINE), [ma] "s"(ma), [mb] "s"(mb),
        [va0] "v"(ra + 32 * (0 ^ f)), [va1] "v"(ra + 32 * (1 ^ f)), [va2] "v"(ra + 32 * (2 ^ f)), [va3] "v"(ra + 32 * (3 ^ f)),
        [vb0] "v"(rb + 32 * (g & 1)), [vb1] "v"(rb + 32 * (1 ^ (g & 1))),
        [vd0] "v"(vd[0]), [vd1] "v"(vd[1]), [vd2] "v"(vd[2]), [vd3] "v"(vd[3]), [vd4] "v"(vd4),
        [vx0] "v"(lane * 16 + 1024 * mr), [vx1] "v"(lane * 16 + 1024 * (1 ^ mr)), [vx2] "v"(lane * 16 + 1024 * (2 ^ mr)), [vx3] "v"(lane * 16 + 1024 * (3 ^ mr)),
        [dm] "v"(dm),
        [callc] "s"(call), [gam] "s"(my + OFF_GAM), [out] "s"(a.out), [ct] "s"(ct),
        [vgo] "v"(GSEL % GREP * 512 + lane * 8), [vpub] "v"(8192 + src * 2048 + (idx * 4 + wv) * 8),
        [out2] "s"(a.out + 16384), [vw] "v"(lds + wv * 9216 + c * 144 + g * 16), [vr] "v"(lds + wv * 9216 + (lane >> 3) * 144 + (lane & 7) * 16),
        [vo0] "v"(ra0 + mr * 32768), [vo1] "v"(ra0 + (1 ^ mr) * 32768), [vo2] "v"(ra0 + (2 ^ mr) * 32768),
        [vo3] "v"(ra0 + (3 ^ mr) * 32768), [voB] "v"(ro + (g ^ mr) * 32768 + colB), [voX] "v"(ro + (3 ^ mr) * 32768 + colX + 4 * g)
      : "memory", "vcc", "scc", "s60", "s61", "s62", "s63", "s64", "s65", CLOB8(v, 6), CLOB8(v, 7), CLOB8(v, 8), CLOB8(v, 9), CLOB8(v, 10), CLOB8(v, 11), CLOB8(v, 12), CLOB8(v, 13), CLOB8(v, 14),
        CLOB8(v, 15), CLOB8(v, 16), CLOB8(v, 17), CLOB8(v, 18), CLOB8(v, 19), CLOB8(v, 20), CLOB8(v, 21), CLOB8(v, 22), CLOB8(v, 23), CLOB8(v, 24),
        "a0", "a1", "a2", "a3", "a4", "a5", "a6", "a7", "a8", "a9", CLOB8(a, 1), CLOB8(a, 2), CLOB8(a, 3), CLOB8(a, 4), CLOB8(a, 5), CLOB8(a, 6),
        CLOB8(a, 7), CLOB8(a, 8), CLOB8(a, 9), CLOB8(a, 10), CLOB8(a, 11), "a120", "a121", "a122", "a123", "a124", "a125", "a126", "a127");
  }
  STAMP(5);
}

extern "C" void kernel_launch(int rank, int nranks, ncclComm_t comm, hipStream_t stream,
                              void* const* d_in, const int* in_sizes, int n_in, void* d_out, int out_size,
                              const int* in_dtypes, int out_dtype, void* const* d_out_all,
                              void* const* d_ws_all, size_t ws_bytes) {
  if (nranks != NG || (unsigned)rank >= (unsigned)NG || ws_bytes < WS_NEED || n_in < 2 || in_sizes[0] != 512 * 4096 ||
      in_sizes[1] != 4096 * 2048 || out_size != 4096 * 2048) {
    fprintf(stderr, "kernel_launch: unexpected shapes\n");
    return;
  }
  Args a;
  for (int p = 0; p < NG; ++p) a.ws[p] = (char*)d_ws_all[p];
  a.my = a.ws[rank]; a.x = (const char*)d_in[0]; a.w = (const char*)d_in[1]; a.out = (char*)d_out; a.rank = (u32)rank;
  ag_gemm_q<<<256, 256, 0, stream>>>(a);
}
```
